```python
import numpy as np

kernel_name = "gpu_ag_gemm_m4096_k4096_n4096_f32_qe4m3"


ABSMAX_RTOL = 0.09

COMPUTE_DTYPE = "float32"
EPILOGUE = "quant_e4m3"
IN_QUANT = None
OUT_QUANT = "e4m3"
IN_DTYPES = {"x": "float32", "w_mat": "float32"}
WORLD_SIZE = 8

def setup_inputs():

    x = np.random.randn(4096, 4096).astype(np.float32)
    w_mat = (np.random.randn(4096, 4096) / np.sqrt(4096)).astype(np.float32)
    return {"x": x, "w_mat": w_mat}

def _snap_fp8(x, mant_bits, max_abs, min_normal):
    a = np.abs(x.astype(np.float64))
    _m, _e = np.frexp(a)
    q = np.where(a >= min_normal, np.ldexp(1.0, _e - 1 - mant_bits),
                 min_normal * 2.0 ** -mant_bits)
    snapped = np.minimum(np.round(a / q) * q, max_abs)
    return (np.sign(x) * snapped).astype(np.float32)
def _quant_dequant(y, amax):
    scale = amax / 448.0
    q = _snap_fp8(y / scale, 3, 448.0, 2.0 ** -6)
    return (q * scale).astype(np.float32)
def _epilogue(y):
    return _quant_dequant(y, np.abs(y).max())

def reference(x, w_mat):
    y = (x @ w_mat).astype(np.float32)
    return _epilogue(y)
```

```cpp
#include <hip/hip_runtime.h>
#include <rccl/rccl.h>
#include <cstdio>
#include <cstdint>

typedef unsigned u32x4 __attribute__((ext_vector_type(4)));
typedef unsigned u32x2 __attribute__((ext_vector_type(2)));
typedef float f32x4 __attribute__((ext_vector_type(4)));
typedef __bf16 bf16x8 __attribute__((ext_vector_type(8)));
typedef __bf16 bf16x2 __attribute__((ext_vector_type(2)));
typedef float f32x2 __attribute__((ext_vector_type(2)));

namespace {
constexpr int NR = 8;
constexpr int KDIM = 4096;
constexpr int SH = 512;
constexpr unsigned BLK = 4096;
constexpr unsigned UNIT_BYTES = 128 * BLK;
constexpr size_t STG_KIND_BYTES = (size_t)NR * 8 * UNIT_BYTES;
constexpr size_t FLAG_OFF = 2 * STG_KIND_BYTES;
constexpr size_t READY_OFF = FLAG_OFF;
constexpr size_t ARR_OFF = READY_OFF + NR * 16 * 128;
constexpr size_t AMAX_OFF = ARR_OFF + NR * 16 * 128;
constexpr size_t DONE_OFF = AMAX_OFF + NR * 128 * 8;
constexpr size_t CTR_OFF = DONE_OFF + 1024;
constexpr size_t TMO_OFF = CTR_OFF + 128;
constexpr size_t DBG_OFF = TMO_OFF + 128;
constexpr size_t WS_NEED = DBG_OFF + 512 * 16 * 8;
constexpr int NGEMM = 128, NPULL = 35, NWG = 256;
constexpr unsigned long long DEADLINE = 30000000ull;
constexpr unsigned STAGE_BYTES = 8 * BLK;
constexpr int NBUF = 4;

__device__ const unsigned TILE_TAB[NGEMM] = {16420,2145,5154,5283,14530,10241,8928,10432,69476,67649,7906,47971,70370,15553,17664,15424,3116,74817,8450,78019,77058,16577,113792,55360,18284,75905,12130,83011,3238,113377,121888,75872,70924,82785,73762,70820,13574,1059,3115,77856,77964,5158,78690,82660,69798,18147,7435,4311,8207,55430,3939,112708,78566,76,47243,47287,121935,75622,6307,75,112902,11404,113803,55351,1081,3112,73859,11115,10991,13484,4373,69751,9433,77064,10340,75979,12335,48876,9973,12377,12153,121928,12420,78923,76015,9271,67637,47225,17561,5167,74948,55518,79599,74935,73845,75833,83033,75023,6216,70846,11477,78103,77685,2234,2238,2173,12008,78590,69845,12381,47258,55386,8478,70813,73864,81950,9501,48349,68890,69850,68766,76669,78024,112734,10493,69373,75642,79034};
__device__ const unsigned char PULL_TAB[NPULL] = {16,24,25,28,29,39,40,41,46,47,51,56,57,62,63,65,66,68,69,70,80,90,91,94,95,103,106,107,108,109,115,122,123,124,125};
__device__ const unsigned char DEST_MASK[16] = {35,17,17,137,17,17,17,69,15,15,225,225,195,195,45,45};

struct Args {
  const float* x;
  const float* w;
  char* ws[NR];
  unsigned char* out[NR];
  int rank;
  int pad;
};

typedef __attribute__((address_space(1))) unsigned gu32;
typedef __attribute__((address_space(1))) unsigned long long gu64;
typedef __attribute__((address_space(3))) unsigned lu32;

__device__ __forceinline__ unsigned ld_sys(const void* p) { return __hip_atomic_load((const gu32*)p, __ATOMIC_RELAXED, __HIP_MEMORY_SCOPE_SYSTEM); }
__device__ __forceinline__ void st_sys(void* p, unsigned v) { __hip_atomic_store((gu32*)p, v, __ATOMIC_RELAXED, __HIP_MEMORY_SCOPE_SYSTEM); }
__device__ __forceinline__ unsigned ld_agent(const void* p) { return __hip_atomic_load((const gu32*)p, __ATOMIC_RELAXED, __HIP_MEMORY_SCOPE_AGENT); }
__device__ __forceinline__ void st_agent(void* p, unsigned v) { __hip_atomic_store((gu32*)p, v, __ATOMIC_RELAXED, __HIP_MEMORY_SCOPE_AGENT); }

__device__ __forceinline__ __amdgpu_buffer_rsrc_t mkrsrc(const void* p, unsigned bytes) {
  unsigned long long a = (unsigned long long)p;
  unsigned lo = __builtin_amdgcn_readfirstlane((unsigned)a), hi = __builtin_amdgcn_readfirstlane((unsigned)(a >> 32));
  void* q = (void*)(((unsigned long long)hi << 32) | lo);
  return __builtin_amdgcn_make_buffer_rsrc(q, 0, bytes, 0x00020000);
}
__device__ __forceinline__ size_t stg_off(int kind, int src, int unit) {
  return (size_t)kind * STG_KIND_BYTES + (size_t)(src * 8 + unit) * UNIT_BYTES;
}
__device__ __forceinline__ unsigned cvt2(float a, float b) {
  f32x2 v = {a, b};
  bf16x2 r = __builtin_convertvector(v, bf16x2);
  return __builtin_bit_cast(unsigned, r);
}
#define STAMP(i) do {} while (0)

__device__ __forceinline__ bool wave_wait(const void* p, unsigned thr, bool act, char* me, unsigned call) {
  unsigned long long t0 = 0;
  for (unsigned it = 0;; ++it) {
    unsigned v = act ? ld_sys(p) : 0xffffffffu;
    if (__all(v >= thr)) return true;
    if (it == 0) t0 = __builtin_amdgcn_s_memrealtime();
    if (ld_agent(me + TMO_OFF) == call || __builtin_amdgcn_s_memrealtime() - t0 > DEADLINE) {
      st_agent(me + TMO_OFF, call);
      return false;
    }
    __builtin_amdgcn_s_sleep(2);
  }
}

constexpr int AUX_SYS = 17, AUX_AGENT = 16;
#define GEMM_AUX 17
#define PRE_AUX 17
#define PULL_LAUX 17

#define STAGGER 15
__global__ __launch_bounds__(512, 2) void fused_ag_gemm(Args a) {
  __shared__ __attribute__((aligned(16))) char smem[NBUF * STAGE_BYTES + 64];
  const int tid = threadIdx.x, lane = tid & 63;
  const int wv = __builtin_amdgcn_readfirstlane(tid >> 6);
  const int rank = a.rank;
  const int b = blockIdx.x;
  char* me = a.ws[rank];
  const unsigned long long tstart = __builtin_amdgcn_s_memrealtime();
  STAMP(0);
  const unsigned call = __builtin_amdgcn_readfirstlane(ld_agent(me + CTR_OFF)) + 1;
  const unsigned base = call * 64;

  {
    const int task = b >= NGEMM ? (b < NGEMM + NPULL ? b - NGEMM : b) : b + NPULL;
    const int c16 = task >> 4, u16 = task & 15, kind = u16 >> 3, unit = u16 & 7;
    char* dst = me + stg_off(kind, rank, unit);
    const __amdgpu_buffer_rsrc_t rd = mkrsrc(dst, UNIT_BYTES);
    const int ks = c16 * 8 + wv;
    const int kq = lane >> 4, l15 = lane & 15;
    if (STAGGER > 0) {
      while (__builtin_amdgcn_s_memrealtime() - tstart < (unsigned long long)(c16 * STAGGER)) __builtin_amdgcn_s_sleep(1);
    }
    f32x4 v[8];
    if (kind == 0) {
      const float* p = a.x + (size_t)(unit * 64 + l15) * KDIM + ks * 32 + 4 * kq;
#pragma unroll
      for (int f = 0; f < 4; f++) {
        v[2 * f] = *(const f32x4*)(p + (size_t)f * 16 * KDIM);
        v[2 * f + 1] = *(const f32x4*)(p + (size_t)f * 16 * KDIM + 16);
      }
#pragma unroll
      for (int f = 0; f < 4; f++) {
        u32x4 o = {cvt2(v[2 * f][0], v[2 * f][1]), cvt2(v[2 * f][2], v[2 * f][3]),
                   cvt2(v[2 * f + 1][0], v[2 * f + 1][1]), cvt2(v[2 * f + 1][2], v[2 * f + 1][3])};
        __builtin_amdgcn_raw_buffer_store_b128(o, rd, ks * BLK + f * 1024 + lane * 16, 0, AUX_SYS);
      }
    } else {
      const float* p = a.w + (size_t)(ks * 32 + 4 * kq) * SH + unit * 64 + 4 * l15;
#pragma unroll
      for (int i = 0; i < 8; i++) v[i] = *(const f32x4*)(p + (size_t)((i & 3) + (i >> 2) * 16) * SH);
      const int q = l15 >> 2, j = l15 & 3;
#pragma unroll
      for (int r = 0; r < 4; r++) {
        u32x4 o = {cvt2(v[0][r], v[1][r]), cvt2(v[2][r], v[3][r]), cvt2(v[4][r], v[5][r]), cvt2(v[6][r], v[7][r])};
        __builtin_amdgcn_raw_buffer_store_b128(o, rd, ks * BLK + r * 1024 + kq * 256 + q * 64 + ((j ^ r) * 16), 0, AUX_SYS);
      }
    }
    asm volatile("s_waitcnt vmcnt(0)" ::: "memory");
    __builtin_amdgcn_s_barrier();
    if (tid < NR) {
      if ((DEST_MASK[u16] >> tid) & 1)
        st_sys(a.ws[rank ^ tid] + READY_OFF + (size_t)((rank * 2 + kind) * 8 + unit) * 128 + c16 * 4, base + 1);
    }
  }
  STAMP(1);

  if (b >= NGEMM + NPULL) {
    if (b == NWG - 1 && wv == 0) {
      bool ok = true;
#pragma unroll
      for (int i = 0; i < NGEMM / 64; i++) {
        const int c = i * 64 + lane;
        ok = wave_wait(me + DONE_OFF + c * 4, call, ((TILE_TAB[c] >> 2) & 7) != 0, me, call) && ok;
      }
      if (lane == 0) {
        if (!ok) a.out[rank][0] = 0x7f;
        st_agent(me + CTR_OFF, call);
      }
      STAMP(6);
    }
    return;
  }
  if (b >= NGEMM) {
    if (wv >= 4) return;
    const unsigned e = PULL_TAB[b - NGEMM];
    const int t = e >> 4, kind = (e >> 3) & 1, unit = e & 7;
    const unsigned src = (unsigned)(rank ^ t);
    if (src >= NR) return;
    const __amdgpu_buffer_rsrc_t rs = mkrsrc(a.ws[src] + stg_off(kind, src, unit), UNIT_BYTES);
    const __amdgpu_buffer_rsrc_t rd = mkrsrc(me + stg_off(kind, src, unit), UNIT_BYTES);
    const char* rdy = me + READY_OFF + (size_t)((src * 2 + kind) * 8 + unit) * 128;
    char* flag = me + ARR_OFF + (size_t)((src * 2 + kind) * 8 + unit) * 128 + wv * 4;
    const unsigned off0 = wv * BLK + lane * 16;
    int nready = 0;
    bool ok = true;
    auto need = [&](int c) {
      const int c2 = c >> 1;
      if (c2 < nready || !ok) return;
      unsigned long long t0 = 0;
      for (unsigned it = 0;; ++it) {
        unsigned v = lane < 16 ? ld_sys(rdy + lane * 4) : 0u;
        unsigned long long m = __ballot(v >= base + 1);
        nready = __builtin_ctzll(~m);
        if (c2 < nready) return;
        if (it == 0) t0 = __builtin_amdgcn_s_memrealtime();
        if (ld_agent(me + TMO_OFF) == call || __builtin_amdgcn_s_memrealtime() - t0 > DEADLINE) {
          st_agent(me + TMO_OFF, call); ok = false; return;
        }
        __builtin_amdgcn_s_sleep(2);
      }
    };
    u32x4 A[4], B[4];
    need(0);
    STAMP(2);
#pragma unroll
    for (int u = 0; u < 4; u++) A[u] = __builtin_amdgcn_raw_buffer_load_b128(rs, off0 + u * 1024, 0, AUX_SYS);
    for (int c = 0; c < 32; c += 2) {
      need(c + 1);
      {
        const unsigned o = off0 + (c + 1) * 4 * BLK;
#pragma unroll
        for (int u = 0; u < 4; u++) B[u] = __builtin_amdgcn_raw_buffer_load_b128(rs, o + u * 1024, 0, AUX_SYS);
        const unsigned od = off0 + c * 4 * BLK;
#pragma unroll
        for (int u = 0; u < 4; u++) __builtin_amdgcn_raw_buffer_store_b128(A[u], rd, od + u * 1024, 0, AUX_AGENT);
        asm volatile("s_waitcnt vmcnt(8)" ::: "memory");
        if (c > 0 && lane == 0) st_agent(flag, base + c);
      }
      const int cn = c + 2 < 32 ? c + 2 : 31;
      need(cn);
      {
        const unsigned o = off0 + cn * 4 * BLK;
#pragma unroll
        for (int u = 0; u < 4; u++) A[u] = __builtin_amdgcn_raw_buffer_load_b128(rs, o + u * 1024, 0, AUX_SYS);
        const unsigned od = off0 + (c + 1) * 4 * BLK;
#pragma unroll
        for (int u = 0; u < 4; u++) __builtin_amdgcn_raw_buffer_store_b128(B[u], rd, od + u * 1024, 0, AUX_AGENT);
        asm volatile("s_waitcnt vmcnt(8)" ::: "memory");
        if (lane == 0) st_agent(flag, base + c + 1);
      }
    }
    asm volatile("s_waitcnt vmcnt(0)" ::: "memory");
    if (lane == 0) st_agent(flag, base + 32);
    STAMP(3);
    return;
  }

  const unsigned te = TILE_TAB[b];
  const int t1 = (te >> 14) & 7, i1 = (te >> 11) & 7, t2 = (te >> 8) & 7, i2 = (te >> 5) & 7, tu = (te >> 2) & 7, pj = te & 3;
  const int srcW = rank ^ tu;
  lu32* sbad = (lu32*)(smem + NBUF * STAGE_BYTES);
  lu32* sm = (lu32*)(smem + NBUF * STAGE_BYTES + 16);
  if (tid == 0) *sbad = 0;
  if (wv >= 4) {
    const int lw = wv - 4;
    auto unit_of = [&](int u, int& kind, int& src, int& unit) {
      kind = u >> 1;
      src = u == 0 ? rank ^ t1 : (u == 1 ? rank ^ t2 : srcW);
      unit = u == 0 ? i1 : (u == 1 ? i2 : 2 * pj + (u & 1));
    };
    int kind, src, unit;
    unit_of(lw, kind, src, unit);
    const char* g = me + stg_off(kind, src, unit) + lane * 16;
    const char* pollp; bool pollLocal;
    {
      int k2, s2, u2;
      unit_of(lane >> 4, k2, s2, u2);
      pollLocal = s2 == rank;
      pollp = me + (pollLocal ? READY_OFF : ARR_OFF) + (size_t)(((s2 & 7) * 2 + k2) * 8 + u2) * 128 + (pollLocal ? (lane & 15) : (lane & 3)) * 4;
    }
    int avail = 0;
    bool wok = true;
    for (int s = -3; s < 64; s++) {
      if (lw == 0 && s + 3 < 64 && wok) {
        const int c = (s + 3) >> 1;
        if (c >= avail) {
          unsigned long long t0 = 0;
          for (unsigned it = 0;; ++it) {
            const unsigned v = ld_sys(pollp);
            int pr = pollLocal ? (v >= base + 1 ? 64 : 2 * (lane & 15)) : (v >= base ? (int)(v - base) : 0);
#pragma unroll
            for (int o = 32; o >= 1; o >>= 1) pr = min(pr, __shfl_xor(pr, o));
            avail = __builtin_amdgcn_readfirstlane(pr);
            if (c < avail) break;
            if (it == 0) t0 = __builtin_amdgcn_s_memrealtime();
            if (ld_agent(me + TMO_OFF) == call || __builtin_amdgcn_s_memrealtime() - t0 > DEADLINE) {
              st_agent(me + TMO_OFF, call); wok = false; break;
            }
            __builtin_amdgcn_s_sleep(1);
          }
        }
      }
      __builtin_amdgcn_s_barrier();
      asm volatile("" ::: "memory");
      if (s + 3 < 64) {
        char* l = smem + ((s + 3) & (NBUF - 1)) * STAGE_BYTES + lw * (2 * BLK);
        const char* gs = g + (size_t)(s + 3) * (2 * BLK);
#pragma unroll
        for (int i = 0; i < 8; i++)
          __builtin_amdgcn_global_load_lds((const unsigned*)(gs + i * 1024), (unsigned*)(l + i * 1024), 16, 0, AUX_SYS);
        asm volatile("s_waitcnt vmcnt(16)" ::: "memory");
      } else if (s == 61) {
        asm volatile("s_waitcnt vmcnt(8)" ::: "memory");
      } else {
        asm volatile("s_waitcnt vmcnt(0)" ::: "memory");
      }
    }
    if (!wok && lane == 0) *sbad = 1;
    asm volatile("s_waitcnt lgkmcnt(0)" ::: "memory");
    __builtin_amdgcn_s_barrier();
    return;
  }
  const int xu = wv >> 1, wu = wv & 1;
  f32x4 acc[4][4];
#pragma unroll
  for (int f = 0; f < 4; f++)
#pragma unroll
    for (int j = 0; j < 4; j++) acc[f][j] = f32x4{0.f, 0.f, 0.f, 0.f};
  const unsigned xoff = xu * (2 * BLK) + lane * 16;
  unsigned woff[4];
  {
    const int r = lane & 3, q = (lane >> 2) & 3, kq = lane >> 4;
#pragma unroll
    for (int j = 0; j < 4; j++) woff[j] = (2 + wu) * (2 * BLK) + r * 1024 + kq * 256 + q * 64 + ((j ^ r) * 16);
  }
  for (int s = -3; s < 64; s++) {
    asm volatile("s_waitcnt lgkmcnt(0)" ::: "memory");
    __builtin_amdgcn_s_barrier();
    asm volatile("" ::: "memory");
    if (s == 0) STAMP(2);
    if (s < 0) continue;
    const char* sb = smem + (s & (NBUF - 1)) * STAGE_BYTES;
#pragma unroll
    for (int ks = 0; ks < 2; ks++) {
      bf16x8 xf[4], wf[4];
#pragma unroll
      for (int f = 0; f < 4; f++) xf[f] = *(const bf16x8*)(sb + xoff + ks * BLK + f * 1024);
#pragma unroll
      for (int j = 0; j < 4; j++) wf[j] = *(const bf16x8*)(sb + woff[j] + ks * BLK);
#pragma unroll
      for (int f = 0; f < 4; f++)
#pragma unroll
        for (int j = 0; j < 4; j++)
          acc[f][j] = __builtin_amdgcn_mfma_f32_16x16x32_bf16(wf[j], xf[f], acc[f][j], 0, 0, 0);
    }
  }
  STAMP(3);
  unsigned m = 0;
#pragma unroll
  for (int f = 0; f < 4; f++)
#pragma unroll
    for (int j = 0; j < 4; j++)
#pragma unroll
      for (int r = 0; r < 4; r++) { const float t = acc[f][j][r]; m = max(m, __float_as_uint(t) & 0x7fffffffu); }
#pragma unroll
  for (int o = 32; o >= 1; o >>= 1) m = max(m, (unsigned)__shfl_xor((int)m, o));
  if (lane == 0) sm[wv] = m;
  asm volatile("s_waitcnt lgkmcnt(0)" ::: "memory");
  __builtin_amdgcn_s_barrier();
  asm volatile("" ::: "memory");
  m = max(max(sm[0], sm[1]), max(sm[2], sm[3]));
  bool bad = *sbad != 0;
  if (tid < NR) {
    gu64* g = (gu64*)(a.ws[tid] + AMAX_OFF + (size_t)(rank * NGEMM + b) * 8);
    __hip_atomic_store(g, ((unsigned long long)call << 32) | m, __ATOMIC_RELAXED, __HIP_MEMORY_SCOPE_SYSTEM);
  }
  unsigned gm = 0;
  {
    const char* gp = me + AMAX_OFF + tid * 32;
    unsigned long long t0 = 0;
    for (unsigned it = 0;; ++it) {
      u32x4 g0, g1;
      asm volatile("global_load_dwordx4 %0, %2, off sc0 sc1\n\tglobal_load_dwordx4 %1, %2, off offset:16 sc0 sc1\n\t"
                   "s_waitcnt vmcnt(0)"
                   : "=&v"(g0), "=&v"(g1) : "v"(gp) : "memory");
      const bool okk = g0[1] == call && g0[3] == call && g1[1] == call && g1[3] == call;
      if (__all(okk)) {
        gm = max(max(g0[0], g0[2]), max(g1[0], g1[2]));
        break;
      }
      if (it == 0) t0 = __builtin_amdgcn_s_memrealtime();
      if (ld_agent(me + TMO_OFF) == call || __builtin_amdgcn_s_memrealtime() - t0 > DEADLINE) {
        st_agent(me + TMO_OFF, call);
        bad = true;
        break;
      }
      __builtin_amdgcn_s_sleep(1);
    }
  }
#pragma unroll
  for (int o = 32; o >= 1; o >>= 1) gm = max(gm, (unsigned)__shfl_xor((int)gm, o));
  if (lane == 0) sm[4 + wv] = gm;
  if (bad && lane == 0) *sbad = 1;
  __syncthreads();
  gm = max(max(sm[4], sm[5]), max(sm[6], sm[7]));
  bad = *sbad != 0 || ld_agent(me + TMO_OFF) == call;
  STAMP(4);
  const float inv = 448.0f / __uint_as_float(gm);
  {
    const int tx = xu == 0 ? t1 : t2, ix = xu == 0 ? i1 : i2;
    unsigned char* o = a.out[srcW & 7] + (size_t)((rank ^ tx) * SH + ix * 64 + (lane & 15)) * SH + pj * 128 + wu * 64 + (lane >> 4) * 16;
#pragma unroll
    for (int f = 0; f < 4; f++) {
      u32x4 q;
#pragma unroll
      for (int j = 0; j < 4; j++) {
        int w = __builtin_amdgcn_cvt_pk_fp8_f32(acc[f][j][0] * inv, acc[f][j][1] * inv, 0, false);
        w = __builtin_amdgcn_cvt_pk_fp8_f32(acc[f][j][2] * inv, acc[f][j][3] * inv, w, true);
        q[j] = bad ? 0x7f7f7f7fu : (unsigned)w;
      }
      if (tu != 0) asm volatile("global_store_dwordx4 %0, %1, off sc0 sc1\n\ts_nop 1" ::"v"(o + (size_t)f * 16 * SH), "v"(q) : "memory");
      else *(u32x4*)(o + (size_t)f * 16 * SH) = q;
    }
  }
  if (tu != 0) {
    asm volatile("s_waitcnt vmcnt(0)" ::: "memory");
    __builtin_amdgcn_s_barrier();
    if (tid == 0) st_sys(a.ws[srcW & 7] + DONE_OFF + b * 4, call);
  }
  STAMP(5);
}
}

extern "C" void kernel_launch(int rank, int nranks, ncclComm_t comm, hipStream_t stream, void* const* d_in,
                              const int* in_sizes, int n_in, void* d_out, int out_size, const int* in_dtypes,
                              int out_dtype, void* const* d_out_all, void* const* d_ws_all, size_t ws_bytes) {
  (void)comm; (void)in_sizes; (void)n_in; (void)d_out; (void)out_size; (void)in_dtypes; (void)out_dtype;
  if (nranks != NR || rank < 0 || rank >= NR || ws_bytes < WS_NEED) {
    fprintf(stderr, "unsupported configuration\n");
    return;
  }
  Args a;
  a.x = (const float*)d_in[0];
  a.w = (const float*)d_in[1];
  for (int p = 0; p < NR; p++) {
    a.ws[p] = (char*)d_ws_all[p];
    a.out[p] = (unsigned char*)d_out_all[p];
  }
  a.rank = rank;
  a.pad = 0;
  fused_ag_gemm<<<NWG, 512, 0, stream>>>(a);
  hipError_t e = hipGetLastError();
  if (e != hipSuccess) fprintf(stderr, "launch error %s\n", hipGetErrorString(e));
}
```

```cpp
#include <hip/hip_runtime.h>
#include <rccl/rccl.h>
#include <cstdio>
#include <cstdint>

typedef unsigned u32x4 __attribute__((ext_vector_type(4)));
typedef float f32x4 __attribute__((ext_vector_type(4)));
typedef __bf16 bf16x8 __attribute__((ext_vector_type(8)));
typedef __bf16 bf16x2 __attribute__((ext_vector_type(2)));
typedef float f32x2 __attribute__((ext_vector_type(2)));

#define NPW 4
#define NPUSH 1
#define STAGGER 15

namespace {
constexpr int NR = 8;
constexpr int KDIM = 4096;
constexpr int SH = 512;
constexpr unsigned BLK = 4096;
constexpr unsigned UNIT_BYTES = 128 * BLK;
constexpr unsigned PBLK = 3072;
constexpr unsigned PCHUNK = 4 * PBLK + 4 * 64;
constexpr unsigned PUNIT_BYTES = 32 * PCHUNK;
constexpr size_t STG_KIND_BYTES = (size_t)NR * 8 * UNIT_BYTES;
constexpr size_t EXP_OFF = 2 * STG_KIND_BYTES;
constexpr size_t FLAG_OFF = EXP_OFF + (size_t)16 * PUNIT_BYTES;
constexpr size_t READY_OFF = FLAG_OFF;
constexpr size_t ARR_OFF = READY_OFF + NR * 16 * 128;
constexpr size_t AMAX_OFF = ARR_OFF + NR * 16 * 128;
constexpr size_t DONE_OFF = AMAX_OFF + NR * 128 * 8;
constexpr size_t CTR_OFF = DONE_OFF + 1024;
constexpr size_t TMO_OFF = CTR_OFF + 128;
constexpr size_t QCTR_OFF = TMO_OFF + 128;
constexpr size_t DBG_OFF = QCTR_OFF + 16 * 128;
constexpr size_t WS_NEED = DBG_OFF + 512 * 16 * 8;
static_assert(FLAG_OFF % 128 == 0, "align");
constexpr int NGEMM = 128, NPULL = 35, NWG = 256;
constexpr unsigned long long DEADLINE = 30000000ull;
constexpr unsigned STAGE_BYTES = 8 * BLK;
#define NBUFS 4
constexpr int NBUF = NBUFS;
constexpr int PF = NBUF - 1;

__constant__ unsigned TILE_TAB[NGEMM] = {16420,2145,5154,5283,14530,10241,8928,10432,69476,67649,7906,47971,70370,15553,17664,15424,3116,74817,8450,78019,77058,16577,113792,55360,18284,75905,12130,83011,3238,113377,121888,75872,70924,82785,73762,70820,13574,1059,3115,77856,77964,5158,78690,82660,69798,18147,7435,4311,8207,55430,3939,112708,78566,76,47243,47287,121935,75622,6307,75,112902,11404,113803,55351,1081,3112,73859,11115,10991,13484,4373,69751,9433,77064,10340,75979,12335,48876,9973,12377,12153,121928,12420,78923,76015,9271,67637,47225,17561,5167,74948,55518,79599,74935,73845,75833,83033,75023,6216,70846,11477,78103,77685,2234,2238,2173,12008,78590,69845,12381,47258,55386,8478,70813,73864,81950,9501,48349,68890,69850,68766,76669,78024,112734,10493,69373,75642,79034};
__constant__ unsigned PULL_TAB[8] = {0, 0xdc980, 0xfe987, 0xfe983, 0x65421, 0xfeba0, 0xdcba7, 0xdcba3};
constexpr unsigned long long DEST_MASK_X = 0x4511111189111123ull, DEST_MASK_W = 0x2d2dc3c3e1e10f0full;

struct Args {
  const float* x;
  const float* w;
  char* ws[NR];
  unsigned char* out[NR];
  int rank;
  int pad;
};

typedef __attribute__((address_space(1))) unsigned gu32;
typedef __attribute__((address_space(1))) unsigned long long gu64;
typedef __attribute__((address_space(3))) unsigned lu32;
typedef __attribute__((address_space(3))) unsigned char lu8;
typedef __attribute__((address_space(3))) u32x4 lu32x4;

__device__ __forceinline__ unsigned ld_sys(const void* p) { return __hip_atomic_load((const gu32*)p, __ATOMIC_RELAXED, __HIP_MEMORY_SCOPE_SYSTEM); }
__device__ __forceinline__ void st_sys(void* p, unsigned v) { __hip_atomic_store((gu32*)p, v, __ATOMIC_RELAXED, __HIP_MEMORY_SCOPE_SYSTEM); }
__device__ __forceinline__ unsigned ld_agent(const void* p) { return __hip_atomic_load((const gu32*)p, __ATOMIC_RELAXED, __HIP_MEMORY_SCOPE_AGENT); }
__device__ __forceinline__ void st_agent(void* p, unsigned v) { __hip_atomic_store((gu32*)p, v, __ATOMIC_RELAXED, __HIP_MEMORY_SCOPE_AGENT); }

__device__ __forceinline__ __amdgpu_buffer_rsrc_t mkrsrc(const void* p, unsigned bytes) {
  unsigned long long a = (unsigned long long)p;
  unsigned lo = __builtin_amdgcn_readfirstlane((unsigned)a), hi = __builtin_amdgcn_readfirstlane((unsigned)(a >> 32));
  void* q = (void*)(((unsigned long long)hi << 32) | lo);
  return __builtin_amdgcn_make_buffer_rsrc(q, 0, bytes, 0x00020000);
}
__device__ __forceinline__ u32x4 mkrsrc_words(const void* p, unsigned bytes) {
  unsigned long long a = (unsigned long long)p;
  return u32x4{(unsigned)__builtin_amdgcn_readfirstlane((unsigned)a), (unsigned)__builtin_amdgcn_readfirstlane((unsigned)(a >> 32)) & 0xffffu,
               bytes, 0x00020000u};
}
__device__ __forceinline__ size_t stg_off(int kind, int src, int unit) {
  return (size_t)kind * STG_KIND_BYTES + (size_t)(src * 8 + unit) * UNIT_BYTES;
}
__device__ __forceinline__ unsigned cvt2(float a, float b) {
  f32x2 v = {a, b};
  bf16x2 r = __builtin_convertvector(v, bf16x2);
  return __builtin_bit_cast(unsigned, r);
}
__device__ __forceinline__ unsigned perm(unsigned hi, unsigned lo, unsigned sel) { return __builtin_amdgcn_perm(hi, lo, sel); }
__device__ __forceinline__ unsigned wave_max_u32(unsigned v) {
  v = max(v, (unsigned)__builtin_amdgcn_update_dpp(0, (int)v, 0x111, 0xf, 0xf, false));
  v = max(v, (unsigned)__builtin_amdgcn_update_dpp(0, (int)v, 0x112, 0xf, 0xf, false));
  v = max(v, (unsigned)__builtin_amdgcn_update_dpp(0, (int)v, 0x114, 0xf, 0xf, false));
  v = max(v, (unsigned)__builtin_amdgcn_update_dpp(0, (int)v, 0x118, 0xf, 0xf, false));
  v = max(v, (unsigned)__builtin_amdgcn_update_dpp(0, (int)v, 0x142, 0xa, 0xf, false));
  v = max(v, (unsigned)__builtin_amdgcn_update_dpp(0, (int)v, 0x143, 0xc, 0xf, false));
  return (unsigned)__builtin_amdgcn_readlane((int)v, 63);
}
#define STAMP(i) do {} while (0)
#define WSTAMP(i) do {} while (0)

struct Spin {
  unsigned it = 0;
  unsigned long long t0 = 0;
  __device__ __forceinline__ bool expired(char* me, unsigned call) {
    if (it == 0) t0 = __builtin_amdgcn_s_memrealtime();
    if ((++it & 15) == 0) {
      if (ld_agent(me + TMO_OFF) == call || __builtin_amdgcn_s_memrealtime() - t0 > DEADLINE) {
        st_agent(me + TMO_OFF, call);
        return true;
      }
    }
    __builtin_amdgcn_s_sleep(1);
    return false;
  }
};
__device__ __forceinline__ bool wave_wait(const void* p, unsigned thr, bool act, char* me, unsigned call) {
  Spin sp;
  for (;;) {
    unsigned v = act ? ld_sys(p) : 0xffffffffu;
    if (__all(v >= thr)) return true;
    if (sp.expired(me, call)) return false;
  }
}

constexpr int AUX_SYS = 17, AUX_AGENT = 16;

__device__ __forceinline__ unsigned piece_off(int kind, int p, int lane) {
  const int kq = lane >> 4, q = (lane >> 2) & 3, j = lane & 3;
  return kind == 0 ? p * 1024 + lane * 16 : p * 1024 + kq * 256 + q * 64 + ((j ^ p) * 16);
}

__global__ __launch_bounds__(512, 2) void fused_ag_gemm(Args a) {
  __shared__ __attribute__((aligned(16))) char smem[NBUF * STAGE_BYTES];
  const int tid = threadIdx.x, lane = tid & 63;
  const int wv = __builtin_amdgcn_readfirstlane(tid >> 6);
  const int rank = a.rank;
  const int b = blockIdx.x;
  char* me = a.ws[rank];
  const unsigned long long tstart = __builtin_amdgcn_s_memrealtime();
  STAMP(0);
  const unsigned ctr_v = ld_agent(me + CTR_OFF);
  const unsigned te = TILE_TAB[b & (NGEMM - 1)];
  const bool is_pull = b >= NGEMM && b < NGEMM + NPULL;
  const int plink = is_pull ? 1 + (b - NGEMM) / 5 : 0;
  const unsigned ue = PULL_TAB[plink];
  char* const wsl = a.ws[lane & 7];
  unsigned char* const outp = a.out[(rank ^ (int)(te >> 2)) & 7];
  const u32x4 rfs = mkrsrc_words(me + FLAG_OFF, (unsigned)(DBG_OFF - FLAG_OFF));
  const unsigned l0 = lane == 0 ? 0u : 0x40000000u;
  unsigned call, base;

  {
    constexpr int NH0 = 16 * NPUSH;
    const int hb = b - (NGEMM + NPULL);
    const int task = b < NGEMM ? b + NH0 + NPULL : (b < NGEMM + NPULL ? b - NGEMM + NH0 : (hb < NH0 ? hb : b));
    const int c16 = task >> 4, u16 = task & 15, kind = u16 >> 3, unit = u16 & 7;
    const __amdgpu_buffer_rsrc_t rd = mkrsrc(me + stg_off(kind, rank, unit), UNIT_BYTES);
    const int ks = c16 * 8 + wv;
    const int kq = lane >> 4, l15 = lane & 15;
    if (STAGGER > 0) {
      while (__builtin_amdgcn_s_memrealtime() - tstart < (unsigned long long)(c16 * STAGGER)) __builtin_amdgcn_s_sleep(1);
    }
    f32x4 v[8];
    if (kind == 0) {
      const float* p = a.x + (size_t)(unit * 64 + l15) * KDIM + ks * 32 + 4 * kq;
#pragma unroll
      for (int f = 0; f < 4; f++) {
        v[2 * f] = *(const f32x4*)(p + (size_t)f * 16 * KDIM);
        v[2 * f + 1] = *(const f32x4*)(p + (size_t)f * 16 * KDIM + 16);
      }
    } else {
      const float* p = a.w + (size_t)(ks * 32 + 4 * kq) * SH + unit * 64 + 4 * l15;
#pragma unroll
      for (int i = 0; i < 8; i++) v[i] = *(const f32x4*)(p + (size_t)((i & 3) + (i >> 2) * 16) * SH);
    }
    call = __builtin_amdgcn_readfirstlane(ctr_v) + 1;
    base = call * 64;
    if (is_pull && wv < NPW) {
      const unsigned qoff = (unsigned)(QCTR_OFF - FLAG_OFF) + ((call & 1) * 8 + plink) * 128;
      asm volatile("v_mov_b32 v200, 2\n\ts_nop 4\n\tbuffer_atomic_add v200, %0, %1, %2 offen sc0"
                   :: "v"(l0), "s"(rfs), "s"(qoff) : "v200", "memory");
    }
    u32x4 o[4];
    if (kind == 0) {
#pragma unroll
      for (int f = 0; f < 4; f++)
        o[f] = u32x4{cvt2(v[2 * f][0], v[2 * f][1]), cvt2(v[2 * f][2], v[2 * f][3]),
                     cvt2(v[2 * f + 1][0], v[2 * f + 1][1]), cvt2(v[2 * f + 1][2], v[2 * f + 1][3])};
    } else {
#pragma unroll
      for (int r = 0; r < 4; r++)
        o[r] = u32x4{cvt2(v[0][r], v[1][r]), cvt2(v[2][r], v[3][r]), cvt2(v[4][r], v[5][r]), cvt2(v[6][r], v[7][r])};
    }
    const unsigned dmask = (unsigned)((kind ? DEST_MASK_W : DEST_MASK_X) >> (8 * unit)) & 0xff;
    if (c16 < NPUSH) {
#pragma unroll
      for (int p = 1; p < NR; p++)
        if ((dmask >> p) & 1) {
          const __amdgpu_buffer_rsrc_t rq = mkrsrc(a.ws[rank ^ p] + stg_off(kind, rank, unit), UNIT_BYTES);
#pragma unroll
          for (int i = 0; i < 4; i++)
            __builtin_amdgcn_raw_buffer_store_b128(o[i], rq, ks * BLK + piece_off(kind, i, lane), 0, AUX_SYS);
        }
    } else {
      float mx = 0.f;
#pragma unroll
      for (int i = 0; i < 8; i++) {
        mx = fmaxf(mx, fmaxf(fabsf(v[i][0]), fabsf(v[i][1])));
        mx = fmaxf(mx, fmaxf(fabsf(v[i][2]), fabsf(v[i][3])));
      }
      const unsigned mxb = wave_max_u32(__float_as_uint(mx));
      const unsigned e7 = (cvt2(__uint_as_float(mxb), 0.f) >> 8) & 0x7f;
      const unsigned b80 = e7 * 0x01010101u | 0x80808080u;
      const unsigned T0 = (b80 - 0x03020100u) & 0x7f7f7f7fu, T1 = (b80 - 0x07060504u) & 0x007f7f7fu;
      lu32* hdr = (lu32*)(smem + wv * 64);
      unsigned cnt = 0;
      u32x4 PA, PB, PC;
#pragma unroll
      for (int p = 0; p < 4; p++) {
        unsigned L[2], N = 0;
#pragma unroll
        for (int h = 0; h < 2; h++) {
          const unsigned d0 = o[p][2 * h], d1 = o[p][2 * h + 1];
          L[h] = perm(d1, d0, 0x06040200u);
          const unsigned H = perm(d1, d0, 0x07050301u);
          const unsigned t = H & 0x7f7f7f7fu;
          unsigned code = (b80 - t) & 0x07070707u;
          const unsigned z = ~(t + 0x7f7f7f7fu) & 0x80808080u;
          code |= (z >> 5) | (z >> 6) | (z >> 7);
          const unsigned mis = perm(T1, T0, code) ^ t;
          N |= h == 0 ? (((H & 0x80808080u) >> 4) | code) : ((H & 0x80808080u) | (code << 4));
          if (__any(mis != 0)) {
            const unsigned po = piece_off(kind, p, lane) + 8 * h + 1;
            for (int bb = 0; bb < 4; bb++) {
              const bool bad = ((mis >> (8 * bb)) & 0xff) != 0;
              const unsigned long long mk = __ballot(bad);
              const unsigned idx = cnt + __builtin_amdgcn_mbcnt_hi((unsigned)(mk >> 32), __builtin_amdgcn_mbcnt_lo((unsigned)mk, 0));
              if (bad && idx < 15) hdr[1 + idx] = ((po + 2 * bb) << 8) | ((H >> (8 * bb)) & 0xff);
              cnt += __builtin_popcountll(mk);
            }
          }
        }
        if (p < 2) { PA[2 * p] = L[0]; PA[2 * p + 1] = L[1]; }
        else { PB[2 * p - 4] = L[0]; PB[2 * p - 3] = L[1]; }
        PC[p] = N;
      }
      const __amdgpu_buffer_rsrc_t rp = mkrsrc(me + EXP_OFF + (size_t)u16 * PUNIT_BYTES, PUNIT_BYTES);
      const unsigned pc = (ks >> 2) * PCHUNK, pw = ks & 3;
      const unsigned po = pc + pw * PBLK + lane * 16;
      __builtin_amdgcn_raw_buffer_store_b128(PA, rp, po, 0, AUX_SYS);
      __builtin_amdgcn_raw_buffer_store_b128(PB, rp, po + 1024, 0, AUX_SYS);
      __builtin_amdgcn_raw_buffer_store_b128(PC, rp, po + 2048, 0, AUX_SYS);
      unsigned hv = hdr[l15];
      if (lane == 0) hv = e7 | (cnt > 15 ? 0x10000u : cnt << 8);
      if (lane < 16) __builtin_amdgcn_raw_buffer_store_b32(hv, rp, pc + 4 * PBLK + pw * 64 + lane * 4, 0, AUX_SYS);
    }
#pragma unroll
    for (int p = 0; p < 4; p++)
      __builtin_amdgcn_raw_buffer_store_b128(o[p], rd, ks * BLK + piece_off(kind, p, lane), 0, AUX_SYS);
    STAMP(15);
    asm volatile("s_waitcnt vmcnt(0)" ::: "memory");
    const int rel = rank ^ lane;
    if (c16 < NPUSH) {
      if (lane < NR && rel != 0 && ((dmask >> rel) & 1))
        __hip_atomic_fetch_add((gu32*)(wsl + ARR_OFF + (size_t)(rank * 16 + u16) * 128 + (ks >> 2) * 4), 1u, __ATOMIC_RELAXED,
                               __HIP_MEMORY_SCOPE_SYSTEM);
    }
    __builtin_amdgcn_s_barrier();
    if (wv == 0 && lane < NR && ((dmask >> rel) & 1) && (rel == 0 || c16 >= NPUSH))
      st_sys(wsl + READY_OFF + (size_t)(rank * 16 + u16) * 128 + c16 * 4, base + 1);
  }
  STAMP(1);

  if (b >= NGEMM + NPULL) {
    if (b == NWG - 1 && wv == 0) {
      bool ok = true;
#pragma unroll
      for (int i = 0; i < NGEMM / 64; i++) {
        const int c = i * 64 + lane;
        ok = wave_wait(me + DONE_OFF + c * 4, call, ((TILE_TAB[c] >> 2) & 7) != 0, me, call) && ok;
      }
      if (lane < 8) st_agent(me + QCTR_OFF + (size_t)(((call + 1) & 1) * 8 + lane) * 128, 0u);
      if (lane == 0) {
        if (!ok) a.out[rank][0] = 0x7f;
        st_agent(me + CTR_OFF, call);
      }
      STAMP(6);
    }
    return;
  }
  if (is_pull) {
    if (wv >= NPW) return;
    const unsigned src = (unsigned)(rank ^ plink);
    if (src >= NR) return;
    const __amdgpu_buffer_rsrc_t rraw = mkrsrc(a.ws[src], 2 * STG_KIND_BYTES);
    const __amdgpu_buffer_rsrc_t rd = mkrsrc(me, 2 * STG_KIND_BYTES);
    const u32x4 rss = mkrsrc_words(a.ws[src] + EXP_OFF, 16 * PUNIT_BYTES);
    const unsigned wperm = piece_off(1, 0, lane);
    char* scr = smem + 1024 + wv * BLK;
    constexpr unsigned SLOT = 3072 + 256;
    char* ring = smem + 1024 + NPW * BLK + wv * 2 * SLOT;
    unsigned long long rdy03 = 0;
    unsigned rdy4 = 0;
    bool ok = true;
    constexpr unsigned C0 = 2 * NPUSH, NITEM = (32 - C0) * 20;
    const unsigned qoff = (unsigned)(QCTR_OFF - FLAG_OFF) + ((call & 1) * 8 + plink) * 128;
    auto grab = [&]() {
      asm volatile("v_mov_b32 v200, 1\n\ts_nop 4\n\tbuffer_atomic_add v200, %0, %1, %2 offen sc0"
                   :: "v"(l0), "s"(rfs), "s"(qoff) : "v200", "memory");
    };
    auto ticket = [&]() -> unsigned {
      unsigned v;
      asm volatile("v_mov_b32 %0, v200" : "=v"(v) :: "v200", "memory");
      return __builtin_amdgcn_readfirstlane(v);
    };
    auto need = [&](unsigned n) {
      const unsigned cq = n / 20, c = C0 + cq, ui = (n - cq * 20) >> 2, c2 = c >> 1;
      for (Spin sp;;) {
        const unsigned m = ui == 4 ? rdy4 : (unsigned)(rdy03 >> (16 * ui));
        if (((m >> c2) & 1) || !ok) return;
        const unsigned u16 = (ue >> (4 * ui)) & 15;
        const unsigned v = lane < 16 ? ld_sys(me + READY_OFF + (size_t)(src * 16 + u16) * 128 + lane * 4) : 0u;
        const unsigned mk = (unsigned)__ballot(v >= base + 1) & 0xffffu;
        if (ui == 4) rdy4 = mk; else rdy03 |= (unsigned long long)mk << (16 * ui);
        if ((mk >> c2) & 1) return;
        if (sp.expired(me, call)) ok = false;
      }
    };
    const unsigned vpay = lane * 16, vhdr = lane < 16 ? lane * 4 : 0x40000000u;
    auto load = [&](unsigned slot, unsigned n) {
      const bool val = n < NITEM;
      const unsigned nn = val ? n : 0;
      const unsigned cq = nn / 20, c = C0 + cq, r = nn - cq * 20, u16 = (ue >> (4 * (r >> 2))) & 15, w = r & 3;
      const unsigned o = u16 * PUNIT_BYTES + c * PCHUNK;
      const unsigned oob = val ? 0u : 0x40000000u;
      const unsigned so = o + w * PBLK, sh = o + 4 * PBLK + w * 64 - 3072;
      const unsigned lds = __builtin_amdgcn_readfirstlane((unsigned)(unsigned long long)(lu8*)(ring + slot * SLOT));
      unsigned keep;
      asm volatile("s_mov_b32 %0, m0\n\ts_mov_b32 m0, %1\n\ts_nop 0\n\t"
                   "buffer_load_dwordx4 %2, %4, %5 offen sc0 sc1 lds\n\t"
                   "buffer_load_dwordx4 %2, %4, %5 offen offset:1024 sc0 sc1 lds\n\t"
                   "buffer_load_dwordx4 %2, %4, %5 offen offset:2048 sc0 sc1 lds\n\t"
                   "buffer_load_dword %3, %4, %6 offen offset:3072 sc0 sc1 lds\n\t"
                   "s_mov_b32 m0, %0"
                   : "=&s"(keep) : "s"(lds), "v"(vpay | oob), "v"(vhdr | oob), "s"(rss), "s"(so), "s"(sh) : "memory");
    };
    auto emit = [&](unsigned slot, unsigned n) {
      const char* sl = ring + slot * SLOT;
      const unsigned H = *(const lu32*)(sl + 3072 + (lane & 15) * 4);
      const u32x4 A = *(const lu32x4*)(sl + lane * 16), B = *(const lu32x4*)(sl + 1024 + lane * 16), C = *(const lu32x4*)(sl + 2048 + lane * 16);
      const unsigned cq = n / 20, c = C0 + cq, r = n - cq * 20, u16 = (ue >> (4 * (r >> 2))) & 15, w = r & 3;
      const int kind = u16 >> 3;
      const unsigned dst = (unsigned)stg_off(kind, src, u16 & 7) + (c * 4 + w) * BLK;
      const unsigned h0 = __builtin_amdgcn_readfirstlane(H);
      if (h0 & 0x10000u) {
        u32x4 rr[4];
#pragma unroll
        for (int p = 0; p < 4; p++) rr[p] = __builtin_amdgcn_raw_buffer_load_b128(rraw, p * 1024 + lane * 16, dst, AUX_SYS);
#pragma unroll
        for (int p = 0; p < 4; p++) __builtin_amdgcn_raw_buffer_store_b128(rr[p], rd, p * 1024 + lane * 16, dst, AUX_AGENT);
        return;
      }
      const unsigned b80 = (h0 & 0x7f) * 0x01010101u | 0x80808080u;
      const unsigned T0 = (b80 - 0x03020100u) & 0x7f7f7f7fu, T1 = (b80 - 0x07060504u) & 0x007f7f7fu;
      u32x4 o[4];
#pragma unroll
      for (int p = 0; p < 4; p++) {
        const unsigned L0 = p < 2 ? A[2 * p] : B[2 * p - 4], L1 = p < 2 ? A[2 * p + 1] : B[2 * p - 3];
        const unsigned N = C[p];
        const unsigned h0e = perm(T1, T0, N & 0x07070707u) | ((N << 4) & 0x80808080u);
        const unsigned h1o = perm(T1, T0, (N >> 4) & 0x07070707u) | (N & 0x80808080u);
        o[p] = u32x4{perm(h0e, L0, 0x05010400u), perm(h0e, L0, 0x07030602u), perm(h1o, L1, 0x05010400u), perm(h1o, L1, 0x07030602u)};
      }
      const unsigned cnt = (h0 >> 8) & 0xff;
      if (cnt == 0) {
#pragma unroll
        for (int p = 0; p < 4; p++)
          __builtin_amdgcn_raw_buffer_store_b128(o[p], rd, p * 1024 + (kind ? (wperm ^ (p * 16)) : lane * 16), dst, AUX_AGENT);
      } else {
#pragma unroll
        for (int p = 0; p < 4; p++) *(lu32x4*)(scr + p * 1024 + (kind ? (wperm ^ (p * 16)) : lane * 16)) = o[p];
        if (lane >= 1 && lane <= (int)cnt) *(lu8*)(scr + ((H >> 8) & 0xfff)) = (unsigned char)H;
#pragma unroll
        for (int p = 0; p < 4; p++) o[p] = *(const lu32x4*)(scr + p * 1024 + lane * 16);
#pragma unroll
        for (int p = 0; p < 4; p++) __builtin_amdgcn_raw_buffer_store_b128(o[p], rd, p * 1024 + lane * 16, dst, AUX_AGENT);
      }
    };
    auto arrive = [&](unsigned n) {
      const unsigned cq = n / 20, c = C0 + cq, r = n - cq * 20, u16 = (ue >> (4 * (r >> 2))) & 15;
      const unsigned so = (unsigned)(ARR_OFF - FLAG_OFF) + (src * 16 + u16) * 128 + c * 4;
      asm volatile("buffer_atomic_add %0, %1, %2, %3 offen" :: "v"(1u), "v"(l0), "s"(rfs), "s"(so) : "memory");
    };
    unsigned cur = ticket();
    unsigned nxt = cur + 1, prev = NITEM, slot = 0;
    grab();
    if (cur < NITEM) need(cur);
    STAMP(2);
    load(0, cur);
    while (cur < NITEM) {
      if (nxt < NITEM) need(nxt);
      load(slot ^ 1, nxt);
      asm volatile("s_waitcnt vmcnt(4)" ::: "memory");
      if (prev < NITEM) arrive(prev);
      const unsigned nn = ticket();
      grab();
      emit(slot, cur);
      prev = cur; cur = nxt; nxt = nn; slot ^= 1;
    }
    asm volatile("s_waitcnt vmcnt(0)" ::: "memory");
    if (prev < NITEM) arrive(prev);
    STAMP(3);
    WSTAMP(8 + (wv & 3));
    return;
  }

  const int t1 = (te >> 14) & 7, i1 = (te >> 11) & 7, t2 = (te >> 8) & 7, i2 = (te >> 5) & 7, tu = (te >> 2) & 7, pj = te & 3;
  const int srcW = rank ^ tu;
  char* const donep = a.ws[srcW & 7] + DONE_OFF + b * 4;
  lu32* sm = (lu32*)smem;
  static_assert(63 % NBUF != 0, "sm overlays stage buffer 0");
  if (wv >= 4) {
    const int lw = wv - 4;
    auto unit_of = [&](int u, int& kind, int& src, int& unit) {
      kind = u >> 1;
      src = u == 0 ? rank ^ t1 : (u == 1 ? rank ^ t2 : srcW);
      unit = u == 0 ? i1 : (u == 1 ? i2 : 2 * pj + (u & 1));
    };
    int kind, src, unit;
    unit_of(lw, kind, src, unit);
    const char* g = me + stg_off(kind, src, unit) + lane * 16;
    const char* pollp; bool pollLocal;
    {
      int k2, s2, u2;
      unit_of(lane >> 4, k2, s2, u2);
      pollLocal = s2 == rank;
      pollp = me + (pollLocal ? READY_OFF : ARR_OFF) + (size_t)(((s2 & 7) * 2 + k2) * 8 + u2) * 128;
    }
    int avail = 0;
    bool wok = true;
    int nb = 0;
    for (int s = -PF; s < 64; s++) {
      if (lw == 0 && s + PF < 64 && wok) {
        const int c = (s + PF) >> 1;
        if (c >= avail) {
          const int wi = (pollLocal ? c >> 1 : c) + (lane & 15);
          const bool inr = wi < (pollLocal ? 16 : 32);
          const unsigned thr = pollLocal ? base + 1 : 4 * call;
          const int first = pollLocal ? 2 * wi : wi;
          Spin sp;
          for (;;) {
            const unsigned v = inr ? ld_sys(pollp + wi * 4) : thr;
            const int pr = v >= thr ? 64 : first;
            if (__all(pr > c)) {
              avail = 64 - (int)wave_max_u32(64u - (unsigned)pr);
              if (c == 31) WSTAMP(6); if (c == 29) WSTAMP(7); if (c == 27) WSTAMP(8); if (c == 0) WSTAMP(9); if (c == 16) WSTAMP(10);
              break;
            }
            if (sp.expired(me, call)) { wok = false; break; }
          }
        }
      }
      __builtin_amdgcn_s_barrier();
      asm volatile("" ::: "memory");
      if (s + PF < 64) {
        const unsigned* gp = (const unsigned*)(g + (size_t)(s + PF) * (2 * BLK));
        unsigned* lp = (unsigned*)(smem + nb * STAGE_BYTES + lw * (2 * BLK));
        nb = nb == NBUF - 1 ? 0 : nb + 1;
        __builtin_amdgcn_global_load_lds(gp, lp, 16, 0, AUX_SYS);
        __builtin_amdgcn_global_load_lds(gp, lp, 16, 1024, AUX_SYS);
        __builtin_amdgcn_global_load_lds(gp, lp, 16, 2048, AUX_SYS);
        __builtin_amdgcn_global_load_lds(gp, lp, 16, 3072, AUX_SYS);
        __builtin_amdgcn_global_load_lds(gp + 1024, lp + 1024, 16, 0, AUX_SYS);
        __builtin_amdgcn_global_load_lds(gp + 1024, lp + 1024, 16, 1024, AUX_SYS);
        __builtin_amdgcn_global_load_lds(gp + 1024, lp + 1024, 16, 2048, AUX_SYS);
        __builtin_amdgcn_global_load_lds(gp + 1024, lp + 1024, 16, 3072, AUX_SYS);
        if (PF == 4) asm volatile("s_waitcnt vmcnt(24)" ::: "memory");
        else if (PF == 3) asm volatile("s_waitcnt vmcnt(16)" ::: "memory");
        else asm volatile("s_waitcnt vmcnt(8)" ::: "memory");
      } else {
        const int rem = 62 - s;
        if (rem >= 2) asm volatile("s_waitcnt vmcnt(16)" ::: "memory");
        else if (rem == 1) asm volatile("s_waitcnt vmcnt(8)" ::: "memory");
        else asm volatile("s_waitcnt vmcnt(0)" ::: "memory");
      }
    }
    return;
  }
  const int xu = wv >> 1, wu = wv & 1;
  f32x4 acc[4][4];
#pragma unroll
  for (int f = 0; f < 4; f++)
#pragma unroll
    for (int j = 0; j < 4; j++) acc[f][j] = f32x4{0.f, 0.f, 0.f, 0.f};
  const unsigned xoff = xu * (2 * BLK) + lane * 16;
  unsigned woff[4];
  {
    const int r = lane & 3, q = (lane >> 2) & 3, kq = lane >> 4;
#pragma unroll
    for (int j = 0; j < 4; j++) woff[j] = (2 + wu) * (2 * BLK) + r * 1024 + kq * 256 + q * 64 + ((j ^ r) * 16);
  }
  {
    bf16x8 xa[4], wa[4], xb[4], wb[4];
#pragma unroll
    for (int f = 0; f < 4; f++) { xb[f] = bf16x8{}; wb[f] = bf16x8{}; }
    for (int s = -PF; s < 0; s++) __builtin_amdgcn_s_barrier();
    int cb = 0;
#define ILV() do { _Pragma("unroll") for (int q_ = 0; q_ < 8; q_++) { __builtin_amdgcn_sched_group_barrier(0x008, 1, 0); \
    __builtin_amdgcn_sched_group_barrier(0x100, 1, 0); } __builtin_amdgcn_sched_group_barrier(0x008, 8, 0); } while (0)
    for (int s = 0; s < 64; s++) {
      asm volatile("s_waitcnt lgkmcnt(0)" ::: "memory");
      __builtin_amdgcn_s_barrier();
      asm volatile("" ::: "memory");
      if (s == 0) STAMP(2);
      if (s == 56) STAMP(11);
      if (s == 60) STAMP(12);
      if (s == 62) STAMP(13);
      const char* sb = smem + cb * STAGE_BYTES;
      cb = cb == NBUF - 1 ? 0 : cb + 1;
      __builtin_amdgcn_sched_barrier(0);
      wa[0] = *(const bf16x8*)(sb + woff[0]); xa[0] = *(const bf16x8*)(sb + xoff);
#pragma unroll
      for (int f = 1; f < 4; f++) wa[f] = *(const bf16x8*)(sb + woff[f]);
#pragma unroll
      for (int f = 1; f < 4; f++) xa[f] = *(const bf16x8*)(sb + xoff + f * 1024);
#pragma unroll
      for (int f = 0; f < 4; f++)
#pragma unroll
        for (int j = 0; j < 4; j++) acc[f][j] = __builtin_amdgcn_mfma_f32_16x16x32_bf16(wb[j], xb[f], acc[f][j], 0, 0, 0);
      ILV();
      __builtin_amdgcn_sched_barrier(0);
      wb[0] = *(const bf16x8*)(sb + woff[0] + BLK); xb[0] = *(const bf16x8*)(sb + xoff + BLK);
#pragma unroll
      for (int f = 1; f < 4; f++) wb[f] = *(const bf16x8*)(sb + woff[f] + BLK);
#pragma unroll
      for (int f = 1; f < 4; f++) xb[f] = *(const bf16x8*)(sb + xoff + BLK + f * 1024);
#pragma unroll
      for (int f = 0; f < 4; f++)
#pragma unroll
        for (int j = 0; j < 4; j++) acc[f][j] = __builtin_amdgcn_mfma_f32_16x16x32_bf16(wa[j], xa[f], acc[f][j], 0, 0, 0);
      ILV();
      __builtin_amdgcn_sched_barrier(0);
    }
#pragma unroll
    for (int f = 0; f < 4; f++)
#pragma unroll
      for (int j = 0; j < 4; j++) acc[f][j] = __builtin_amdgcn_mfma_f32_16x16x32_bf16(wb[j], xb[f], acc[f][j], 0, 0, 0);
  }
  STAMP(3);
  float mf = 0.f;
#pragma unroll
  for (int f = 0; f < 4; f++)
#pragma unroll
    for (int j = 0; j < 4; j++) {
      const f32x4 t = acc[f][j];
      mf = fmaxf(fmaxf(mf, fabsf(t[0])), fabsf(t[1]));
      mf = fmaxf(fmaxf(mf, fabsf(t[2])), fabsf(t[3]));
    }
  unsigned m = wave_max_u32(__float_as_uint(mf));
  if (lane == 0) sm[wv] = m;
  asm volatile("s_waitcnt lgkmcnt(0)" ::: "memory");
  __builtin_amdgcn_s_barrier();
  asm volatile("" ::: "memory");
  m = max(max(sm[0], sm[1]), max(sm[2], sm[3]));
  bool bad = false;
  if (tid < NR) {
    gu64* g = (gu64*)(wsl + AMAX_OFF + (size_t)(rank * NGEMM + b) * 8);
    __hip_atomic_store(g, ((unsigned long long)call << 32) | m, __ATOMIC_RELAXED, __HIP_MEMORY_SCOPE_SYSTEM);
  }
  unsigned gm = 0;
  {
    const char* gp = me + AMAX_OFF + tid * 32;
    Spin sp;
    for (;;) {
      u32x4 g0, g1;
      unsigned tm;
      asm volatile("global_load_dwordx4 %0, %3, off sc0 sc1\n\tglobal_load_dwordx4 %1, %3, off offset:16 sc0 sc1\n\t"
                   "global_load_dword %2, %4, off sc1\n\ts_waitcnt vmcnt(0)"
                   : "=&v"(g0), "=&v"(g1), "=&v"(tm) : "v"(gp), "v"(me + TMO_OFF) : "memory");
      const bool okk = g0[1] == call && g0[3] == call && g1[1] == call && g1[3] == call;
      if (__all(okk)) {
        gm = max(max(g0[0], g0[2]), max(g1[0], g1[2]));
        if (__builtin_amdgcn_readfirstlane(tm) == call) bad = true;
        break;
      }
      if (sp.expired(me, call)) { bad = true; break; }
    }
  }
  gm = wave_max_u32(gm);
  if (lane == 0) { sm[4 + wv] = gm; sm[8 + wv] = bad; }
  __syncthreads();
  gm = max(max(sm[4], sm[5]), max(sm[6], sm[7]));
  bad = (sm[8] | sm[9] | sm[10] | sm[11]) != 0;
  STAMP(4);
  const float inv = 448.0f / __uint_as_float(gm);
  {
    const int tx = xu == 0 ? t1 : t2, ix = xu == 0 ? i1 : i2;
    unsigned char* o = outp + (size_t)((rank ^ tx) * SH + ix * 64 + (lane & 15)) * SH + pj * 128 + wu * 64 + (lane >> 4) * 16;
#pragma unroll
    for (int f = 0; f < 4; f++) {
      u32x4 q;
#pragma unroll
      for (int j = 0; j < 4; j++) {
        int w = __builtin_amdgcn_cvt_pk_fp8_f32(acc[f][j][0] * inv, acc[f][j][1] * inv, 0, false);
        w = __builtin_amdgcn_cvt_pk_fp8_f32(acc[f][j][2] * inv, acc[f][j][3] * inv, w, true);
        q[j] = bad ? 0x7f7f7f7fu : (unsigned)w;
      }
      if (tu != 0) asm volatile("global_store_dwordx4 %0, %1, off sc0 sc1\n\ts_nop 1" ::"v"(o + (size_t)f * 16 * SH), "v"(q) : "memory");
      else *(u32x4*)(o + (size_t)f * 16 * SH) = q;
    }
  }
  if (tu != 0) {
    asm volatile("s_waitcnt vmcnt(0)" ::: "memory");
    __builtin_amdgcn_s_barrier();
    if (tid == 0) st_sys(donep, call);
  }
  STAMP(5);
}
}

extern "C" void kernel_launch(int rank, int nranks, ncclComm_t comm, hipStream_t stream, void* const* d_in,
                              const int* in_sizes, int n_in, void* d_out, int out_size, const int* in_dtypes,
                              int out_dtype, void* const* d_out_all, void* const* d_ws_all, size_t ws_bytes) {
  (void)comm; (void)in_sizes; (void)n_in; (void)d_out; (void)out_size; (void)in_dtypes; (void)out_dtype;
  if (nranks != NR || rank < 0 || rank >= NR || ws_bytes < WS_NEED) {
    fprintf(stderr, "unsupported configuration\n");
    return;
  }
  Args a;
  a.x = (const float*)d_in[0];
  a.w = (const float*)d_in[1];
  for (int p = 0; p < NR; p++) {
    a.ws[p] = (char*)d_ws_all[p];
    a.out[p] = (unsigned char*)d_out_all[p];
  }
  a.rank = rank;
  a.pad = 0;
  fused_ag_gemm<<<NWG, 512, 0, stream>>>(a);
  hipError_t e = hipGetLastError();
  if (e != hipSuccess) fprintf(stderr, "launch error %s\n", hipGetErrorString(e));
}
```

```cpp
#include <hip/hip_runtime.h>
#include <rccl/rccl.h>
#include <cstdio>
#include <cstdint>

typedef unsigned u32x4 __attribute__((ext_vector_type(4)));
typedef unsigned u32x3 __attribute__((ext_vector_type(3)));
typedef float f32x4 __attribute__((ext_vector_type(4)));
typedef __bf16 bf16x8 __attribute__((ext_vector_type(8)));
typedef __bf16 bf16x2 __attribute__((ext_vector_type(2)));
typedef float f32x2 __attribute__((ext_vector_type(2)));

#define NPW 5
#define NPUSH 1
#define LOOK 2
#define LOOKMASK 1
#define STAGGER 15

namespace {
constexpr int NR = 8;
constexpr int KDIM = 4096;
constexpr int SH = 512;
constexpr unsigned BLK = 4096;
constexpr unsigned UNIT_BYTES = 128 * BLK;
constexpr unsigned PBLK = 2944;
constexpr unsigned PUNIT_BYTES = 128 * PBLK;
constexpr unsigned NEXC = 41;
constexpr size_t STG_KIND_BYTES = (size_t)NR * 8 * UNIT_BYTES;
constexpr size_t EXP_OFF = 2 * STG_KIND_BYTES;
constexpr size_t FLAG_OFF = EXP_OFF + (size_t)16 * PUNIT_BYTES;
constexpr size_t READY_OFF = FLAG_OFF;
constexpr size_t ARR_OFF = READY_OFF + NR * 16 * 128;
constexpr size_t AMAX_OFF = ARR_OFF + NR * 16 * 128;
constexpr size_t DONE_OFF = AMAX_OFF + NR * 128 * 8;
constexpr size_t CTR_OFF = DONE_OFF + 1024;
constexpr size_t TMO_OFF = CTR_OFF + 128;
constexpr size_t QCTR_OFF = TMO_OFF + 128;
constexpr size_t DBG_OFF = QCTR_OFF + 16 * 128;
constexpr size_t WS_NEED = DBG_OFF + 512 * 16 * 8;
static_assert(FLAG_OFF % 128 == 0, "align");
constexpr int NGEMM = 128, NPULL = 35, NWG = 256;
constexpr unsigned long long DEADLINE = 30000000ull;
constexpr unsigned STAGE_BYTES = 8 * BLK;
#define NBUFS 4
constexpr int NBUF = NBUFS;
constexpr int PF = NBUF - 1;

__constant__ unsigned TILE_TAB[NGEMM] = {16420,2145,5154,5283,14530,10241,8928,10432,69476,67649,7906,47971,70370,15553,17664,15424,3116,74817,8450,78019,77058,16577,113792,55360,18284,75905,12130,83011,3238,113377,121888,75872,70924,82785,73762,70820,13574,1059,3115,77856,77964,5158,78690,82660,69798,18147,7435,4311,8207,55430,3939,112708,78566,76,47243,47287,121935,75622,6307,75,112902,11404,113803,55351,1081,3112,73859,11115,10991,13484,4373,69751,9433,77064,10340,75979,12335,48876,9973,12377,12153,121928,12420,78923,76015,9271,67637,47225,17561,5167,74948,55518,79599,74935,73845,75833,83033,75023,6216,70846,11477,78103,77685,2234,2238,2173,12008,78590,69845,12381,47258,55386,8478,70813,73864,81950,9501,48349,68890,69850,68766,76669,78024,112734,10493,69373,75642,79034};
__constant__ unsigned PULL_TAB[8] = {0, 0xdc980, 0xfe987, 0xfe983, 0x65421, 0xfeba0, 0xdcba7, 0xdcba3};
constexpr unsigned long long DEST_MASK_X = 0x4511111189111123ull, DEST_MASK_W = 0x2d2dc3c3e1e10f0full;

struct Args {
  const float* x;
  const float* w;
  char* ws[NR];
  unsigned char* out[NR];
  int rank;
  int pad;
};

typedef __attribute__((address_space(1))) unsigned gu32;
typedef __attribute__((address_space(1))) unsigned long long gu64;
typedef __attribute__((address_space(3))) unsigned lu32;
typedef __attribute__((address_space(3))) unsigned char lu8;
typedef __attribute__((address_space(3))) unsigned short lu16;
typedef __attribute__((address_space(3))) u32x4 lu32x4;

__device__ __forceinline__ unsigned ld_sys(const void* p) { return __hip_atomic_load((const gu32*)p, __ATOMIC_RELAXED, __HIP_MEMORY_SCOPE_SYSTEM); }
__device__ __forceinline__ void st_sys(void* p, unsigned v) { __hip_atomic_store((gu32*)p, v, __ATOMIC_RELAXED, __HIP_MEMORY_SCOPE_SYSTEM); }
__device__ __forceinline__ unsigned ld_agent(const void* p) { return __hip_atomic_load((const gu32*)p, __ATOMIC_RELAXED, __HIP_MEMORY_SCOPE_AGENT); }
__device__ __forceinline__ void st_agent(void* p, unsigned v) { __hip_atomic_store((gu32*)p, v, __ATOMIC_RELAXED, __HIP_MEMORY_SCOPE_AGENT); }

__device__ __forceinline__ __amdgpu_buffer_rsrc_t mkrsrc(const void* p, unsigned bytes) {
  unsigned long long a = (unsigned long long)p;
  unsigned lo = __builtin_amdgcn_readfirstlane((unsigned)a), hi = __builtin_amdgcn_readfirstlane((unsigned)(a >> 32));
  void* q = (void*)(((unsigned long long)hi << 32) | lo);
  return __builtin_amdgcn_make_buffer_rsrc(q, 0, bytes, 0x00020000);
}
__device__ __forceinline__ u32x4 mkrsrc_words(const void* p, unsigned bytes) {
  unsigned long long a = (unsigned long long)p;
  return u32x4{(unsigned)__builtin_amdgcn_readfirstlane((unsigned)a), (unsigned)__builtin_amdgcn_readfirstlane((unsigned)(a >> 32)) & 0xffffu,
               bytes, 0x00020000u};
}
__device__ __forceinline__ size_t stg_off(int kind, int src, int unit) {
  return (size_t)kind * STG_KIND_BYTES + (size_t)(src * 8 + unit) * UNIT_BYTES;
}
__device__ __forceinline__ unsigned cvt2(float a, float b) {
  f32x2 v = {a, b};
  bf16x2 r = __builtin_convertvector(v, bf16x2);
  return __builtin_bit_cast(unsigned, r);
}
__device__ __forceinline__ unsigned perm(unsigned hi, unsigned lo, unsigned sel) { return __builtin_amdgcn_perm(hi, lo, sel); }
__device__ __forceinline__ unsigned wave_max_u32(unsigned v) {
  v = max(v, (unsigned)__builtin_amdgcn_update_dpp(0, (int)v, 0x111, 0xf, 0xf, false));
  v = max(v, (unsigned)__builtin_amdgcn_update_dpp(0, (int)v, 0x112, 0xf, 0xf, false));
  v = max(v, (unsigned)__builtin_amdgcn_update_dpp(0, (int)v, 0x114, 0xf, 0xf, false));
  v = max(v, (unsigned)__builtin_amdgcn_update_dpp(0, (int)v, 0x118, 0xf, 0xf, false));
  v = max(v, (unsigned)__builtin_amdgcn_update_dpp(0, (int)v, 0x142, 0xa, 0xf, false));
  v = max(v, (unsigned)__builtin_amdgcn_update_dpp(0, (int)v, 0x143, 0xc, 0xf, false));
  return (unsigned)__builtin_amdgcn_readlane((int)v, 63);
}
#define STAMP(i) do {} while (0)
#define WSTAMP(i) do {} while (0)

struct Spin {
  unsigned it = 0;
  unsigned long long t0 = 0;
  __device__ __forceinline__ bool expired(char* me, unsigned call) {
    if (it == 0) t0 = __builtin_amdgcn_s_memrealtime();
    if ((++it & 15) == 0) {
      if (ld_agent(me + TMO_OFF) == call || __builtin_amdgcn_s_memrealtime() - t0 > DEADLINE) {
        st_agent(me + TMO_OFF, call);
        return true;
      }
    }
    __builtin_amdgcn_s_sleep(1);
    return false;
  }
};
__device__ __forceinline__ bool wave_wait(const void* p, unsigned thr, bool act, char* me, unsigned call) {
  Spin sp;
  for (;;) {
    unsigned v = act ? ld_sys(p) : 0xffffffffu;
    if (__all(v >= thr)) return true;
    if (sp.expired(me, call)) return false;
  }
}

constexpr int AUX_SYS = 17, AUX_AGENT = 16;

__device__ __forceinline__ unsigned piece_off(int kind, int p, int lane) {
  const int kq = lane >> 4, q = (lane >> 2) & 3, j = lane & 3;
  return kind == 0 ? p * 1024 + lane * 16 : p * 1024 + kq * 256 + q * 64 + ((j ^ p) * 16);
}

__global__ __launch_bounds__(512, 2) void fused_ag_gemm(Args a) {
  __shared__ __attribute__((aligned(16))) char smem[NBUF * STAGE_BYTES];
  const int tid = threadIdx.x, lane = tid & 63;
  const int wv = __builtin_amdgcn_readfirstlane(tid >> 6);
  const int b = blockIdx.x;
  constexpr int NH0 = 16 * NPUSH;
  const int task = b < NGEMM ? b + NH0 + NPULL
                             : (b < NGEMM + NPULL ? b - NGEMM + NH0 : (b - (NGEMM + NPULL) < NH0 ? b - (NGEMM + NPULL) : b));
  const int c16 = task >> 4, u16 = task & 15, kind = u16 >> 3, unit = u16 & 7;
  const int ks = c16 * 8 + wv;
  const int kq = lane >> 4, l15 = lane & 15;
  if (STAGGER > 0 && c16 >= NPUSH) {
    const unsigned long long tstart = __builtin_amdgcn_s_memrealtime();
    while (__builtin_amdgcn_s_memrealtime() - tstart < (unsigned long long)(c16 * STAGGER)) __builtin_amdgcn_s_sleep(1);
  }
  f32x4 v[8];
  {
    const float* px = a.x + (size_t)(unit * 64 + l15) * KDIM + ks * 32 + 4 * kq;
    const float* pw = a.w + (size_t)(ks * 32 + 4 * kq) * SH + unit * 64 + 4 * l15;
    const float* p = kind == 0 ? px : pw;
#pragma unroll
    for (int i = 0; i < 8; i++) {
      const unsigned ox = (i >> 1) * 16 * KDIM + (i & 1) * 16, ow = ((i & 3) + (i >> 2) * 16) * SH;
      v[i] = *(const f32x4*)(p + (kind == 0 ? ox : ow));
    }
  }
  asm volatile("" ::: "memory");
  const int rank = a.rank;
  char* me = a.ws[rank];
  const unsigned ctr_v = ld_agent(me + CTR_OFF);
  const unsigned te = TILE_TAB[b & (NGEMM - 1)];
  const bool is_pull = b >= NGEMM && b < NGEMM + NPULL;
  const int plink = is_pull ? 1 + (b - NGEMM) / 5 : 0;
  const unsigned ue = PULL_TAB[plink];
  char* const wsl = a.ws[lane & 7];
  unsigned char* const outp = a.out[(rank ^ (int)(te >> 2)) & 7];
  const u32x4 rfs = mkrsrc_words(me + FLAG_OFF, (unsigned)(DBG_OFF - FLAG_OFF));
  const unsigned l0 = lane == 0 ? 0u : 0x40000000u;
  unsigned call, base;
  {
    const __amdgpu_buffer_rsrc_t rd = mkrsrc(me + stg_off(kind, rank, unit), UNIT_BYTES);
    call = __builtin_amdgcn_readfirstlane(ctr_v) + 1;
    base = call * 64;
    if (is_pull && wv < NPW) {
      const unsigned qoff = (unsigned)(QCTR_OFF - FLAG_OFF) + ((call & 1) * 8 + plink) * 128;
      asm volatile("v_mov_b32 v200, 2\n\ts_nop 4\n\tbuffer_atomic_add v200, %0, %1, %2 offen sc0"
                   :: "v"(l0), "s"(rfs), "s"(qoff) : "v200", "memory");
    }
    u32x4 o[4];
    if (kind == 0) {
#pragma unroll
      for (int f = 0; f < 4; f++)
        o[f] = u32x4{cvt2(v[2 * f][0], v[2 * f][1]), cvt2(v[2 * f][2], v[2 * f][3]),
                     cvt2(v[2 * f + 1][0], v[2 * f + 1][1]), cvt2(v[2 * f + 1][2], v[2 * f + 1][3])};
    } else {
#pragma unroll
      for (int r = 0; r < 4; r++)
        o[r] = u32x4{cvt2(v[0][r], v[1][r]), cvt2(v[2][r], v[3][r]), cvt2(v[4][r], v[5][r]), cvt2(v[6][r], v[7][r])};
    }
    const unsigned dmask = (unsigned)((kind ? DEST_MASK_W : DEST_MASK_X) >> (8 * unit)) & 0xff;
    if (c16 < NPUSH) {
#pragma unroll
      for (int p = 1; p < NR; p++)
        if ((dmask >> p) & 1) {
          const __amdgpu_buffer_rsrc_t rq = mkrsrc(a.ws[rank ^ p] + stg_off(kind, rank, unit), UNIT_BYTES);
#pragma unroll
          for (int i = 0; i < 4; i++)
            __builtin_amdgcn_raw_buffer_store_b128(o[i], rq, ks * BLK + piece_off(kind, i, lane), 0, AUX_SYS);
        }
    } else {
      float mx = 0.f;
#pragma unroll
      for (int i = 0; i < 8; i++) {
        mx = fmaxf(mx, fmaxf(fabsf(v[i][0]), fabsf(v[i][1])));
        mx = fmaxf(mx, fmaxf(fabsf(v[i][2]), fabsf(v[i][3])));
      }
      const unsigned mxb = wave_max_u32(__float_as_uint(mx));
      const unsigned emax = (cvt2(__uint_as_float(mxb), 0.f) >> 7) & 0xff;
      const float thr1 = __uint_as_float(emax << 23), thr2 = thr1 * 0.5f;
      unsigned n1 = 0, n2 = 0;
#pragma unroll
      for (int i = 0; i < 8; i++) {
        n1 += __builtin_popcountll(__ballot(fabsf(v[i][0]) >= thr1)) + __builtin_popcountll(__ballot(fabsf(v[i][2]) >= thr1));
        n2 += __builtin_popcountll(__ballot(fabsf(v[i][0]) >= thr2)) + __builtin_popcountll(__ballot(fabsf(v[i][2]) >= thr2));
      }
      const int top = (int)emax - (n1 <= 10 ? 1 : 0) - (n2 <= 10 ? 1 : 0);
      const unsigned delta = (top & 1) ^ 1, tp = ((top + (int)delta) >> 1) & 0x7f;
      const unsigned Md = delta ? 0x80808080u : 0u;
      const unsigned b80 = tp * 0x01010101u | 0x80808080u;
      const unsigned T = (b80 - 0x03020100u) & 0x7f7f7f7fu;
      unsigned LP[8], HH[8], S = 0, K0 = 0, K1 = 0, M = 0;
#pragma unroll
      for (int i = 0; i < 8; i++) {
        const unsigned d0 = o[i >> 1][2 * (i & 1)], d1 = o[i >> 1][2 * (i & 1) + 1];
        const unsigned L = perm(d1, d0, 0x06040200u), H = perm(d1, d0, 0x07050301u);
        const unsigned pair = ((H & 0x7f7f7f7fu) + ((L & Md) >> 7)) & 0x7f7f7f7fu;
        const unsigned code = (b80 - pair) & 0x03030303u;
        const unsigned mis = perm(T, T, code) ^ pair;
        LP[i] = L ^ Md;
        HH[i] = H;
        if (i < 4) K0 |= code << (2 * i); else K1 |= code << (2 * (i - 4));
        S |= (H & 0x80808080u) >> (7 - i);
        M |= __builtin_amdgcn_udot4(((mis + 0x7f7f7f7fu) & 0x80808080u) >> 7, 0x08040201u, 0u, false) << (4 * i);
      }
      char* hsc = smem + wv * 2304;
      lu32* hdr = (lu32*)(hsc + 2048);
      const unsigned nl = __builtin_popcount(M);
      unsigned inc = nl;
      inc += __builtin_amdgcn_update_dpp(0, (int)inc, 0x111, 0xf, 0xf, false);
      inc += __builtin_amdgcn_update_dpp(0, (int)inc, 0x112, 0xf, 0xf, false);
      inc += __builtin_amdgcn_update_dpp(0, (int)inc, 0x114, 0xf, 0xf, false);
      inc += __builtin_amdgcn_update_dpp(0, (int)inc, 0x118, 0xf, 0xf, false);
      inc += __builtin_amdgcn_update_dpp(0, (int)inc, 0x142, 0xa, 0xf, false);
      inc += __builtin_amdgcn_update_dpp(0, (int)inc, 0x143, 0xc, 0xf, false);
      const unsigned cnt = (unsigned)__builtin_amdgcn_readlane((int)inc, 63);
      if (cnt > 0 && cnt <= NEXC) {
        *(lu32x4*)(hsc + lane * 32) = u32x4{HH[0], HH[1], HH[2], HH[3]};
        *(lu32x4*)(hsc + lane * 32 + 16) = u32x4{HH[4], HH[5], HH[6], HH[7]};
        unsigned at = inc - nl;
        while (__any(M != 0)) {
          if (M != 0) {
            const unsigned jv = __builtin_ctz(M);
            M &= M - 1;
            const unsigned char hi = *(const lu8*)(hsc + lane * 32 + jv);
            const unsigned idx = (piece_off(kind, jv >> 3, lane) >> 1) + (jv & 7);
            *(lu16*)((char*)hdr + 4 + 2 * at) = (unsigned short)idx;
            *(lu8*)((char*)hdr + 4 + 2 * NEXC + at) = hi;
            at++;
          }
        }
      }
      const __amdgpu_buffer_rsrc_t rp = mkrsrc(me + EXP_OFF + (size_t)u16 * PUNIT_BYTES, PUNIT_BYTES);
      const unsigned pc = ks * PBLK;
      __builtin_amdgcn_raw_buffer_store_b128(u32x4{LP[0], LP[1], LP[2], LP[3]}, rp, pc + lane * 16, 0, AUX_SYS);
      __builtin_amdgcn_raw_buffer_store_b128(u32x4{LP[4], LP[5], LP[6], LP[7]}, rp, pc + 1024 + lane * 16, 0, AUX_SYS);
      __builtin_amdgcn_raw_buffer_store_b96(u32x3{S, K0, K1}, rp, pc + 2048 + lane * 12, 0, AUX_SYS);
      unsigned hv = hdr[lane & 31];
      if (lane == 0) hv = tp | (delta << 7) | (cnt > NEXC ? 0x10000u : cnt << 8);
      if (lane < 32) __builtin_amdgcn_raw_buffer_store_b32(hv, rp, pc + 2816 + lane * 4, 0, AUX_SYS);
    }
#pragma unroll
    for (int p = 0; p < 4; p++)
      __builtin_amdgcn_raw_buffer_store_b128(o[p], rd, ks * BLK + piece_off(kind, p, lane), 0, AUX_SYS);
    STAMP(15);
    asm volatile("s_waitcnt vmcnt(0)" ::: "memory");
    const int rel = rank ^ lane;
    if (c16 < NPUSH) {
      if (lane < NR && rel != 0 && ((dmask >> rel) & 1))
        __hip_atomic_fetch_add((gu32*)(wsl + ARR_OFF + (size_t)(rank * 16 + u16) * 128 + (ks >> 2) * 4), 1u, __ATOMIC_RELAXED,
                               __HIP_MEMORY_SCOPE_SYSTEM);
    }
    __builtin_amdgcn_s_barrier();
    if (wv == 0 && lane < NR && ((dmask >> rel) & 1) && (rel == 0 || c16 >= NPUSH))
      st_sys(wsl + READY_OFF + (size_t)(rank * 16 + u16) * 128 + c16 * 4, base + 1);
  }
  STAMP(1);

  if (b >= NGEMM + NPULL) {
    if (b == NWG - 1 && wv == 0) {
      bool ok = true;
#pragma unroll
      for (int i = 0; i < NGEMM / 64; i++) {
        const int c = i * 64 + lane;
        ok = wave_wait(me + DONE_OFF + c * 4, call, ((TILE_TAB[c] >> 2) & 7) != 0, me, call) && ok;
      }
      if (lane < 8) st_agent(me + QCTR_OFF + (size_t)(((call + 1) & 1) * 8 + lane) * 128, 0u);
      if (lane == 0) {
        if (!ok) a.out[rank][0] = 0x7f;
        st_agent(me + CTR_OFF, call);
      }
      STAMP(6);
    }
    return;
  }
  if (is_pull) {
    if (wv >= NPW) return;
    const unsigned src = (unsigned)(rank ^ plink);
    if (src >= NR) return;
    const __amdgpu_buffer_rsrc_t rraw = mkrsrc(a.ws[src], 2 * STG_KIND_BYTES);
    const __amdgpu_buffer_rsrc_t rd = mkrsrc(me, 2 * STG_KIND_BYTES);
    const u32x4 rss = mkrsrc_words(a.ws[src] + EXP_OFF, 16 * PUNIT_BYTES);
    const unsigned wperm = piece_off(1, 0, lane);
    char* scr = smem + 1024 + wv * BLK;
    constexpr unsigned SLOT = 3072;
    char* ring = smem + 1024 + NPW * BLK + wv * 2 * SLOT;
    unsigned long long rdy03 = 0;
    unsigned rdy4 = 0;
    bool ok = true;
    constexpr unsigned C0 = 2 * NPUSH, NITEM = (32 - C0) * 20;
    const unsigned qoff = (unsigned)(QCTR_OFF - FLAG_OFF) + ((call & 1) * 8 + plink) * 128;
    auto grab = [&]() {
      asm volatile("v_mov_b32 v200, 1\n\ts_nop 4\n\tbuffer_atomic_add v200, %0, %1, %2 offen sc0"
                   :: "v"(l0), "s"(rfs), "s"(qoff) : "v200", "memory");
    };
    auto ticket = [&]() -> unsigned {
      unsigned v;
      asm volatile("v_mov_b32 %0, v200" : "=v"(v) :: "v200", "memory");
      return __builtin_amdgcn_readfirstlane(v);
    };
    auto need = [&](unsigned n) {
      const unsigned cq = n / 20, c = C0 + cq, ui = (n - cq * 20) >> 2, c2 = c >> 1;
      for (Spin sp;;) {
        const unsigned m = ui == 4 ? rdy4 : (unsigned)(rdy03 >> (16 * ui));
        if (((m >> c2) & 1) || !ok) return;
        const unsigned u16 = (ue >> (4 * ui)) & 15;
        const unsigned v = lane < 16 ? ld_sys(me + READY_OFF + (size_t)(src * 16 + u16) * 128 + lane * 4) : 0u;
        const unsigned mk = (unsigned)__ballot(v >= base + 1) & 0xffffu;
        if (ui == 4) rdy4 = mk; else rdy03 |= (unsigned long long)mk << (16 * ui);
        if ((mk >> c2) & 1) return;
        if (sp.expired(me, call)) ok = false;
      }
    };
    const unsigned vpay = lane * 16, vlast = lane < 56 ? lane * 16 : 0x40000000u;
    auto load = [&](unsigned slot, unsigned n) {
      const bool val = n < NITEM;
      const unsigned nn = val ? n : 0;
      const unsigned cq = nn / 20, c = C0 + cq, r = nn - cq * 20, u16 = (ue >> (4 * (r >> 2))) & 15, w = r & 3;
      const unsigned oob = val ? 0u : 0x40000000u;
      const unsigned so = u16 * PUNIT_BYTES + (c * 4 + w) * PBLK;
      const unsigned lds = __builtin_amdgcn_readfirstlane((unsigned)(unsigned long long)(lu8*)(ring + slot * SLOT));
      unsigned keep;
      asm volatile("s_mov_b32 %0, m0\n\ts_mov_b32 m0, %1\n\ts_nop 0\n\t"
                   "buffer_load_dwordx4 %2, %4, %5 offen sc0 sc1 lds\n\t"
                   "buffer_load_dwordx4 %2, %4, %5 offen offset:1024 sc0 sc1 lds\n\t"
                   "buffer_load_dwordx4 %3, %4, %5 offen offset:2048 sc0 sc1 lds\n\t"
                   "s_mov_b32 m0, %0"
                   : "=&s"(keep) : "s"(lds), "v"(vpay | oob), "v"(vlast | oob), "s"(rss), "s"(so) : "memory");
    };
    auto emit = [&](unsigned slot, unsigned n) {
      const char* sl = ring + slot * SLOT;
      const unsigned h0 = __builtin_amdgcn_readfirstlane(*(const lu32*)(sl + 2816));
      const u32x4 A = *(const lu32x4*)(sl + lane * 16), B = *(const lu32x4*)(sl + 1024 + lane * 16);
      const unsigned S = *(const lu32*)(sl + 2048 + lane * 12), K0 = *(const lu32*)(sl + 2052 + lane * 12), K1 = *(const lu32*)(sl + 2056 + lane * 12);
      const unsigned cq = n / 20, c = C0 + cq, r = n - cq * 20, u16 = (ue >> (4 * (r >> 2))) & 15, w = r & 3;
      const int kind = u16 >> 3;
      const unsigned dst = (unsigned)stg_off(kind, src, u16 & 7) + (c * 4 + w) * BLK;
      if (h0 & 0x10000u) {
        u32x4 rr[4];
#pragma unroll
        for (int p = 0; p < 4; p++) rr[p] = __builtin_amdgcn_raw_buffer_load_b128(rraw, p * 1024 + lane * 16, dst, AUX_SYS);
#pragma unroll
        for (int p = 0; p < 4; p++) __builtin_amdgcn_raw_buffer_store_b128(rr[p], rd, p * 1024 + lane * 16, dst, AUX_AGENT);
        return;
      }
      const unsigned Md = (h0 & 0x80) ? 0x80808080u : 0u;
      const unsigned TG = (((h0 & 0x7f) * 0x01010101u | 0x80808080u) - 0x03020100u) | 0x80808080u;
      u32x4 o[4];
#pragma unroll
      for (int i = 0; i < 8; i++) {
        const unsigned Lp = i < 4 ? A[i] : B[i - 4];
        const unsigned code = ((i < 4 ? K0 : K1) >> (2 * (i & 3))) & 0x03030303u;
        const unsigned x = perm(TG, TG, code) - ((~Lp & Md) >> 7);
        const unsigned hi = (x & 0x7f7f7f7fu) | ((S << (7 - i)) & 0x80808080u);
        const unsigned L = Lp ^ Md;
        o[i >> 1][2 * (i & 1)] = perm(hi, L, 0x05010400u);
        o[i >> 1][2 * (i & 1) + 1] = perm(hi, L, 0x07030602u);
      }
      const unsigned cnt = (h0 >> 8) & 0xff;
      if (cnt == 0) {
#pragma unroll
        for (int p = 0; p < 4; p++)
          __builtin_amdgcn_raw_buffer_store_b128(o[p], rd, p * 1024 + (kind ? (wperm ^ (p * 16)) : lane * 16), dst, AUX_AGENT);
      } else {
#pragma unroll
        for (int p = 0; p < 4; p++) *(lu32x4*)(scr + p * 1024 + (kind ? (wperm ^ (p * 16)) : lane * 16)) = o[p];
        if (lane < (int)cnt) {
          const unsigned idx = *(const lu16*)(sl + 2816 + 4 + 2 * lane);
          *(lu8*)(scr + 2 * (idx & 0x7ff) + 1) = *(const lu8*)(sl + 2816 + 4 + 2 * NEXC + lane);
        }
#pragma unroll
        for (int p = 0; p < 4; p++) o[p] = *(const lu32x4*)(scr + p * 1024 + lane * 16);
#pragma unroll
        for (int p = 0; p < 4; p++) __builtin_amdgcn_raw_buffer_store_b128(o[p], rd, p * 1024 + lane * 16, dst, AUX_AGENT);
      }
    };
    auto arrive = [&](unsigned n) {
      const unsigned cq = n / 20, c = C0 + cq, r = n - cq * 20, u16 = (ue >> (4 * (r >> 2))) & 15;
      const unsigned so = (unsigned)(ARR_OFF - FLAG_OFF) + (src * 16 + u16) * 128 + c * 4;
      asm volatile("buffer_atomic_add %0, %1, %2, %3 offen" :: "v"(1u), "v"(l0), "s"(rfs), "s"(so) : "memory");
    };
    unsigned cur = ticket();
    unsigned nxt = cur + 1, prev = NITEM, slot = 0;
    grab();
    if (cur < NITEM) need(cur);
    STAMP(2);
    load(0, cur);
    while (cur < NITEM) {
      if (nxt < NITEM) need(nxt);
      load(slot ^ 1, nxt);
      asm volatile("s_waitcnt vmcnt(3)" ::: "memory");
      if (prev < NITEM) arrive(prev);
      const unsigned nn = ticket();
      grab();
      emit(slot, cur);
      prev = cur; cur = nxt; nxt = nn; slot ^= 1;
    }
    asm volatile("s_waitcnt vmcnt(0)" ::: "memory");
    if (prev < NITEM) arrive(prev);
    STAMP(3);
    WSTAMP(8 + (wv & 3));
    return;
  }

  const int t1 = (te >> 14) & 7, i1 = (te >> 11) & 7, t2 = (te >> 8) & 7, i2 = (te >> 5) & 7, tu = (te >> 2) & 7, pj = te & 3;
  const int srcW = rank ^ tu;
  char* const donep = a.ws[srcW & 7] + DONE_OFF + b * 4;
  lu32* sm = (lu32*)smem;
  static_assert(63 % NBUF != 0, "sm overlays stage buffer 0");
  if (wv >= 4) {
    const int lw = wv - 4;
    const int kind = lw >> 1;
    const int src = lw == 0 ? rank ^ t1 : (lw == 1 ? rank ^ t2 : srcW);
    const int unit = lw == 0 ? i1 : (lw == 1 ? i2 : 2 * pj + (lw & 1));
    const char* g = me + stg_off(kind, src, unit) + lane * 16;
    const bool loc = src == rank;
    const char* pollp = me + (loc ? READY_OFF : ARR_OFF) + (size_t)(((src & 7) * 2 + kind) * 8 + unit) * 128;
    const unsigned pthr = loc ? base + 1 : 4 * call;
    const int nword = loc ? 16 : 32;
    auto eval = [&](int c, unsigned v) -> int {
      const int w0 = loc ? c >> 1 : c;
      const unsigned long long nr = __ballot(lane < 16 && w0 + lane < nword && v < pthr);
      const int f = __builtin_ctz((unsigned)nr | 0x10000u);
      return min(32, loc ? 2 * (w0 + f) : w0 + f);
    };
    auto wordp = [&](int c) { return pollp + min((loc ? c >> 1 : c) + (lane & 15), nword - 1) * 4; };
    int avail = 0, pend = -1;
    int nb = 0;
    for (int s = -PF; s < 64; s++) {
      __builtin_amdgcn_s_barrier();
      asm volatile("" ::: "memory");
      if (s + PF < 64) {
        const int q = s + PF, c = q >> 1;
        if (c >= avail) {
          for (Spin sp;;) {
            avail = eval(c, ld_sys(wordp(c)));
            if (avail > c) break;
            if (sp.expired(me, call)) { avail = 64; break; }
          }
          if (c == 0 && lw == 0) WSTAMP(9);
        }
        const unsigned* gp = (const unsigned*)(g + (size_t)q * (2 * BLK));
        unsigned* lp = (unsigned*)(smem + nb * STAGE_BYTES + lw * (2 * BLK));
        nb = nb == NBUF - 1 ? 0 : nb + 1;
        __builtin_amdgcn_global_load_lds(gp, lp, 16, 0, AUX_SYS);
        __builtin_amdgcn_global_load_lds(gp, lp, 16, 1024, AUX_SYS);
        __builtin_amdgcn_global_load_lds(gp, lp, 16, 2048, AUX_SYS);
        __builtin_amdgcn_global_load_lds(gp, lp, 16, 3072, AUX_SYS);
        __builtin_amdgcn_global_load_lds(gp + 1024, lp + 1024, 16, 0, AUX_SYS);
        __builtin_amdgcn_global_load_lds(gp + 1024, lp + 1024, 16, 1024, AUX_SYS);
        __builtin_amdgcn_global_load_lds(gp + 1024, lp + 1024, 16, 2048, AUX_SYS);
        __builtin_amdgcn_global_load_lds(gp + 1024, lp + 1024, 16, 3072, AUX_SYS);
        if ((q & 1) == 0) {
          if (avail < 32 && avail <= c + LOOK) {
            asm volatile("global_load_dword v200, %0, off sc0 sc1" :: "v"(wordp(avail)) : "v200", "memory");
            pend = avail;
          }
        } else if (pend >= 0) {
          unsigned pv;
          asm volatile("s_waitcnt vmcnt(8)\n\tv_mov_b32 %0, v200" : "=v"(pv) :: "v200", "memory");
          avail = max(avail, eval(pend, pv));
          pend = -1;
        }
        if (PF == 4) asm volatile("s_waitcnt vmcnt(24)" ::: "memory");
        else if (PF == 3) asm volatile("s_waitcnt vmcnt(16)" ::: "memory");
        else asm volatile("s_waitcnt vmcnt(8)" ::: "memory");
      } else {
        const int rem = 62 - s;
        if (rem >= 2) asm volatile("s_waitcnt vmcnt(16)" ::: "memory");
        else if (rem == 1) asm volatile("s_waitcnt vmcnt(8)" ::: "memory");
        else asm volatile("s_waitcnt vmcnt(0)" ::: "memory");
      }
    }
    return;
  }
  const int xu = wv >> 1, wu = wv & 1;
  f32x4 acc[4][4];
#pragma unroll
  for (int f = 0; f < 4; f++)
#pragma unroll
    for (int j = 0; j < 4; j++) acc[f][j] = f32x4{0.f, 0.f, 0.f, 0.f};
  const unsigned xoff = xu * (2 * BLK) + lane * 16;
  unsigned woff[4];
  {
    const int r = lane & 3, q = (lane >> 2) & 3, kq = lane >> 4;
#pragma unroll
    for (int j = 0; j < 4; j++) woff[j] = (2 + wu) * (2 * BLK) + r * 1024 + kq * 256 + q * 64 + ((j ^ r) * 16);
  }
  {
    bf16x8 xa[4], wa[4], xb[4], wb[4];
#pragma unroll
    for (int f = 0; f < 4; f++) { xb[f] = bf16x8{}; wb[f] = bf16x8{}; }
    for (int s = -PF; s < 0; s++) __builtin_amdgcn_s_barrier();
    int cb = 0;
#define ILV() do { _Pragma("unroll") for (int q_ = 0; q_ < 8; q_++) { __builtin_amdgcn_sched_group_barrier(0x008, 1, 0); \
    __builtin_amdgcn_sched_group_barrier(0x100, 1, 0); } __builtin_amdgcn_sched_group_barrier(0x008, 8, 0); } while (0)
    for (int s = 0; s < 64; s++) {
      asm volatile("s_waitcnt lgkmcnt(0)" ::: "memory");
      __builtin_amdgcn_s_barrier();
      asm volatile("" ::: "memory");
      if (s == 0) STAMP(2);
      if (s == 56) STAMP(11);
      if (s == 60) STAMP(12);
      if (s == 62) STAMP(13);
      const char* sb = smem + cb * STAGE_BYTES;
      cb = cb == NBUF - 1 ? 0 : cb + 1;
      __builtin_amdgcn_sched_barrier(0);
      wa[0] = *(const bf16x8*)(sb + woff[0]); xa[0] = *(const bf16x8*)(sb + xoff);
#pragma unroll
      for (int f = 1; f < 4; f++) wa[f] = *(const bf16x8*)(sb + woff[f]);
#pragma unroll
      for (int f = 1; f < 4; f++) xa[f] = *(const bf16x8*)(sb + xoff + f * 1024);
#pragma unroll
      for (int f = 0; f < 4; f++)
#pragma unroll
        for (int j = 0; j < 4; j++) acc[f][j] = __builtin_amdgcn_mfma_f32_16x16x32_bf16(wb[j], xb[f], acc[f][j], 0, 0, 0);
      ILV();
      __builtin_amdgcn_sched_barrier(0);
      wb[0] = *(const bf16x8*)(sb + woff[0] + BLK); xb[0] = *(const bf16x8*)(sb + xoff + BLK);
#pragma unroll
      for (int f = 1; f < 4; f++) wb[f] = *(const bf16x8*)(sb + woff[f] + BLK);
#pragma unroll
      for (int f = 1; f < 4; f++) xb[f] = *(const bf16x8*)(sb + xoff + BLK + f * 1024);
#pragma unroll
      for (int f = 0; f < 4; f++)
#pragma unroll
        for (int j = 0; j < 4; j++) acc[f][j] = __builtin_amdgcn_mfma_f32_16x16x32_bf16(wa[j], xa[f], acc[f][j], 0, 0, 0);
      ILV();
      __builtin_amdgcn_sched_barrier(0);
    }
#pragma unroll
    for (int f = 0; f < 4; f++)
#pragma unroll
      for (int j = 0; j < 4; j++) acc[f][j] = __builtin_amdgcn_mfma_f32_16x16x32_bf16(wb[j], xb[f], acc[f][j], 0, 0, 0);
  }
  STAMP(3);
  float mf = 0.f;
#pragma unroll
  for (int f = 0; f < 4; f++)
#pragma unroll
    for (int j = 0; j < 4; j++) {
      const f32x4 t = acc[f][j];
      mf = fmaxf(fmaxf(mf, fabsf(t[0])), fabsf(t[1]));
      mf = fmaxf(fmaxf(mf, fabsf(t[2])), fabsf(t[3]));
    }
  unsigned m = wave_max_u32(__float_as_uint(mf));
  if (lane == 0) sm[wv] = m;
  asm volatile("s_waitcnt lgkmcnt(0)" ::: "memory");
  __builtin_amdgcn_s_barrier();
  asm volatile("" ::: "memory");
  m = max(max(sm[0], sm[1]), max(sm[2], sm[3]));
  bool bad = false;
  if (tid < NR) {
    gu64* g = (gu64*)(wsl + AMAX_OFF + (size_t)(rank * NGEMM + b) * 8);
    __hip_atomic_store(g, ((unsigned long long)call << 32) | m, __ATOMIC_RELAXED, __HIP_MEMORY_SCOPE_SYSTEM);
  }
  unsigned gm = 0;
  {
    const char* gp = me + AMAX_OFF + tid * 32;
    Spin sp;
    bool have = false;
    for (;;) {
      unsigned tm = 0;
      if (!have) {
        u32x4 g0, g1;
        asm volatile("global_load_dwordx4 %0, %3, off sc0 sc1\n\tglobal_load_dwordx4 %1, %3, off offset:16 sc0 sc1\n\t"
                     "global_load_dword %2, %4, off sc1\n\ts_waitcnt vmcnt(0)"
                     : "=&v"(g0), "=&v"(g1), "=&v"(tm) : "v"(gp), "v"(me + TMO_OFF) : "memory");
        have = g0[1] == call && g0[3] == call && g1[1] == call && g1[3] == call;
        gm = max(max(g0[0], g0[2]), max(g1[0], g1[2]));
      }
      if (__all(have)) {
        if (__any(tm == call)) bad = true;
        break;
      }
      if (sp.expired(me, call)) { bad = true; break; }
    }
  }
  gm = wave_max_u32(gm);
  if (lane == 0) { sm[4 + wv] = gm; sm[8 + wv] = bad; }
  __syncthreads();
  gm = max(max(sm[4], sm[5]), max(sm[6], sm[7]));
  bad = (sm[8] | sm[9] | sm[10] | sm[11]) != 0;
  STAMP(4);
  const float inv = 448.0f / __uint_as_float(gm);
  {
    const int tx = xu == 0 ? t1 : t2, ix = xu == 0 ? i1 : i2;
    unsigned char* o = outp + (size_t)((rank ^ tx) * SH + ix * 64 + (lane & 15)) * SH + pj * 128 + wu * 64 + (lane >> 4) * 16;
#pragma unroll
    for (int f = 0; f < 4; f++) {
      u32x4 q;
#pragma unroll
      for (int j = 0; j < 4; j++) {
        int w = __builtin_amdgcn_cvt_pk_fp8_f32(acc[f][j][0] * inv, acc[f][j][1] * inv, 0, false);
        w = __builtin_amdgcn_cvt_pk_fp8_f32(acc[f][j][2] * inv, acc[f][j][3] * inv, w, true);
        q[j] = bad ? 0x7f7f7f7fu : (unsigned)w;
      }
      if (tu != 0) asm volatile("global_store_dwordx4 %0, %1, off sc0 sc1\n\ts_nop 1" ::"v"(o + (size_t)f * 16 * SH), "v"(q) : "memory");
      else *(u32x4*)(o + (size_t)f * 16 * SH) = q;
    }
  }
  if (tu != 0) {
    asm volatile("s_waitcnt vmcnt(0)" ::: "memory");
    __builtin_amdgcn_s_barrier();
    if (tid == 0) st_sys(donep, call);
  }
  STAMP(5);
}
}

extern "C" void kernel_launch(int rank, int nranks, ncclComm_t comm, hipStream_t stream, void* const* d_in,
                              const int* in_sizes, int n_in, void* d_out, int out_size, const int* in_dtypes,
                              int out_dtype, void* const* d_out_all, void* const* d_ws_all, size_t ws_bytes) {
  (void)comm; (void)in_sizes; (void)n_in; (void)d_out; (void)out_size; (void)in_dtypes; (void)out_dtype;
  if (nranks != NR || rank < 0 || rank >= NR || ws_bytes < WS_NEED) {
    fprintf(stderr, "unsupported configuration\n");
    return;
  }
  Args a;
  a.x = (const float*)d_in[0];
  a.w = (const float*)d_in[1];
  for (int p = 0; p < NR; p++) {
    a.ws[p] = (char*)d_ws_all[p];
    a.out[p] = (unsigned char*)d_out_all[p];
  }
  a.rank = rank;
  a.pad = 0;
  fused_ag_gemm<<<NWG, 512, 0, stream>>>(a);
  hipError_t e = hipGetLastError();
  if (e != hipSuccess) fprintf(stderr, "launch error %s\n", hipGetErrorString(e));
}
```

```cpp
#include <hip/hip_runtime.h>
#include <rccl/rccl.h>
#include <cstdio>
#include <cstdint>

typedef unsigned u32x4 __attribute__((ext_vector_type(4)));
typedef unsigned u32x3 __attribute__((ext_vector_type(3)));
typedef unsigned u32x2 __attribute__((ext_vector_type(2)));
typedef float f32x4 __attribute__((ext_vector_type(4)));
typedef __bf16 bf16x8 __attribute__((ext_vector_type(8)));
typedef __bf16 bf16x2 __attribute__((ext_vector_type(2)));
typedef float f32x2 __attribute__((ext_vector_type(2)));

#define NPW 5
#define NPUSH 1
#define LOOK 1
#define STAG0 60
#define STAGGER 25

namespace {
constexpr int NR = 8;
constexpr int KDIM = 4096;
constexpr int SH = 512;
constexpr unsigned BLK = 4096;
constexpr unsigned UNIT_BYTES = 128 * BLK;
constexpr unsigned PBLK = 2944;
constexpr unsigned PUNIT_BYTES = 128 * PBLK;
constexpr unsigned NEXC = 41;
constexpr size_t STG_KIND_BYTES = (size_t)NR * 8 * UNIT_BYTES;
constexpr size_t EXP_OFF = 2 * STG_KIND_BYTES;
constexpr size_t FLAG_OFF = EXP_OFF + (size_t)16 * PUNIT_BYTES;
constexpr size_t READY_OFF = FLAG_OFF;
constexpr size_t ARR_OFF = READY_OFF + NR * 16 * 128;
constexpr size_t AMAX_OFF = ARR_OFF + NR * 16 * 256;
constexpr size_t DONE_OFF = AMAX_OFF + NR * 128 * 8;
constexpr size_t CTR_OFF = DONE_OFF + 1024;
constexpr size_t TMO_OFF = CTR_OFF + 128;
constexpr size_t QCTR_OFF = TMO_OFF + 128;
constexpr size_t RES_OFF = QCTR_OFF + 16 * 128;
constexpr size_t END_OFF = RES_OFF + 8 * 128;
constexpr size_t WS_NEED = END_OFF;
static_assert(FLAG_OFF % 128 == 0, "align");
constexpr int NGEMM = 128, NPULL = 35, NWG = 256;
constexpr unsigned long long DEADLINE = 30000000ull;
constexpr unsigned STAGE_BYTES = 8 * BLK;
#define NBUFS 4
constexpr int NBUF = NBUFS;
constexpr int PF = NBUF - 1;

__constant__ unsigned TILE_TAB[NGEMM] = {16420,2145,5154,5283,14530,10241,8928,10432,69476,67649,7906,47971,70370,15553,17664,15424,3116,74817,8450,78019,77058,16577,113792,55360,18284,75905,12130,83011,3238,113377,121888,75872,70924,82785,73762,70820,13574,1059,3115,77856,77964,5158,78690,82660,69798,18147,7435,4311,8207,55430,3939,112708,78566,76,47243,47287,121935,75622,6307,75,112902,11404,113803,55351,1081,3112,73859,11115,10991,13484,4373,69751,9433,77064,10340,75979,12335,48876,9973,12377,12153,121928,12420,78923,76015,9271,67637,47225,17561,5167,74948,55518,79599,74935,73845,75833,83033,75023,6216,70846,11477,78103,77685,2234,2238,2173,12008,78590,69845,12381,47258,55386,8478,70813,73864,81950,9501,48349,68890,69850,68766,76669,78024,112734,10493,69373,75642,79034};
__constant__ unsigned PULL_TAB[8] = {0, 0xdc980, 0xfe987, 0xfe983, 0x65421, 0xfeba0, 0xdcba7, 0xdcba3};
constexpr unsigned long long DEST_MASK_X = 0x4511111189111123ull, DEST_MASK_W = 0x2d2dc3c3e1e10f0full;

struct Args {
  const float* x;
  const float* w;
  char* ws[NR];
  unsigned char* out[NR];
  int rank;
  int pad;
};

typedef __attribute__((address_space(1))) unsigned gu32;
typedef __attribute__((address_space(1))) unsigned long long gu64;
typedef __attribute__((address_space(3))) unsigned lu32;
typedef __attribute__((address_space(3))) unsigned char lu8;
typedef __attribute__((address_space(3))) unsigned short lu16;
typedef __attribute__((address_space(3))) u32x4 lu32x4;

__device__ __forceinline__ unsigned ld_sys(const void* p) { return __hip_atomic_load((const gu32*)p, __ATOMIC_RELAXED, __HIP_MEMORY_SCOPE_SYSTEM); }
__device__ __forceinline__ void st_sys(void* p, unsigned v) { __hip_atomic_store((gu32*)p, v, __ATOMIC_RELAXED, __HIP_MEMORY_SCOPE_SYSTEM); }
__device__ __forceinline__ unsigned ld_agent(const void* p) { return __hip_atomic_load((const gu32*)p, __ATOMIC_RELAXED, __HIP_MEMORY_SCOPE_AGENT); }
__device__ __forceinline__ void st_agent(void* p, unsigned v) { __hip_atomic_store((gu32*)p, v, __ATOMIC_RELAXED, __HIP_MEMORY_SCOPE_AGENT); }

__device__ __forceinline__ __amdgpu_buffer_rsrc_t mkrsrc(const void* p, unsigned bytes) {
  unsigned long long a = (unsigned long long)p;
  unsigned lo = __builtin_amdgcn_readfirstlane((unsigned)a), hi = __builtin_amdgcn_readfirstlane((unsigned)(a >> 32));
  void* q = (void*)(((unsigned long long)hi << 32) | lo);
  return __builtin_amdgcn_make_buffer_rsrc(q, 0, bytes, 0x00020000);
}
__device__ __forceinline__ u32x4 mkrsrc_words(const void* p, unsigned bytes) {
  unsigned long long a = (unsigned long long)p;
  return u32x4{(unsigned)__builtin_amdgcn_readfirstlane((unsigned)a), (unsigned)__builtin_amdgcn_readfirstlane((unsigned)(a >> 32)) & 0xffffu,
               bytes, 0x00020000u};
}
__device__ __forceinline__ size_t stg_off(int kind, int src, int unit) {
  return (size_t)kind * STG_KIND_BYTES + (size_t)(src * 8 + unit) * UNIT_BYTES;
}
__device__ __forceinline__ unsigned cvt2(float a, float b) {
  f32x2 v = {a, b};
  bf16x2 r = __builtin_convertvector(v, bf16x2);
  return __builtin_bit_cast(unsigned, r);
}
__device__ __forceinline__ unsigned perm(unsigned hi, unsigned lo, unsigned sel) { return __builtin_amdgcn_perm(hi, lo, sel); }
__device__ __forceinline__ unsigned wave_max_u32(unsigned v) {
  v = max(v, (unsigned)__builtin_amdgcn_update_dpp(0, (int)v, 0x111, 0xf, 0xf, false));
  v = max(v, (unsigned)__builtin_amdgcn_update_dpp(0, (int)v, 0x112, 0xf, 0xf, false));
  v = max(v, (unsigned)__builtin_amdgcn_update_dpp(0, (int)v, 0x114, 0xf, 0xf, false));
  v = max(v, (unsigned)__builtin_amdgcn_update_dpp(0, (int)v, 0x118, 0xf, 0xf, false));
  v = max(v, (unsigned)__builtin_amdgcn_update_dpp(0, (int)v, 0x142, 0xa, 0xf, false));
  v = max(v, (unsigned)__builtin_amdgcn_update_dpp(0, (int)v, 0x143, 0xc, 0xf, false));
  return (unsigned)__builtin_amdgcn_readlane((int)v, 63);
}

struct Spin {
  unsigned it = 0;
  unsigned long long t0 = 0;
  __device__ __forceinline__ bool expired(char* me, unsigned call) {
    if (it == 0) t0 = __builtin_amdgcn_s_memrealtime();
    if ((++it & 15) == 0) {
      if (ld_agent(me + TMO_OFF) == call || __builtin_amdgcn_s_memrealtime() - t0 > DEADLINE) {
        st_agent(me + TMO_OFF, call);
        return true;
      }
    }
    __builtin_amdgcn_s_sleep(1);
    return false;
  }
};
constexpr int AUX_SYS = 17, AUX_AGENT = 16;

__device__ __forceinline__ unsigned piece_off(int kind, int p, int lane) {
  const int kq = lane >> 4, q = (lane >> 2) & 3, j = lane & 3;
  return kind == 0 ? p * 1024 + lane * 16 : p * 1024 + kq * 256 + q * 64 + ((j ^ p) * 16);
}

__global__ __launch_bounds__(512, 2) void fused_ag_gemm(Args a) {
  __shared__ __attribute__((aligned(16))) char smem[NBUF * STAGE_BYTES + 128];
  const int tid = threadIdx.x, lane = tid & 63;
  const int wv = __builtin_amdgcn_readfirstlane(tid >> 6);
  const int b = blockIdx.x;
  constexpr int NH0 = 16 * NPUSH;
  const int task = b < NGEMM ? b + NH0 + NPULL
                             : (b < NGEMM + NPULL ? b - NGEMM + NH0 : (b - (NGEMM + NPULL) < NH0 ? b - (NGEMM + NPULL) : b));
  const int c16 = task >> 4, u16 = task & 15, kind = u16 >> 3, unit = u16 & 7;
  const int ks = c16 * 8 + wv;
  const int kq = lane >> 4, l15 = lane & 15;
  if (c16 >= 2) {
    const unsigned long long tstart = __builtin_amdgcn_s_memrealtime();
    while (__builtin_amdgcn_s_memrealtime() - tstart < (unsigned long long)(STAG0 + (c16 - 2) * STAGGER)) __builtin_amdgcn_s_sleep(1);
  }
  f32x4 v[8];
  {
    const float* px = a.x + (size_t)(unit * 64 + l15) * KDIM + ks * 32 + 4 * kq;
    const float* pw = a.w + (size_t)(ks * 32 + 4 * kq) * SH + unit * 64 + 4 * l15;
    const float* p = kind == 0 ? px : pw;
#pragma unroll
    for (int i = 0; i < 8; i++) {
      const unsigned ox = (i >> 1) * 16 * KDIM + (i & 1) * 16, ow = ((i & 3) + (i >> 2) * 16) * SH;
      v[i] = *(const f32x4*)(p + (kind == 0 ? ox : ow));
    }
  }
  asm volatile("" ::: "memory");
  const int rank = a.rank;
  char* me = a.ws[rank];
  const unsigned ctr_v = ld_agent(me + CTR_OFF);
  const unsigned te = TILE_TAB[b & (NGEMM - 1)];
  const bool is_pull = b >= NGEMM && b < NGEMM + NPULL;
  const int plink = is_pull ? 1 + (b - NGEMM) / 5 : 0;
  const unsigned ue = PULL_TAB[plink];
  char* const wsl = a.ws[lane & 7];
  unsigned char* const outp = a.out[(rank ^ (int)(te >> 2)) & 7];
  const u32x4 rfs = mkrsrc_words(me + FLAG_OFF, (unsigned)(END_OFF - FLAG_OFF));
  const unsigned l0 = lane == 0 ? 0u : 0x40000000u;
  unsigned call, base;
  {
    const __amdgpu_buffer_rsrc_t rd = mkrsrc(me + stg_off(kind, rank, unit), UNIT_BYTES);
    call = __builtin_amdgcn_readfirstlane(ctr_v) + 1;
    base = call * 64;
    if (is_pull && wv < NPW) {
      const unsigned qoff = (unsigned)(QCTR_OFF - FLAG_OFF) + ((call & 1) * 8 + plink) * 128;
      asm volatile("v_mov_b32 v200, 2\n\ts_nop 4\n\tbuffer_atomic_add v200, %0, %1, %2 offen sc0"
                   :: "v"(l0), "s"(rfs), "s"(qoff) : "v200", "memory");
    }
    u32x4 o[4];
    if (kind == 0) {
#pragma unroll
      for (int f = 0; f < 4; f++)
        o[f] = u32x4{cvt2(v[2 * f][0], v[2 * f][1]), cvt2(v[2 * f][2], v[2 * f][3]),
                     cvt2(v[2 * f + 1][0], v[2 * f + 1][1]), cvt2(v[2 * f + 1][2], v[2 * f + 1][3])};
    } else {
#pragma unroll
      for (int r = 0; r < 4; r++)
        o[r] = u32x4{cvt2(v[0][r], v[1][r]), cvt2(v[2][r], v[3][r]), cvt2(v[4][r], v[5][r]), cvt2(v[6][r], v[7][r])};
    }
    const unsigned dmask = (unsigned)((kind ? DEST_MASK_W : DEST_MASK_X) >> (8 * unit)) & 0xff;
    if (c16 < NPUSH) {
#pragma unroll
      for (int p = 1; p < NR; p++)
        if ((dmask >> p) & 1) {
          const __amdgpu_buffer_rsrc_t rq = mkrsrc(a.ws[rank ^ p] + stg_off(kind, rank, unit), UNIT_BYTES);
#pragma unroll
          for (int i = 0; i < 4; i++)
            __builtin_amdgcn_raw_buffer_store_b128(o[i], rq, ks * BLK + piece_off(kind, i, lane), 0, AUX_SYS);
        }
    } else {
      float mx = 0.f;
#pragma unroll
      for (int i = 0; i < 8; i++) {
        mx = fmaxf(mx, fmaxf(fabsf(v[i][0]), fabsf(v[i][1])));
        mx = fmaxf(mx, fmaxf(fabsf(v[i][2]), fabsf(v[i][3])));
      }
      const unsigned mxb = wave_max_u32(__float_as_uint(mx));
      const unsigned emax = (cvt2(__uint_as_float(mxb), 0.f) >> 7) & 0xff;
      const float thr1 = __uint_as_float(emax << 23), thr2 = thr1 * 0.5f;
      unsigned n1 = 0, n2 = 0;
#pragma unroll
      for (int i = 0; i < 8; i++) {
        n1 += __builtin_popcountll(__ballot(fabsf(v[i][0]) >= thr1)) + __builtin_popcountll(__ballot(fabsf(v[i][2]) >= thr1));
        n2 += __builtin_popcountll(__ballot(fabsf(v[i][0]) >= thr2)) + __builtin_popcountll(__ballot(fabsf(v[i][2]) >= thr2));
      }
      const int top = (int)emax - (n1 <= 10 ? 1 : 0) - (n2 <= 10 ? 1 : 0);
      const unsigned delta = (top & 1) ^ 1, tp = ((top + (int)delta) >> 1) & 0x7f;
      const unsigned Md = delta ? 0x80808080u : 0u;
      const unsigned b80 = tp * 0x01010101u | 0x80808080u;
      const unsigned T = (b80 - 0x03020100u) & 0x7f7f7f7fu;
      unsigned LP[8], HH[8], S = 0, K0 = 0, K1 = 0, M = 0;
#pragma unroll
      for (int i = 0; i < 8; i++) {
        const unsigned d0 = o[i >> 1][2 * (i & 1)], d1 = o[i >> 1][2 * (i & 1) + 1];
        const unsigned L = perm(d1, d0, 0x06040200u), H = perm(d1, d0, 0x07050301u);
        const unsigned pair = ((H & 0x7f7f7f7fu) + ((L & Md) >> 7)) & 0x7f7f7f7fu;
        const unsigned code = (b80 - pair) & 0x03030303u;
        const unsigned mis = perm(T, T, code) ^ pair;
        LP[i] = L ^ Md;
        HH[i] = H;
        if (i < 4) K0 |= code << (2 * i); else K1 |= code << (2 * (i - 4));
        S |= (H & 0x80808080u) >> (7 - i);
        M |= __builtin_amdgcn_udot4(((mis + 0x7f7f7f7fu) & 0x80808080u) >> 7, 0x08040201u, 0u, false) << (4 * i);
      }
      char* hsc = smem + wv * 2304;
      lu32* hdr = (lu32*)(hsc + 2048);
      const unsigned nl = __builtin_popcount(M);
      unsigned inc = nl;
      inc += __builtin_amdgcn_update_dpp(0, (int)inc, 0x111, 0xf, 0xf, false);
      inc += __builtin_amdgcn_update_dpp(0, (int)inc, 0x112, 0xf, 0xf, false);
      inc += __builtin_amdgcn_update_dpp(0, (int)inc, 0x114, 0xf, 0xf, false);
      inc += __builtin_amdgcn_update_dpp(0, (int)inc, 0x118, 0xf, 0xf, false);
      inc += __builtin_amdgcn_update_dpp(0, (int)inc, 0x142, 0xa, 0xf, false);
      inc += __builtin_amdgcn_update_dpp(0, (int)inc, 0x143, 0xc, 0xf, false);
      const unsigned cnt = (unsigned)__builtin_amdgcn_readlane((int)inc, 63);
      if (cnt > 0 && cnt <= NEXC) {
        *(lu32x4*)(hsc + lane * 32) = u32x4{HH[0], HH[1], HH[2], HH[3]};
        *(lu32x4*)(hsc + lane * 32 + 16) = u32x4{HH[4], HH[5], HH[6], HH[7]};
        unsigned at = inc - nl;
        while (__any(M != 0)) {
          if (M != 0) {
            const unsigned jv = __builtin_ctz(M);
            M &= M - 1;
            const unsigned char hi = *(const lu8*)(hsc + lane * 32 + jv);
            const unsigned idx = (piece_off(kind, jv >> 3, lane) >> 1) + (jv & 7);
            *(lu16*)((char*)hdr + 4 + 2 * at) = (unsigned short)idx;
            *(lu8*)((char*)hdr + 4 + 2 * NEXC + at) = hi;
            at++;
          }
        }
      }
      const __amdgpu_buffer_rsrc_t rp = mkrsrc(me + EXP_OFF + (size_t)u16 * PUNIT_BYTES, PUNIT_BYTES);
      const unsigned pc = ks * PBLK;
      __builtin_amdgcn_raw_buffer_store_b128(u32x4{LP[0], LP[1], LP[2], LP[3]}, rp, pc + lane * 16, 0, AUX_SYS);
      __builtin_amdgcn_raw_buffer_store_b128(u32x4{LP[4], LP[5], LP[6], LP[7]}, rp, pc + 1024 + lane * 16, 0, AUX_SYS);
      __builtin_amdgcn_raw_buffer_store_b96(u32x3{S, K0, K1}, rp, pc + 2048 + lane * 12, 0, AUX_SYS);
      unsigned hv = hdr[lane & 31];
      if (lane == 0) hv = tp | (delta << 7) | (cnt > NEXC ? 0x10000u : cnt << 8);
      if (lane < 32) __builtin_amdgcn_raw_buffer_store_b32(hv, rp, pc + 2816 + lane * 4, 0, AUX_SYS);
    }
#pragma unroll
    for (int p = 0; p < 4; p++)
      __builtin_amdgcn_raw_buffer_store_b128(o[p], rd, ks * BLK + piece_off(kind, p, lane), 0, AUX_SYS);
    asm volatile("s_waitcnt vmcnt(0)" ::: "memory");
    const int rel = rank ^ lane;
    if (c16 < NPUSH) {
      if (lane < NR && rel != 0 && ((dmask >> rel) & 1))
        __hip_atomic_fetch_add((gu32*)(wsl + ARR_OFF + (size_t)(rank * 16 + u16) * 256 + (ks >> 1) * 4), 1u, __ATOMIC_RELAXED,
                               __HIP_MEMORY_SCOPE_SYSTEM);
    }
    __builtin_amdgcn_s_barrier();
    if (wv == 0 && lane < NR && ((dmask >> rel) & 1) && (rel == 0 || c16 >= NPUSH))
      st_sys(wsl + READY_OFF + (size_t)(rank * 16 + u16) * 128 + c16 * 4, base + 1);
  }

  if (b >= NGEMM + NPULL) {
    if (b == NWG - 1) {
      {
        lu32* sv = (lu32*)smem;
        const char* gp = me + AMAX_OFF + (size_t)tid * 8;
        bool have = false;
        unsigned gm = 0;
        __builtin_amdgcn_s_barrier();
        int par = 0;
        for (Spin sp;; par ^= 1) {
          unsigned tm = 0;
          if (!have) {
            u32x2 g0, g1;
            asm volatile("global_load_dwordx2 %0, %3, off sc0 sc1\n\tglobal_load_dwordx2 %1, %4, off sc0 sc1\n\t"
                         "global_load_dword %2, %5, off sc1\n\ts_waitcnt vmcnt(0)"
                         : "=&v"(g0), "=&v"(g1), "=&v"(tm) : "v"(gp), "v"(gp + 512 * 8), "v"(me + TMO_OFF) : "memory");
            have = g0[1] == call && g1[1] == call;
            gm = max(g0[1] == call ? g0[0] : 0u, g1[1] == call ? g1[0] : 0u);
          }
          const unsigned wm = wave_max_u32(gm);
          unsigned miss = 2 * __builtin_popcountll(__ballot(!have));
          if (__any(tm == call) || (miss != 0 && sp.expired(me, call))) miss = 0x10000;
          if (lane == 0) { sv[par * 16 + wv * 2] = wm; sv[par * 16 + wv * 2 + 1] = miss; }
          asm volatile("s_waitcnt lgkmcnt(0)" ::: "memory");
          __builtin_amdgcn_s_barrier();
          asm volatile("" ::: "memory");
          unsigned am = 0, mt = 0;
#pragma unroll
          for (int i = 0; i < 8; i++) { am = max(am, sv[par * 16 + 2 * i]); mt += sv[par * 16 + 2 * i + 1]; }
          am = __builtin_amdgcn_readfirstlane(am);
          mt = __builtin_amdgcn_readfirstlane(mt);
          if (mt == 0 || mt >= 0x10000) {
            if (tid < 8) __hip_atomic_store((gu64*)(me + RES_OFF + tid * 128), ((unsigned long long)call << 32) | (mt ? 0x7fffffffu : am),
                                            __ATOMIC_RELAXED, __HIP_MEMORY_SCOPE_AGENT);
            break;
          }
        }
      }
      if (wv == 0) {
        bool ok = true;
        {
          const bool a0 = ((TILE_TAB[2 * lane] >> 2) & 7) != 0, a1 = ((TILE_TAB[2 * lane + 1] >> 2) & 7) != 0;
          for (Spin sp;;) {
            u32x2 d;
            asm volatile("global_load_dwordx2 %0, %1, off sc0 sc1\n\ts_waitcnt vmcnt(0)" : "=v"(d) : "v"(me + DONE_OFF + lane * 8) : "memory");
            if (__all((!a0 || d[0] >= call) && (!a1 || d[1] >= call))) break;
            if (sp.expired(me, call)) { ok = false; break; }
          }
        }
        if (lane < 8) st_agent(me + QCTR_OFF + (size_t)(((call + 1) & 1) * 8 + lane) * 128, 0u);
        if (lane == 0) {
          if (!ok) a.out[rank][(size_t)(rank * SH + 64) * SH + 128] = 0x7f;
          st_agent(me + CTR_OFF, call);
        }
      }
    }
    return;
  }
  if (is_pull) {
    if (wv >= NPW) return;
    const unsigned src = (unsigned)(rank ^ plink);
    if (src >= NR) return;
    const __amdgpu_buffer_rsrc_t rraw = mkrsrc(a.ws[src], 2 * STG_KIND_BYTES);
    const __amdgpu_buffer_rsrc_t rd = mkrsrc(me, 2 * STG_KIND_BYTES);
    const u32x4 rss = mkrsrc_words(a.ws[src] + EXP_OFF, 16 * PUNIT_BYTES);
    const unsigned wperm = piece_off(1, 0, lane);
    char* scr = smem + 1024 + wv * BLK;
    constexpr unsigned SLOT = 3072;
    char* ring = smem + 1024 + NPW * BLK + wv * 2 * SLOT;
    unsigned long long rdy03 = 0;
    unsigned rdy4 = 0;
    bool ok = true;
    constexpr unsigned C0 = 2 * NPUSH, NITEM = (32 - C0) * 20;
    const unsigned qoff = (unsigned)(QCTR_OFF - FLAG_OFF) + ((call & 1) * 8 + plink) * 128;
    auto grab = [&]() {
      asm volatile("v_mov_b32 v200, 1\n\ts_nop 4\n\tbuffer_atomic_add v200, %0, %1, %2 offen sc0"
                   :: "v"(l0), "s"(rfs), "s"(qoff) : "v200", "memory");
    };
    auto ticket = [&]() -> unsigned {
      unsigned v;
      asm volatile("v_mov_b32 %0, v200" : "=v"(v) :: "v200", "memory");
      return __builtin_amdgcn_readfirstlane(v);
    };
    auto need = [&](unsigned n) {
      const unsigned cq = n / 20, c = C0 + cq, ui = (n - cq * 20) % 5, c2 = c >> 1;
      for (Spin sp;;) {
        const unsigned m = ui == 4 ? rdy4 : (unsigned)(rdy03 >> (16 * ui));
        if (((m >> c2) & 1) || !ok) return;
        const unsigned u16 = (ue >> (4 * ui)) & 15;
        const unsigned v = lane < 16 ? ld_sys(me + READY_OFF + (size_t)(src * 16 + u16) * 128 + lane * 4) : 0u;
        const unsigned mk = (unsigned)__ballot(v >= base + 1) & 0xffffu;
        if (ui == 4) rdy4 = mk; else rdy03 |= (unsigned long long)mk << (16 * ui);
        if ((mk >> c2) & 1) return;
        if (sp.expired(me, call)) ok = false;
      }
    };
    const unsigned vpay = lane * 16, vlast = lane < 56 ? lane * 16 : 0x40000000u;
    auto load = [&](unsigned slot, unsigned n) {
      const bool val = n < NITEM;
      const unsigned nn = val ? n : 0;
      const unsigned cq = nn / 20, c = C0 + cq, r = nn - cq * 20, w = r / 5, u16 = (ue >> (4 * (r - w * 5))) & 15;
      const unsigned oob = val ? 0u : 0x40000000u;
      const unsigned so = u16 * PUNIT_BYTES + (c * 4 + w) * PBLK;
      const unsigned lds = __builtin_amdgcn_readfirstlane((unsigned)(unsigned long long)(lu8*)(ring + slot * SLOT));
      unsigned keep;
      asm volatile("s_mov_b32 %0, m0\n\ts_mov_b32 m0, %1\n\ts_nop 0\n\t"
                   "buffer_load_dwordx4 %2, %4, %5 offen sc0 sc1 lds\n\t"
                   "buffer_load_dwordx4 %2, %4, %5 offen offset:1024 sc0 sc1 lds\n\t"
                   "buffer_load_dwordx4 %3, %4, %5 offen offset:2048 sc0 sc1 lds\n\t"
                   "s_mov_b32 m0, %0"
                   : "=&s"(keep) : "s"(lds), "v"(vpay | oob), "v"(vlast | oob), "s"(rss), "s"(so) : "memory");
    };
    auto emit = [&](unsigned slot, unsigned n) {
      const char* sl = ring + slot * SLOT;
      const unsigned h0 = __builtin_amdgcn_readfirstlane(*(const lu32*)(sl + 2816));
      const u32x4 A = *(const lu32x4*)(sl + lane * 16), B = *(const lu32x4*)(sl + 1024 + lane * 16);
      const unsigned S = *(const lu32*)(sl + 2048 + lane * 12), K0 = *(const lu32*)(sl + 2052 + lane * 12), K1 = *(const lu32*)(sl + 2056 + lane * 12);
      const unsigned cq = n / 20, c = C0 + cq, r = n - cq * 20, w = r / 5, u16 = (ue >> (4 * (r - w * 5))) & 15;
      const int kind = u16 >> 3;
      const unsigned dst = (unsigned)stg_off(kind, src, u16 & 7) + (c * 4 + w) * BLK;
      if (h0 & 0x10000u) {
        u32x4 rr[4];
#pragma unroll
        for (int p = 0; p < 4; p++) rr[p] = __builtin_amdgcn_raw_buffer_load_b128(rraw, p * 1024 + lane * 16, dst, AUX_SYS);
#pragma unroll
        for (int p = 0; p < 4; p++) __builtin_amdgcn_raw_buffer_store_b128(rr[p], rd, p * 1024 + lane * 16, dst, AUX_AGENT);
        return;
      }
      const unsigned Md = (h0 & 0x80) ? 0x80808080u : 0u;
      const unsigned TG = (((h0 & 0x7f) * 0x01010101u | 0x80808080u) - 0x03020100u) | 0x80808080u;
      u32x4 o[4];
#pragma unroll
      for (int i = 0; i < 8; i++) {
        const unsigned Lp = i < 4 ? A[i] : B[i - 4];
        const unsigned code = ((i < 4 ? K0 : K1) >> (2 * (i & 3))) & 0x03030303u;
        const unsigned x = perm(TG, TG, code) - ((~Lp & Md) >> 7);
        const unsigned hi = (x & 0x7f7f7f7fu) | ((S << (7 - i)) & 0x80808080u);
        const unsigned L = Lp ^ Md;
        o[i >> 1][2 * (i & 1)] = perm(hi, L, 0x05010400u);
        o[i >> 1][2 * (i & 1) + 1] = perm(hi, L, 0x07030602u);
      }
      const unsigned cnt = (h0 >> 8) & 0xff;
      if (cnt == 0) {
#pragma unroll
        for (int p = 0; p < 4; p++)
          __builtin_amdgcn_raw_buffer_store_b128(o[p], rd, p * 1024 + (kind ? (wperm ^ (p * 16)) : lane * 16), dst, AUX_AGENT);
      } else {
#pragma unroll
        for (int p = 0; p < 4; p++) *(lu32x4*)(scr + p * 1024 + (kind ? (wperm ^ (p * 16)) : lane * 16)) = o[p];
        if (lane < (int)cnt) {
          const unsigned idx = *(const lu16*)(sl + 2816 + 4 + 2 * lane);
          *(lu8*)(scr + 2 * (idx & 0x7ff) + 1) = *(const lu8*)(sl + 2816 + 4 + 2 * NEXC + lane);
        }
#pragma unroll
        for (int p = 0; p < 4; p++) o[p] = *(const lu32x4*)(scr + p * 1024 + lane * 16);
#pragma unroll
        for (int p = 0; p < 4; p++) __builtin_amdgcn_raw_buffer_store_b128(o[p], rd, p * 1024 + lane * 16, dst, AUX_AGENT);
      }
    };
    auto arrive = [&](unsigned n) {
      const unsigned cq = n / 20, c = C0 + cq, r = n - cq * 20, w = r / 5, u16 = (ue >> (4 * (r - w * 5))) & 15;
      const unsigned so = (unsigned)(ARR_OFF - FLAG_OFF) + (src * 16 + u16) * 256 + (c * 2 + (w >> 1)) * 4;
      asm volatile("buffer_atomic_add %0, %1, %2, %3 offen" :: "v"(1u), "v"(l0), "s"(rfs), "s"(so) : "memory");
    };
    unsigned cur = ticket();
    unsigned nxt = cur + 1, prev = NITEM, slot = 0;
    grab();
    if (cur < NITEM) need(cur);
    load(0, cur);
    while (cur < NITEM) {
      if (nxt < NITEM) need(nxt);
      load(slot ^ 1, nxt);
      asm volatile("s_waitcnt vmcnt(3)" ::: "memory");
      if (prev < NITEM) arrive(prev);
      const unsigned nn = ticket();
      grab();
      emit(slot, cur);
      prev = cur; cur = nxt; nxt = nn; slot ^= 1;
    }
    asm volatile("s_waitcnt vmcnt(0)" ::: "memory");
    if (prev < NITEM) arrive(prev);
    return;
  }

  const int t1 = (te >> 14) & 7, i1 = (te >> 11) & 7, t2 = (te >> 8) & 7, i2 = (te >> 5) & 7, tu = (te >> 2) & 7, pj = te & 3;
  const int srcW = rank ^ tu;
  char* const donep = a.ws[srcW & 7] + DONE_OFF + b * 4;
  lu32* sw = (lu32*)(smem + NBUF * STAGE_BYTES);
  if (wv >= 4) {
    const int lw = wv - 4;
    const int kind = lw >> 1;
    const int src = lw == 0 ? rank ^ t1 : (lw == 1 ? rank ^ t2 : srcW);
    const int unit = lw == 0 ? i1 : (lw == 1 ? i2 : 2 * pj + (lw & 1));
    const char* g = me + stg_off(kind, src, unit) + lane * 16;
    const bool loc = src == rank;
    const char* pollp = loc ? me + READY_OFF + (size_t)(((src & 7) * 2 + kind) * 8 + unit) * 128
                            : me + ARR_OFF + (size_t)(((src & 7) * 2 + kind) * 8 + unit) * 256;
    const unsigned pthr = loc ? base + 1 : 2 * call;
    const int nword = loc ? 16 : 64;
    auto eval = [&](int q, unsigned v) -> int {
      const int w0 = loc ? q >> 2 : q;
      const unsigned long long nr = __ballot(lane < 16 && w0 + lane < nword && v < pthr);
      const int f = __builtin_ctz((unsigned)nr | 0x10000u);
      return min(64, loc ? 4 * (w0 + f) : w0 + f);
    };
    auto wordp = [&](int q) { return pollp + min((loc ? q >> 2 : q) + (lane & 15), nword - 1) * 4; };
    int avail = 0, pend = -1;
    int nb = 0;
    for (int s = -PF; s < 64; s++) {
      __builtin_amdgcn_s_barrier();
      asm volatile("" ::: "memory");
      if (s + PF < 64) {
        const int q = s + PF;
        if (q >= avail) {
          for (Spin sp;;) {
            avail = eval(q, ld_sys(wordp(q)));
            if (avail > q) break;
            if (sp.expired(me, call)) { avail = 64; break; }
          }

        }
        const unsigned* gp = (const unsigned*)(g + (size_t)q * (2 * BLK));
        unsigned* lp = (unsigned*)(smem + nb * STAGE_BYTES + lw * (2 * BLK));
        nb = nb == NBUF - 1 ? 0 : nb + 1;
        __builtin_amdgcn_global_load_lds(gp, lp, 16, 0, AUX_SYS);
        __builtin_amdgcn_global_load_lds(gp, lp, 16, 1024, AUX_SYS);
        __builtin_amdgcn_global_load_lds(gp, lp, 16, 2048, AUX_SYS);
        __builtin_amdgcn_global_load_lds(gp, lp, 16, 3072, AUX_SYS);
        __builtin_amdgcn_global_load_lds(gp + 1024, lp + 1024, 16, 0, AUX_SYS);
        __builtin_amdgcn_global_load_lds(gp + 1024, lp + 1024, 16, 1024, AUX_SYS);
        __builtin_amdgcn_global_load_lds(gp + 1024, lp + 1024, 16, 2048, AUX_SYS);
        __builtin_amdgcn_global_load_lds(gp + 1024, lp + 1024, 16, 3072, AUX_SYS);
        if ((q & 1) == 0) {
          if (avail < 64 && avail <= q + 2 * LOOK + 1) {
            asm volatile("global_load_dword v200, %0, off sc0 sc1" :: "v"(wordp(avail)) : "v200", "memory");
            pend = avail;
          }
        } else if (pend >= 0) {
          unsigned pv;
          asm volatile("s_waitcnt vmcnt(8)\n\tv_mov_b32 %0, v200" : "=v"(pv) :: "v200", "memory");
          avail = max(avail, eval(pend, pv));
          pend = -1;
        }
        if (PF == 4) asm volatile("s_waitcnt vmcnt(24)" ::: "memory");
        else if (PF == 3) asm volatile("s_waitcnt vmcnt(16)" ::: "memory");
        else asm volatile("s_waitcnt vmcnt(8)" ::: "memory");
      } else {
        const int rem = 62 - s;
        if (rem >= 2) asm volatile("s_waitcnt vmcnt(16)" ::: "memory");
        else if (rem == 1) asm volatile("s_waitcnt vmcnt(8)" ::: "memory");
        else asm volatile("s_waitcnt vmcnt(0)" ::: "memory");
      }
    }
    if (lw == 0 && lane == 0) sw[4] = 0;
    asm volatile("s_waitcnt lgkmcnt(0)" ::: "memory");
    __builtin_amdgcn_s_barrier();
    if (lw == 0) {
      const char* rp = me + RES_OFF + (b & 7) * 128;
      for (Spin sp;;) {
        u32x2 r;
        unsigned tm;
        asm volatile("global_load_dwordx2 %0, %2, off sc1\n\tglobal_load_dword %1, %3, off sc1\n\ts_waitcnt vmcnt(0)"
                     : "=&v"(r), "=&v"(tm) : "v"(rp), "v"(me + TMO_OFF) : "memory");
        const bool fin = __builtin_amdgcn_readfirstlane(r[1]) == call;
        bool bad = __builtin_amdgcn_readfirstlane(tm) == call || (fin && __builtin_amdgcn_readfirstlane(r[0]) == 0x7fffffffu);
        if (!fin && !bad && sp.expired(me, call)) bad = true;
        if (fin || bad) {
          if (lane == 0) {
            sw[5] = r[0];
            sw[6] = bad;
            __hip_atomic_store(sw + 4, 1u, __ATOMIC_RELEASE, __HIP_MEMORY_SCOPE_WORKGROUP);
          }
          break;
        }
      }
    }
    return;
  }
  const int xu = wv >> 1, wu = wv & 1;
  f32x4 acc[4][4];
#pragma unroll
  for (int f = 0; f < 4; f++)
#pragma unroll
    for (int j = 0; j < 4; j++) acc[f][j] = f32x4{0.f, 0.f, 0.f, 0.f};
  const unsigned xoff = xu * (2 * BLK) + lane * 16;
  unsigned woff[4];
  {
    const int r = lane & 3, q = (lane >> 2) & 3, kq = lane >> 4;
#pragma unroll
    for (int j = 0; j < 4; j++) woff[j] = (2 + wu) * (2 * BLK) + r * 1024 + kq * 256 + q * 64 + ((j ^ r) * 16);
  }
  {
    bf16x8 xa[4], wa[4], xb[4], wb[4];
#pragma unroll
    for (int f = 0; f < 4; f++) { xb[f] = bf16x8{}; wb[f] = bf16x8{}; }
    for (int s = -PF; s < 0; s++) __builtin_amdgcn_s_barrier();
    int cb = 0;
#define ILV() do { _Pragma("unroll") for (int q_ = 0; q_ < 8; q_++) { __builtin_amdgcn_sched_group_barrier(0x008, 1, 0); \
    __builtin_amdgcn_sched_group_barrier(0x100, 1, 0); } __builtin_amdgcn_sched_group_barrier(0x008, 8, 0); } while (0)
    for (int s = 0; s < 64; s++) {
      asm volatile("s_waitcnt lgkmcnt(0)" ::: "memory");
      __builtin_amdgcn_s_barrier();
      asm volatile("" ::: "memory");

      const char* sb = smem + cb * STAGE_BYTES;
      cb = cb == NBUF - 1 ? 0 : cb + 1;
      __builtin_amdgcn_sched_barrier(0);
      wa[0] = *(const bf16x8*)(sb + woff[0]); xa[0] = *(const bf16x8*)(sb + xoff);
#pragma unroll
      for (int f = 1; f < 4; f++) wa[f] = *(const bf16x8*)(sb + woff[f]);
#pragma unroll
      for (int f = 1; f < 4; f++) xa[f] = *(const bf16x8*)(sb + xoff + f * 1024);
#pragma unroll
      for (int f = 0; f < 4; f++)
#pragma unroll
        for (int j = 0; j < 4; j++) acc[f][j] = __builtin_amdgcn_mfma_f32_16x16x32_bf16(wb[j], xb[f], acc[f][j], 0, 0, 0);
      ILV();
      __builtin_amdgcn_sched_barrier(0);
      wb[0] = *(const bf16x8*)(sb + woff[0] + BLK); xb[0] = *(const bf16x8*)(sb + xoff + BLK);
#pragma unroll
      for (int f = 1; f < 4; f++) wb[f] = *(const bf16x8*)(sb + woff[f] + BLK);
#pragma unroll
      for (int f = 1; f < 4; f++) xb[f] = *(const bf16x8*)(sb + xoff + BLK + f * 1024);
#pragma unroll
      for (int f = 0; f < 4; f++)
#pragma unroll
        for (int j = 0; j < 4; j++) acc[f][j] = __builtin_amdgcn_mfma_f32_16x16x32_bf16(wa[j], xa[f], acc[f][j], 0, 0, 0);
      ILV();
      __builtin_amdgcn_sched_barrier(0);
    }
#pragma unroll
    for (int f = 0; f < 4; f++)
#pragma unroll
      for (int j = 0; j < 4; j++) acc[f][j] = __builtin_amdgcn_mfma_f32_16x16x32_bf16(wb[j], xb[f], acc[f][j], 0, 0, 0);
  }
  float mf = 0.f;
#pragma unroll
  for (int f = 0; f < 4; f++)
#pragma unroll
    for (int j = 0; j < 4; j++) {
      const f32x4 t = acc[f][j];
      mf = fmaxf(fmaxf(mf, fabsf(t[0])), fabsf(t[1]));
      mf = fmaxf(fmaxf(mf, fabsf(t[2])), fabsf(t[3]));
    }
  unsigned m = wave_max_u32(__float_as_uint(mf));
  if (lane == 0) sw[wv] = m;
  asm volatile("s_waitcnt lgkmcnt(0)" ::: "memory");
  __builtin_amdgcn_s_barrier();
  asm volatile("" ::: "memory");
  m = max(max(sw[0], sw[1]), max(sw[2], sw[3]));
  if (tid < NR) {
    gu64* g = (gu64*)(wsl + AMAX_OFF + (size_t)(rank * NGEMM + b) * 8);
    __hip_atomic_store(g, ((unsigned long long)call << 32) | m, __ATOMIC_RELAXED, __HIP_MEMORY_SCOPE_SYSTEM);
  }
  while (__builtin_amdgcn_readfirstlane(__hip_atomic_load(sw + 4, __ATOMIC_ACQUIRE, __HIP_MEMORY_SCOPE_WORKGROUP)) == 0)
    __builtin_amdgcn_s_sleep(1);
  const float inv = 448.0f / __uint_as_float(__builtin_amdgcn_readfirstlane(sw[5]));
  const bool bad = __builtin_amdgcn_readfirstlane(sw[6]) != 0;
  {
    const int tx = xu == 0 ? t1 : t2, ix = xu == 0 ? i1 : i2;
    unsigned char* optr = outp + (size_t)((rank ^ tx) * SH + ix * 64 + (lane & 15)) * SH + pj * 128 + wu * 64 + (lane >> 4) * 16;
#pragma unroll
    for (int f = 0; f < 4; f++) {
      u32x4 q;
#pragma unroll
      for (int j = 0; j < 4; j++) {
        int w = __builtin_amdgcn_cvt_pk_fp8_f32(acc[f][j][0] * inv, acc[f][j][1] * inv, 0, false);
        w = __builtin_amdgcn_cvt_pk_fp8_f32(acc[f][j][2] * inv, acc[f][j][3] * inv, w, true);
        q[j] = bad ? 0x7f7f7f7fu : (unsigned)w;
      }
      if (tu != 0) asm volatile("global_store_dwordx4 %0, %1, off sc0 sc1\n\ts_nop 1" ::"v"(optr + (size_t)f * 16 * SH), "v"(q) : "memory");
      else *(u32x4*)(optr + (size_t)f * 16 * SH) = q;
    }
  }
  if (tu != 0) {
    asm volatile("s_waitcnt vmcnt(0)" ::: "memory");
    __builtin_amdgcn_s_barrier();
    if (tid == 0) st_sys(donep, call);
  }
}
}

extern "C" void kernel_launch(int rank, int nranks, ncclComm_t comm, hipStream_t stream, void* const* d_in,
                              const int* in_sizes, int n_in, void* d_out, int out_size, const int* in_dtypes,
                              int out_dtype, void* const* d_out_all, void* const* d_ws_all, size_t ws_bytes) {
  (void)comm; (void)in_sizes; (void)n_in; (void)d_out; (void)out_size; (void)in_dtypes; (void)out_dtype;
  if (nranks != NR || rank < 0 || rank >= NR || ws_bytes < WS_NEED) {
    fprintf(stderr, "unsupported configuration\n");
    return;
  }
  Args a;
  a.x = (const float*)d_in[0];
  a.w = (const float*)d_in[1];
  for (int p = 0; p < NR; p++) {
    a.ws[p] = (char*)d_ws_all[p];
    a.out[p] = (unsigned char*)d_out_all[p];
  }
  a.rank = rank;
  a.pad = 0;
  fused_ag_gemm<<<NWG, 512, 0, stream>>>(a);
  hipError_t e = hipGetLastError();
  if (e != hipSuccess) fprintf(stderr, "launch error %s\n", hipGetErrorString(e));
}
```

```cpp
#include <hip/hip_runtime.h>
#include <rccl/rccl.h>
#include <cstdio>
#include <cstdint>

typedef unsigned u32x4 __attribute__((ext_vector_type(4)));
typedef unsigned u32x3 __attribute__((ext_vector_type(3)));
typedef unsigned u32x2 __attribute__((ext_vector_type(2)));
typedef float f32x4 __attribute__((ext_vector_type(4)));
typedef __bf16 bf16x8 __attribute__((ext_vector_type(8)));
typedef __bf16 bf16x2 __attribute__((ext_vector_type(2)));
typedef float f32x2 __attribute__((ext_vector_type(2)));

#define NPW 5
#define NPUSH 1
#define LOOK 1
#define STAG0 60
#define STAGGER 25

namespace {
constexpr int NR = 8;
constexpr int KDIM = 4096;
constexpr int SH = 512;
constexpr unsigned BLK = 4096;
constexpr unsigned UNIT_BYTES = 128 * BLK;
constexpr unsigned PBLK = 2944;
constexpr unsigned PUNIT_BYTES = 128 * PBLK;
constexpr unsigned NEXC = 41;
constexpr size_t STG_KIND_BYTES = (size_t)NR * 8 * UNIT_BYTES;
constexpr size_t EXP_OFF = 2 * STG_KIND_BYTES;
constexpr size_t FLAG_OFF = EXP_OFF + (size_t)16 * PUNIT_BYTES;
constexpr size_t READY_OFF = FLAG_OFF;
constexpr size_t ARR_OFF = READY_OFF + NR * 16 * 128;
constexpr size_t AMAX_OFF = ARR_OFF + NR * 16 * 256;
constexpr size_t DONE_OFF = AMAX_OFF + NR * 128 * 8;
constexpr size_t CTR_OFF = DONE_OFF + 1024;
constexpr size_t TMO_OFF = CTR_OFF + 128;
constexpr size_t QCTR_OFF = TMO_OFF + 128;
constexpr size_t RES_OFF = QCTR_OFF + 16 * 128;
constexpr size_t END_OFF = RES_OFF + 8 * 128;
constexpr size_t WS_NEED = END_OFF;
static_assert(FLAG_OFF % 128 == 0, "align");
constexpr int NGEMM = 128, NPULL = 35, NWG = 256;
constexpr unsigned long long DEADLINE = 30000000ull;
constexpr unsigned STAGE_BYTES = 8 * BLK;
#define NBUFS 4
constexpr int NBUF = NBUFS;
constexpr int PF = NBUF - 1;

__constant__ unsigned TILE_TAB[NGEMM] = {16420,2145,5154,5283,14530,10241,8928,10432,69476,67649,7906,47971,70370,15553,17664,15424,3116,74817,8450,78019,77058,16577,113792,55360,18284,75905,12130,83011,3238,113377,121888,75872,70924,82785,73762,70820,13574,1059,3115,77856,77964,5158,78690,82660,69798,18147,7435,4311,8207,55430,3939,112708,78566,76,47243,47287,121935,75622,6307,75,112902,11404,113803,55351,1081,3112,73859,11115,10991,13484,4373,69751,9433,77064,10340,75979,12335,48876,9973,12377,12153,121928,12420,78923,76015,9271,67637,47225,17561,5167,74948,55518,79599,74935,73845,75833,83033,75023,6216,70846,11477,78103,77685,2234,2238,2173,12008,78590,69845,12381,47258,55386,8478,70813,73864,81950,9501,48349,68890,69850,68766,76669,78024,112734,10493,69373,75642,79034};
__constant__ unsigned PULL_TAB[8] = {0, 0xdc980, 0xfe987, 0xfe983, 0x65421, 0xfeba0, 0xdcba7, 0xdcba3};
constexpr unsigned long long DEST_MASK_X = 0x4511111189111123ull, DEST_MASK_W = 0x2d2dc3c3e1e10f0full;

struct Args {
  const float* x;
  const float* w;
  char* ws[NR];
  unsigned char* out[NR];
  int rank;
  int pad;
};

typedef __attribute__((address_space(1))) unsigned gu32;
typedef __attribute__((address_space(1))) unsigned long long gu64;
typedef __attribute__((address_space(3))) unsigned lu32;
typedef __attribute__((address_space(3))) unsigned char lu8;
typedef __attribute__((address_space(3))) unsigned short lu16;
typedef __attribute__((address_space(3))) u32x4 lu32x4;

__device__ __forceinline__ unsigned ld_sys(const void* p) { return __hip_atomic_load((const gu32*)p, __ATOMIC_RELAXED, __HIP_MEMORY_SCOPE_SYSTEM); }
__device__ __forceinline__ void st_sys(void* p, unsigned v) { __hip_atomic_store((gu32*)p, v, __ATOMIC_RELAXED, __HIP_MEMORY_SCOPE_SYSTEM); }
__device__ __forceinline__ unsigned ld_agent(const void* p) { return __hip_atomic_load((const gu32*)p, __ATOMIC_RELAXED, __HIP_MEMORY_SCOPE_AGENT); }
__device__ __forceinline__ void st_agent(void* p, unsigned v) { __hip_atomic_store((gu32*)p, v, __ATOMIC_RELAXED, __HIP_MEMORY_SCOPE_AGENT); }

__device__ __forceinline__ __amdgpu_buffer_rsrc_t mkrsrc(const void* p, unsigned bytes) {
  unsigned long long a = (unsigned long long)p;
  unsigned lo = __builtin_amdgcn_readfirstlane((unsigned)a), hi = __builtin_amdgcn_readfirstlane((unsigned)(a >> 32));
  void* q = (void*)(((unsigned long long)hi << 32) | lo);
  return __builtin_amdgcn_make_buffer_rsrc(q, 0, bytes, 0x00020000);
}
__device__ __forceinline__ u32x4 mkrsrc_words(const void* p, unsigned bytes) {
  unsigned long long a = (unsigned long long)p;
  return u32x4{(unsigned)__builtin_amdgcn_readfirstlane((unsigned)a), (unsigned)__builtin_amdgcn_readfirstlane((unsigned)(a >> 32)) & 0xffffu,
               bytes, 0x00020000u};
}
__device__ __forceinline__ size_t stg_off(int kind, int src, int unit) {
  return (size_t)kind * STG_KIND_BYTES + (size_t)(src * 8 + unit) * UNIT_BYTES;
}
__device__ __forceinline__ unsigned cvt2(float a, float b) {
  f32x2 v = {a, b};
  bf16x2 r = __builtin_convertvector(v, bf16x2);
  return __builtin_bit_cast(unsigned, r);
}
__device__ __forceinline__ unsigned perm(unsigned hi, unsigned lo, unsigned sel) { return __builtin_amdgcn_perm(hi, lo, sel); }
__device__ __forceinline__ unsigned wave_max_u32(unsigned v) {
  v = max(v, (unsigned)__builtin_amdgcn_update_dpp(0, (int)v, 0x111, 0xf, 0xf, false));
  v = max(v, (unsigned)__builtin_amdgcn_update_dpp(0, (int)v, 0x112, 0xf, 0xf, false));
  v = max(v, (unsigned)__builtin_amdgcn_update_dpp(0, (int)v, 0x114, 0xf, 0xf, false));
  v = max(v, (unsigned)__builtin_amdgcn_update_dpp(0, (int)v, 0x118, 0xf, 0xf, false));
  v = max(v, (unsigned)__builtin_amdgcn_update_dpp(0, (int)v, 0x142, 0xa, 0xf, false));
  v = max(v, (unsigned)__builtin_amdgcn_update_dpp(0, (int)v, 0x143, 0xc, 0xf, false));
  return (unsigned)__builtin_amdgcn_readlane((int)v, 63);
}

struct Spin {
  unsigned it = 0;
  unsigned long long t0 = 0;
  __device__ __forceinline__ bool expired(char* me, unsigned call) {
    if (it == 0) t0 = __builtin_amdgcn_s_memrealtime();
    if ((++it & 15) == 0) {
      if (ld_agent(me + TMO_OFF) == call || __builtin_amdgcn_s_memrealtime() - t0 > DEADLINE) {
        st_agent(me + TMO_OFF, call);
        return true;
      }
    }
    __builtin_amdgcn_s_sleep(1);
    return false;
  }
};
constexpr int AUX_SYS = 17, AUX_AGENT = 16;

__device__ __forceinline__ unsigned piece_off(int kind, int p, int lane) {
  const int kq = lane >> 4, q = (lane >> 2) & 3, j = lane & 3;
  return kind == 0 ? p * 1024 + lane * 16 : p * 1024 + kq * 256 + q * 64 + ((j ^ p) * 16);
}

__global__ __launch_bounds__(512, 2) void fused_ag_gemm(Args a) {
  __shared__ __attribute__((aligned(16))) char smem[NBUF * STAGE_BYTES + 128];
  const int tid = threadIdx.x, lane = tid & 63;
  const int wv = __builtin_amdgcn_readfirstlane(tid >> 6);
  const int b = blockIdx.x;
  constexpr int NH0 = 16 * NPUSH;
  const int task = b < NGEMM ? b + NH0 + NPULL
                             : (b < NGEMM + NPULL ? b - NGEMM + NH0 : (b - (NGEMM + NPULL) < NH0 ? b - (NGEMM + NPULL) : b));
  const int c16 = task >> 4, u16 = task & 15, kind = u16 >> 3, unit = u16 & 7;
  const int ks = c16 * 8 + wv;
  const int kq = lane >> 4, l15 = lane & 15;
  if (c16 > NPUSH) {
    const unsigned long long tstart = __builtin_amdgcn_s_memrealtime();
    if (b >= NGEMM + NPULL + 16 * NPUSH && wv == 0) {
      const char* wsp = a.ws[lane & 7];
      const unsigned char* op = a.out[lane & 7];
      const int r8 = lane >> 3;
      const char* t0 = r8 < 3 ? wsp + EXP_OFF + (size_t)r8 * (2u << 20) : (r8 == 3 ? wsp + FLAG_OFF : (r8 == 4 ? (const char*)op : wsp + (size_t)(a.rank * 8) * UNIT_BYTES + (r8 == 5 ? 0 : (r8 == 6 ? (2u << 20) : STG_KIND_BYTES))));
      const char* t1 = a.ws[a.rank] + (size_t)(lane & 31) * (2u << 20);
      unsigned d0, d1;
      asm volatile("global_load_dword %0, %2, off sc0 sc1\n\tglobal_load_dword %1, %3, off sc0 sc1\n\ts_waitcnt vmcnt(0)"
                   : "=&v"(d0), "=&v"(d1) : "v"(t0), "v"(t1) : "memory");
    }
    while (__builtin_amdgcn_s_memrealtime() - tstart < (unsigned long long)(STAG0 + (c16 - NPUSH - 1) * STAGGER)) __builtin_amdgcn_s_sleep(1);
  }
  f32x4 v[8];
  {
    const float* px = a.x + (size_t)(unit * 64 + l15) * KDIM + ks * 32 + 4 * kq;
    const float* pw = a.w + (size_t)(ks * 32 + 4 * kq) * SH + unit * 64 + 4 * l15;
    const float* p = kind == 0 ? px : pw;
#pragma unroll
    for (int i = 0; i < 8; i++) {
      const unsigned ox = (i >> 1) * 16 * KDIM + (i & 1) * 16, ow = ((i & 3) + (i >> 2) * 16) * SH;
      v[i] = *(const f32x4*)(p + (kind == 0 ? ox : ow));
    }
  }
  asm volatile("" ::: "memory");
  const int rank = a.rank;
  char* me = a.ws[rank];
  const unsigned ctr_v = ld_agent(me + CTR_OFF);
  const unsigned te = TILE_TAB[b & (NGEMM - 1)];
  const bool is_pull = b >= NGEMM && b < NGEMM + NPULL;
  const int plink = is_pull ? 1 + (b - NGEMM) / 5 : 0;
  const unsigned ue = PULL_TAB[plink];
  char* const wsl = a.ws[lane & 7];
  unsigned char* const outp = a.out[(rank ^ (int)(te >> 2)) & 7];
  const u32x4 rfs = mkrsrc_words(me + FLAG_OFF, (unsigned)(END_OFF - FLAG_OFF));
  const unsigned l0 = lane == 0 ? 0u : 0x40000000u;
  unsigned call, base;
  {
    const __amdgpu_buffer_rsrc_t rd = mkrsrc(me + stg_off(kind, rank, unit), UNIT_BYTES);
    call = __builtin_amdgcn_readfirstlane(ctr_v) + 1;
    base = call * 64;
    if (is_pull && wv < NPW) {
      const unsigned qoff = (unsigned)(QCTR_OFF - FLAG_OFF) + ((call & 1) * 8 + plink) * 128;
      asm volatile("v_mov_b32 v200, 2\n\ts_nop 4\n\tbuffer_atomic_add v200, %0, %1, %2 offen sc0"
                   :: "v"(l0), "s"(rfs), "s"(qoff) : "v200", "memory");
    }
    u32x4 o[4];
    if (kind == 0) {
#pragma unroll
      for (int f = 0; f < 4; f++)
        o[f] = u32x4{cvt2(v[2 * f][0], v[2 * f][1]), cvt2(v[2 * f][2], v[2 * f][3]),
                     cvt2(v[2 * f + 1][0], v[2 * f + 1][1]), cvt2(v[2 * f + 1][2], v[2 * f + 1][3])};
    } else {
#pragma unroll
      for (int r = 0; r < 4; r++)
        o[r] = u32x4{cvt2(v[0][r], v[1][r]), cvt2(v[2][r], v[3][r]), cvt2(v[4][r], v[5][r]), cvt2(v[6][r], v[7][r])};
    }
    const unsigned dmask = (unsigned)((kind ? DEST_MASK_W : DEST_MASK_X) >> (8 * unit)) & 0xff;
    if (c16 < NPUSH) {
#pragma unroll
      for (int p = 1; p < NR; p++)
        if ((dmask >> p) & 1) {
          const __amdgpu_buffer_rsrc_t rq = mkrsrc(a.ws[rank ^ p] + stg_off(kind, rank, unit), UNIT_BYTES);
#pragma unroll
          for (int i = 0; i < 4; i++)
            __builtin_amdgcn_raw_buffer_store_b128(o[i], rq, ks * BLK + piece_off(kind, i, lane), 0, AUX_SYS);
        }
    } else {
      float mx = 0.f;
#pragma unroll
      for (int i = 0; i < 8; i++) {
        mx = fmaxf(mx, fmaxf(fabsf(v[i][0]), fabsf(v[i][1])));
        mx = fmaxf(mx, fmaxf(fabsf(v[i][2]), fabsf(v[i][3])));
      }
      const unsigned mxb = wave_max_u32(__float_as_uint(mx));
      const unsigned emax = (cvt2(__uint_as_float(mxb), 0.f) >> 7) & 0xff;
      const float thr1 = __uint_as_float(emax << 23), thr2 = thr1 * 0.5f;
      unsigned n1 = 0, n2 = 0;
#pragma unroll
      for (int i = 0; i < 8; i++) {
        n1 += __builtin_popcountll(__ballot(fabsf(v[i][0]) >= thr1)) + __builtin_popcountll(__ballot(fabsf(v[i][2]) >= thr1));
        n2 += __builtin_popcountll(__ballot(fabsf(v[i][0]) >= thr2)) + __builtin_popcountll(__ballot(fabsf(v[i][2]) >= thr2));
      }
      const int top = (int)emax - (n1 <= 10 ? 1 : 0) - (n2 <= 10 ? 1 : 0);
      const unsigned delta = (top & 1) ^ 1, tp = ((top + (int)delta) >> 1) & 0x7f;
      const unsigned Md = delta ? 0x80808080u : 0u;
      const unsigned b80 = tp * 0x01010101u | 0x80808080u;
      const unsigned T = (b80 - 0x03020100u) & 0x7f7f7f7fu;
      unsigned LP[8], HH[8], S = 0, K0 = 0, K1 = 0, M = 0;
#pragma unroll
      for (int i = 0; i < 8; i++) {
        const unsigned d0 = o[i >> 1][2 * (i & 1)], d1 = o[i >> 1][2 * (i & 1) + 1];
        const unsigned L = perm(d1, d0, 0x06040200u), H = perm(d1, d0, 0x07050301u);
        const unsigned pair = ((H & 0x7f7f7f7fu) + ((L & Md) >> 7)) & 0x7f7f7f7fu;
        const unsigned code = (b80 - pair) & 0x03030303u;
        const unsigned mis = perm(T, T, code) ^ pair;
        LP[i] = L ^ Md;
        HH[i] = H;
        if (i < 4) K0 |= code << (2 * i); else K1 |= code << (2 * (i - 4));
        S |= (H & 0x80808080u) >> (7 - i);
        M |= __builtin_amdgcn_udot4(((mis + 0x7f7f7f7fu) & 0x80808080u) >> 7, 0x08040201u, 0u, false) << (4 * i);
      }
      char* hsc = smem + wv * 2304;
      lu32* hdr = (lu32*)(hsc + 2048);
      const unsigned nl = __builtin_popcount(M);
      unsigned inc = nl;
      inc += __builtin_amdgcn_update_dpp(0, (int)inc, 0x111, 0xf, 0xf, false);
      inc += __builtin_amdgcn_update_dpp(0, (int)inc, 0x112, 0xf, 0xf, false);
      inc += __builtin_amdgcn_update_dpp(0, (int)inc, 0x114, 0xf, 0xf, false);
      inc += __builtin_amdgcn_update_dpp(0, (int)inc, 0x118, 0xf, 0xf, false);
      inc += __builtin_amdgcn_update_dpp(0, (int)inc, 0x142, 0xa, 0xf, false);
      inc += __builtin_amdgcn_update_dpp(0, (int)inc, 0x143, 0xc, 0xf, false);
      const unsigned cnt = (unsigned)__builtin_amdgcn_readlane((int)inc, 63);
      if (cnt > 0 && cnt <= NEXC) {
        *(lu32x4*)(hsc + lane * 32) = u32x4{HH[0], HH[1], HH[2], HH[3]};
        *(lu32x4*)(hsc + lane * 32 + 16) = u32x4{HH[4], HH[5], HH[6], HH[7]};
        unsigned at = inc - nl;
        while (__any(M != 0)) {
          if (M != 0) {
            const unsigned jv = __builtin_ctz(M);
            M &= M - 1;
            const unsigned char hi = *(const lu8*)(hsc + lane * 32 + jv);
            const unsigned idx = (piece_off(kind, jv >> 3, lane) >> 1) + (jv & 7);
            *(lu16*)((char*)hdr + 4 + 2 * at) = (unsigned short)idx;
            *(lu8*)((char*)hdr + 4 + 2 * NEXC + at) = hi;
            at++;
          }
        }
      }
      const __amdgpu_buffer_rsrc_t rp = mkrsrc(me + EXP_OFF + (size_t)u16 * PUNIT_BYTES, PUNIT_BYTES);
      const unsigned pc = ks * PBLK;
      __builtin_amdgcn_raw_buffer_store_b128(u32x4{LP[0], LP[1], LP[2], LP[3]}, rp, pc + lane * 16, 0, AUX_SYS);
      __builtin_amdgcn_raw_buffer_store_b128(u32x4{LP[4], LP[5], LP[6], LP[7]}, rp, pc + 1024 + lane * 16, 0, AUX_SYS);
      __builtin_amdgcn_raw_buffer_store_b96(u32x3{S, K0, K1}, rp, pc + 2048 + lane * 12, 0, AUX_SYS);
      unsigned hv = hdr[lane & 31];
      if (lane == 0) hv = tp | (delta << 7) | (cnt > NEXC ? 0x10000u : cnt << 8);
      if (lane < 32) __builtin_amdgcn_raw_buffer_store_b32(hv, rp, pc + 2816 + lane * 4, 0, AUX_SYS);
    }
#pragma unroll
    for (int p = 0; p < 4; p++)
      __builtin_amdgcn_raw_buffer_store_b128(o[p], rd, ks * BLK + piece_off(kind, p, lane), 0, AUX_SYS);
    asm volatile("s_waitcnt vmcnt(0)" ::: "memory");
    const int rel = rank ^ lane;
    if (c16 < NPUSH) {
      if (lane < NR && rel != 0 && ((dmask >> rel) & 1))
        __hip_atomic_fetch_add((gu32*)(wsl + ARR_OFF + (size_t)(rank * 16 + u16) * 256 + (ks >> 1) * 4), 1u, __ATOMIC_RELAXED,
                               __HIP_MEMORY_SCOPE_SYSTEM);
    }
    __builtin_amdgcn_s_barrier();
    if (wv == 0 && lane < NR && ((dmask >> rel) & 1) && (rel == 0 || c16 >= NPUSH))
      st_sys(wsl + READY_OFF + (size_t)(rank * 16 + u16) * 128 + c16 * 4, base + 1);
  }

  if (b >= NGEMM + NPULL) {
    if (b == NWG - 1) {
      {
        lu32* sv = (lu32*)smem;
        __builtin_amdgcn_s_barrier();
        if (tid < 3) sv[tid] = 0;
        asm volatile("s_waitcnt lgkmcnt(0)" ::: "memory");
        __builtin_amdgcn_s_barrier();
        const char* gp = me + AMAX_OFF + (size_t)tid * 16;
        bool have = false, bad = false;
        unsigned gm = 0;
        for (Spin sp;;) {
          unsigned tm = 0;
          if (!have) {
            u32x4 g0;
            asm volatile("global_load_dwordx4 %0, %2, off sc0 sc1\n\tglobal_load_dword %1, %3, off sc1\n\ts_waitcnt vmcnt(0)"
                         : "=&v"(g0), "=&v"(tm) : "v"(gp), "v"(me + TMO_OFF) : "memory");
            have = g0[1] == call && g0[3] == call;
            gm = max(g0[0], g0[2]);
          }
          if (__any(tm == call)) { bad = true; break; }
          if (__all(have)) break;
          if (sp.expired(me, call)) { bad = true; break; }
        }
        const unsigned wm = wave_max_u32(gm);
        unsigned last = 0;
        if (lane == 0) {
          __hip_atomic_fetch_max(sv, wm, __ATOMIC_RELAXED, __HIP_MEMORY_SCOPE_WORKGROUP);
          if (bad) __hip_atomic_store(sv + 2, 1u, __ATOMIC_RELAXED, __HIP_MEMORY_SCOPE_WORKGROUP);
          last = __hip_atomic_fetch_add(sv + 1, 1u, __ATOMIC_ACQ_REL, __HIP_MEMORY_SCOPE_WORKGROUP);
        }
        if (__builtin_amdgcn_readfirstlane(last) == 7) {
          const unsigned am = sv[2] ? 0x7fffffffu : sv[0];
          if (lane < 8) __hip_atomic_store((gu64*)(me + RES_OFF + lane * 128), ((unsigned long long)call << 32) | am,
                                           __ATOMIC_RELAXED, __HIP_MEMORY_SCOPE_AGENT);
        }
      }
      if (wv == 0) {
        bool ok = true;
        {
          const bool a0 = ((TILE_TAB[2 * lane] >> 2) & 7) != 0, a1 = ((TILE_TAB[2 * lane + 1] >> 2) & 7) != 0;
          for (Spin sp;;) {
            u32x2 d;
            asm volatile("global_load_dwordx2 %0, %1, off sc0 sc1\n\ts_waitcnt vmcnt(0)" : "=v"(d) : "v"(me + DONE_OFF + lane * 8) : "memory");
            if (__all((!a0 || d[0] >= call) && (!a1 || d[1] >= call))) break;
            if (sp.expired(me, call)) { ok = false; break; }
          }
        }
        if (lane < 8) st_agent(me + QCTR_OFF + (size_t)(((call + 1) & 1) * 8 + lane) * 128, 0u);
        if (lane == 0) {
          if (!ok) a.out[rank][(size_t)(rank * SH + 64) * SH + 128] = 0x7f;
          st_agent(me + CTR_OFF, call);
        }
      }
    }
    return;
  }
  if (is_pull) {
    if (wv >= NPW) return;
    const unsigned src = (unsigned)(rank ^ plink);
    if (src >= NR) return;
    const __amdgpu_buffer_rsrc_t rraw = mkrsrc(a.ws[src], 2 * STG_KIND_BYTES);
    const __amdgpu_buffer_rsrc_t rd = mkrsrc(me, 2 * STG_KIND_BYTES);
    const u32x4 rss = mkrsrc_words(a.ws[src] + EXP_OFF, 16 * PUNIT_BYTES);
    const unsigned wperm = piece_off(1, 0, lane);
    char* scr = smem + 1024 + wv * BLK;
    constexpr unsigned SLOT = 3072;
    char* ring = smem + 1024 + NPW * BLK + wv * 2 * SLOT;
    unsigned long long rdy03 = 0;
    unsigned rdy4 = 0;
    bool ok = true;
    constexpr unsigned C0 = 2 * NPUSH, NITEM = (32 - C0) * 20;
    const unsigned qoff = (unsigned)(QCTR_OFF - FLAG_OFF) + ((call & 1) * 8 + plink) * 128;
    auto grab = [&]() {
      asm volatile("v_mov_b32 v200, 1\n\ts_nop 4\n\tbuffer_atomic_add v200, %0, %1, %2 offen sc0"
                   :: "v"(l0), "s"(rfs), "s"(qoff) : "v200", "memory");
    };
    auto ticket = [&]() -> unsigned {
      unsigned v;
      asm volatile("v_mov_b32 %0, v200" : "=v"(v) :: "v200", "memory");
      return __builtin_amdgcn_readfirstlane(v);
    };
    auto need = [&](unsigned n) {
      const unsigned cq = n / 20, c = C0 + cq, ui = (n - cq * 20) % 5, c2 = c >> 1;
      for (Spin sp;;) {
        const unsigned m = ui == 4 ? rdy4 : (unsigned)(rdy03 >> (16 * ui));
        if (((m >> c2) & 1) || !ok) return;
        const unsigned u16 = (ue >> (4 * ui)) & 15;
        const unsigned v = lane < 16 ? ld_sys(me + READY_OFF + (size_t)(src * 16 + u16) * 128 + lane * 4) : 0u;
        const unsigned mk = (unsigned)__ballot(v >= base + 1) & 0xffffu;
        if (ui == 4) rdy4 = mk; else rdy03 |= (unsigned long long)mk << (16 * ui);
        if ((mk >> c2) & 1) return;
        if (sp.expired(me, call)) ok = false;
      }
    };
    const unsigned vpay = lane * 16, vlast = lane < 56 ? lane * 16 : 0x40000000u;
    auto load = [&](unsigned slot, unsigned n) {
      const bool val = n < NITEM;
      const unsigned nn = val ? n : 0;
      const unsigned cq = nn / 20, c = C0 + cq, r = nn - cq * 20, w = r / 5, u16 = (ue >> (4 * (r - w * 5))) & 15;
      const unsigned oob = val ? 0u : 0x40000000u;
      const unsigned so = u16 * PUNIT_BYTES + (c * 4 + w) * PBLK;
      const unsigned lds = __builtin_amdgcn_readfirstlane((unsigned)(unsigned long long)(lu8*)(ring + slot * SLOT));
      unsigned keep;
      asm volatile("s_mov_b32 %0, m0\n\ts_mov_b32 m0, %1\n\ts_nop 0\n\t"
                   "buffer_load_dwordx4 %2, %4, %5 offen sc0 sc1 lds\n\t"
                   "buffer_load_dwordx4 %2, %4, %5 offen offset:1024 sc0 sc1 lds\n\t"
                   "buffer_load_dwordx4 %3, %4, %5 offen offset:2048 sc0 sc1 lds\n\t"
                   "s_mov_b32 m0, %0"
                   : "=&s"(keep) : "s"(lds), "v"(vpay | oob), "v"(vlast | oob), "s"(rss), "s"(so) : "memory");
    };
    auto emit = [&](unsigned slot, unsigned n) {
      const char* sl = ring + slot * SLOT;
      const unsigned h0 = __builtin_amdgcn_readfirstlane(*(const lu32*)(sl + 2816));
      const u32x4 A = *(const lu32x4*)(sl + lane * 16), B = *(const lu32x4*)(sl + 1024 + lane * 16);
      const unsigned S = *(const lu32*)(sl + 2048 + lane * 12), K0 = *(const lu32*)(sl + 2052 + lane * 12), K1 = *(const lu32*)(sl + 2056 + lane * 12);
      const unsigned cq = n / 20, c = C0 + cq, r = n - cq * 20, w = r / 5, u16 = (ue >> (4 * (r - w * 5))) & 15;
      const int kind = u16 >> 3;
      const unsigned dst = (unsigned)stg_off(kind, src, u16 & 7) + (c * 4 + w) * BLK;
      if (h0 & 0x10000u) {
        u32x4 rr[4];
#pragma unroll
        for (int p = 0; p < 4; p++) rr[p] = __builtin_amdgcn_raw_buffer_load_b128(rraw, p * 1024 + lane * 16, dst, AUX_SYS);
#pragma unroll
        for (int p = 0; p < 4; p++) __builtin_amdgcn_raw_buffer_store_b128(rr[p], rd, p * 1024 + lane * 16, dst, AUX_AGENT);
        return;
      }
      const unsigned Md = (h0 & 0x80) ? 0x80808080u : 0u;
      const unsigned TG = (((h0 & 0x7f) * 0x01010101u | 0x80808080u) - 0x03020100u) | 0x80808080u;
      u32x4 o[4];
#pragma unroll
      for (int i = 0; i < 8; i++) {
        const unsigned Lp = i < 4 ? A[i] : B[i - 4];
        const unsigned code = ((i < 4 ? K0 : K1) >> (2 * (i & 3))) & 0x03030303u;
        const unsigned x = perm(TG, TG, code) - ((~Lp & Md) >> 7);
        const unsigned hi = (x & 0x7f7f7f7fu) | ((S << (7 - i)) & 0x80808080u);
        const unsigned L = Lp ^ Md;
        o[i >> 1][2 * (i & 1)] = perm(hi, L, 0x05010400u);
        o[i >> 1][2 * (i & 1) + 1] = perm(hi, L, 0x07030602u);
      }
      const unsigned cnt = (h0 >> 8) & 0xff;
      if (cnt == 0) {
#pragma unroll
        for (int p = 0; p < 4; p++)
          __builtin_amdgcn_raw_buffer_store_b128(o[p], rd, p * 1024 + (kind ? (wperm ^ (p * 16)) : lane * 16), dst, AUX_AGENT);
      } else {
#pragma unroll
        for (int p = 0; p < 4; p++) *(lu32x4*)(scr + p * 1024 + (kind ? (wperm ^ (p * 16)) : lane * 16)) = o[p];
        if (lane < (int)cnt) {
          const unsigned idx = *(const lu16*)(sl + 2816 + 4 + 2 * lane);
          *(lu8*)(scr + 2 * (idx & 0x7ff) + 1) = *(const lu8*)(sl + 2816 + 4 + 2 * NEXC + lane);
        }
#pragma unroll
        for (int p = 0; p < 4; p++) o[p] = *(const lu32x4*)(scr + p * 1024 + lane * 16);
#pragma unroll
        for (int p = 0; p < 4; p++) __builtin_amdgcn_raw_buffer_store_b128(o[p], rd, p * 1024 + lane * 16, dst, AUX_AGENT);
      }
    };
    auto arrive = [&](unsigned n) {
      const unsigned cq = n / 20, c = C0 + cq, r = n - cq * 20, w = r / 5, u16 = (ue >> (4 * (r - w * 5))) & 15;
      const unsigned so = (unsigned)(ARR_OFF - FLAG_OFF) + (src * 16 + u16) * 256 + (c * 2 + (w >> 1)) * 4;
      asm volatile("buffer_atomic_add %0, %1, %2, %3 offen" :: "v"(1u), "v"(l0), "s"(rfs), "s"(so) : "memory");
    };
    unsigned cur = ticket();
    unsigned nxt = cur + 1, prev = NITEM, slot = 0;
    grab();
    if (cur < NITEM) need(cur);
    load(0, cur);
    while (cur < NITEM) {
      if (nxt < NITEM) need(nxt);
      load(slot ^ 1, nxt);
      asm volatile("s_waitcnt vmcnt(3)" ::: "memory");
      if (prev < NITEM) arrive(prev);
      const unsigned nn = ticket();
      grab();
      emit(slot, cur);
      prev = cur; cur = nxt; nxt = nn; slot ^= 1;
    }
    asm volatile("s_waitcnt vmcnt(0)" ::: "memory");
    if (prev < NITEM) arrive(prev);
    return;
  }

  const int t1 = (te >> 14) & 7, i1 = (te >> 11) & 7, t2 = (te >> 8) & 7, i2 = (te >> 5) & 7, tu = (te >> 2) & 7, pj = te & 3;
  const int srcW = rank ^ tu;
  char* const donep = a.ws[srcW & 7] + DONE_OFF + b * 4;
  lu32* sw = (lu32*)(smem + NBUF * STAGE_BYTES);
  if (wv >= 4) {
    const int lw = wv - 4;
    const int kind = lw >> 1;
    const int src = lw == 0 ? rank ^ t1 : (lw == 1 ? rank ^ t2 : srcW);
    const int unit = lw == 0 ? i1 : (lw == 1 ? i2 : 2 * pj + (lw & 1));
    const char* g = me + stg_off(kind, src, unit) + lane * 16;
    const bool loc = src == rank;
    const char* pollp = loc ? me + READY_OFF + (size_t)(((src & 7) * 2 + kind) * 8 + unit) * 128
                            : me + ARR_OFF + (size_t)(((src & 7) * 2 + kind) * 8 + unit) * 256;
    const unsigned pthr = loc ? base + 1 : 2 * call;
    const int nword = loc ? 16 : 64;
    auto eval = [&](int q, unsigned v) -> int {
      const int w0 = loc ? q >> 2 : q;
      const unsigned long long nr = __ballot(lane < 16 && w0 + lane < nword && v < pthr);
      const int f = __builtin_ctz((unsigned)nr | 0x10000u);
      return min(64, loc ? 4 * (w0 + f) : w0 + f);
    };
    auto wordp = [&](int q) { return pollp + min((loc ? q >> 2 : q) + (lane & 15), nword - 1) * 4; };
    int avail = 0, pend = -1;
    int nb = 0;
    for (int s = -PF; s < 64; s++) {
      __builtin_amdgcn_s_barrier();
      asm volatile("" ::: "memory");
      if (s + PF < 64) {
        const int q = s + PF;
        if (q >= avail) {
          for (Spin sp;;) {
            avail = eval(q, ld_sys(wordp(q)));
            if (avail > q) break;
            if (sp.expired(me, call)) { avail = 64; break; }
          }

        }
        const unsigned* gp = (const unsigned*)(g + (size_t)q * (2 * BLK));
        unsigned* lp = (unsigned*)(smem + nb * STAGE_BYTES + lw * (2 * BLK));
        nb = nb == NBUF - 1 ? 0 : nb + 1;
        __builtin_amdgcn_global_load_lds(gp, lp, 16, 0, AUX_SYS);
        __builtin_amdgcn_global_load_lds(gp, lp, 16, 1024, AUX_SYS);
        __builtin_amdgcn_global_load_lds(gp, lp, 16, 2048, AUX_SYS);
        __builtin_amdgcn_global_load_lds(gp, lp, 16, 3072, AUX_SYS);
        __builtin_amdgcn_global_load_lds(gp + 1024, lp + 1024, 16, 0, AUX_SYS);
        __builtin_amdgcn_global_load_lds(gp + 1024, lp + 1024, 16, 1024, AUX_SYS);
        __builtin_amdgcn_global_load_lds(gp + 1024, lp + 1024, 16, 2048, AUX_SYS);
        __builtin_amdgcn_global_load_lds(gp + 1024, lp + 1024, 16, 3072, AUX_SYS);
        if ((q & 1) == 0) {
          if (avail < 64 && avail <= q + 2 * LOOK + 1) {
            asm volatile("global_load_dword v200, %0, off sc0 sc1" :: "v"(wordp(avail)) : "v200", "memory");
            pend = avail;
          }
        } else if (pend >= 0) {
          unsigned pv;
          asm volatile("s_waitcnt vmcnt(8)\n\tv_mov_b32 %0, v200" : "=v"(pv) :: "v200", "memory");
          avail = max(avail, eval(pend, pv));
          pend = -1;
        }
        if (PF == 4) asm volatile("s_waitcnt vmcnt(24)" ::: "memory");
        else if (PF == 3) asm volatile("s_waitcnt vmcnt(16)" ::: "memory");
        else asm volatile("s_waitcnt vmcnt(8)" ::: "memory");
      } else {
        const int rem = 62 - s;
        if (rem >= 2) asm volatile("s_waitcnt vmcnt(16)" ::: "memory");
        else if (rem == 1) asm volatile("s_waitcnt vmcnt(8)" ::: "memory");
        else asm volatile("s_waitcnt vmcnt(0)" ::: "memory");
      }
    }
    if (lw == 0 && lane == 0) sw[4] = 0;
    asm volatile("s_waitcnt lgkmcnt(0)" ::: "memory");
    __builtin_amdgcn_s_barrier();
    if (lw == 0) {
      const char* rp = me + RES_OFF + (b & 7) * 128;
      for (Spin sp;;) {
        u32x2 r;
        unsigned tm;
        asm volatile("global_load_dwordx2 %0, %2, off sc1\n\tglobal_load_dword %1, %3, off sc1\n\ts_waitcnt vmcnt(0)"
                     : "=&v"(r), "=&v"(tm) : "v"(rp), "v"(me + TMO_OFF) : "memory");
        const bool fin = __builtin_amdgcn_readfirstlane(r[1]) == call;
        bool bad = __builtin_amdgcn_readfirstlane(tm) == call || (fin && __builtin_amdgcn_readfirstlane(r[0]) == 0x7fffffffu);
        if (!fin && !bad && sp.expired(me, call)) bad = true;
        if (fin || bad) {
          if (lane == 0) {
            sw[5] = r[0];
            sw[6] = bad;
            __hip_atomic_store(sw + 4, 1u, __ATOMIC_RELEASE, __HIP_MEMORY_SCOPE_WORKGROUP);
          }
          break;
        }
      }
    }
    return;
  }
  const int xu = wv >> 1, wu = wv & 1;
  f32x4 acc[4][4];
#pragma unroll
  for (int f = 0; f < 4; f++)
#pragma unroll
    for (int j = 0; j < 4; j++) acc[f][j] = f32x4{0.f, 0.f, 0.f, 0.f};
  const unsigned xoff = xu * (2 * BLK) + lane * 16;
  unsigned woff[4];
  {
    const int r = lane & 3, q = (lane >> 2) & 3, kq = lane >> 4;
#pragma unroll
    for (int j = 0; j < 4; j++) woff[j] = (2 + wu) * (2 * BLK) + r * 1024 + kq * 256 + q * 64 + ((j ^ r) * 16);
  }
  {
    bf16x8 xa[4], wa[4], xb[4], wb[4];
#pragma unroll
    for (int f = 0; f < 4; f++) { xb[f] = bf16x8{}; wb[f] = bf16x8{}; }
    for (int s = -PF; s < 0; s++) __builtin_amdgcn_s_barrier();
    int cb = 0;
#define ILV() do { _Pragma("unroll") for (int q_ = 0; q_ < 8; q_++) { __builtin_amdgcn_sched_group_barrier(0x008, 1, 0); \
    __builtin_amdgcn_sched_group_barrier(0x100, 1, 0); } __builtin_amdgcn_sched_group_barrier(0x008, 8, 0); } while (0)
    for (int s = 0; s < 64; s++) {
      asm volatile("s_waitcnt lgkmcnt(0)" ::: "memory");
      __builtin_amdgcn_s_barrier();
      asm volatile("" ::: "memory");

      const char* sb = smem + cb * STAGE_BYTES;
      cb = cb == NBUF - 1 ? 0 : cb + 1;
      __builtin_amdgcn_sched_barrier(0);
      wa[0] = *(const bf16x8*)(sb + woff[0]); xa[0] = *(const bf16x8*)(sb + xoff);
#pragma unroll
      for (int f = 1; f < 4; f++) wa[f] = *(const bf16x8*)(sb + woff[f]);
#pragma unroll
      for (int f = 1; f < 4; f++) xa[f] = *(const bf16x8*)(sb + xoff + f * 1024);
#pragma unroll
      for (int f = 0; f < 4; f++)
#pragma unroll
        for (int j = 0; j < 4; j++) acc[f][j] = __builtin_amdgcn_mfma_f32_16x16x32_bf16(wb[j], xb[f], acc[f][j], 0, 0, 0);
      ILV();
      __builtin_amdgcn_sched_barrier(0);
      wb[0] = *(const bf16x8*)(sb + woff[0] + BLK); xb[0] = *(const bf16x8*)(sb + xoff + BLK);
#pragma unroll
      for (int f = 1; f < 4; f++) wb[f] = *(const bf16x8*)(sb + woff[f] + BLK);
#pragma unroll
      for (int f = 1; f < 4; f++) xb[f] = *(const bf16x8*)(sb + xoff + BLK + f * 1024);
#pragma unroll
      for (int f = 0; f < 4; f++)
#pragma unroll
        for (int j = 0; j < 4; j++) acc[f][j] = __builtin_amdgcn_mfma_f32_16x16x32_bf16(wa[j], xa[f], acc[f][j], 0, 0, 0);
      ILV();
      __builtin_amdgcn_sched_barrier(0);
    }
#pragma unroll
    for (int f = 0; f < 4; f++)
#pragma unroll
      for (int j = 0; j < 4; j++) acc[f][j] = __builtin_amdgcn_mfma_f32_16x16x32_bf16(wb[j], xb[f], acc[f][j], 0, 0, 0);
  }
  float mf = 0.f;
#pragma unroll
  for (int f = 0; f < 4; f++)
#pragma unroll
    for (int j = 0; j < 4; j++) {
      const f32x4 t = acc[f][j];
      mf = fmaxf(fmaxf(mf, fabsf(t[0])), fabsf(t[1]));
      mf = fmaxf(fmaxf(mf, fabsf(t[2])), fabsf(t[3]));
    }
  unsigned m = wave_max_u32(__float_as_uint(mf));
  if (lane == 0) sw[wv] = m;
  asm volatile("s_waitcnt lgkmcnt(0)" ::: "memory");
  __builtin_amdgcn_s_barrier();
  asm volatile("" ::: "memory");
  m = max(max(sw[0], sw[1]), max(sw[2], sw[3]));
  if (tid < NR) {
    gu64* g = (gu64*)(wsl + AMAX_OFF + (size_t)(rank * NGEMM + b) * 8);
    __hip_atomic_store(g, ((unsigned long long)call << 32) | m, __ATOMIC_RELAXED, __HIP_MEMORY_SCOPE_SYSTEM);
  }
  while (__builtin_amdgcn_readfirstlane(__hip_atomic_load(sw + 4, __ATOMIC_ACQUIRE, __HIP_MEMORY_SCOPE_WORKGROUP)) == 0)
    __builtin_amdgcn_s_sleep(1);
  const float inv = 448.0f / __uint_as_float(__builtin_amdgcn_readfirstlane(sw[5]));
  const bool bad = __builtin_amdgcn_readfirstlane(sw[6]) != 0;
  {
    const int tx = xu == 0 ? t1 : t2, ix = xu == 0 ? i1 : i2;
    unsigned char* optr = outp + (size_t)((rank ^ tx) * SH + ix * 64 + (lane & 15)) * SH + pj * 128 + wu * 64 + (lane >> 4) * 16;
#pragma unroll
    for (int f = 0; f < 4; f++) {
      u32x4 q;
#pragma unroll
      for (int j = 0; j < 4; j++) {
        int w = __builtin_amdgcn_cvt_pk_fp8_f32(acc[f][j][0] * inv, acc[f][j][1] * inv, 0, false);
        w = __builtin_amdgcn_cvt_pk_fp8_f32(acc[f][j][2] * inv, acc[f][j][3] * inv, w, true);
        q[j] = bad ? 0x7f7f7f7fu : (unsigned)w;
      }
      if (tu != 0) asm volatile("global_store_dwordx4 %0, %1, off sc0 sc1\n\ts_nop 1" ::"v"(optr + (size_t)f * 16 * SH), "v"(q) : "memory");
      else *(u32x4*)(optr + (size_t)f * 16 * SH) = q;
    }
  }
  if (tu != 0) {
    asm volatile("s_waitcnt vmcnt(0)" ::: "memory");
    __builtin_amdgcn_s_barrier();
    if (tid == 0) st_sys(donep, call);
  }
}
}

extern "C" void kernel_launch(int rank, int nranks, ncclComm_t comm, hipStream_t stream, void* const* d_in,
                              const int* in_sizes, int n_in, void* d_out, int out_size, const int* in_dtypes,
                              int out_dtype, void* const* d_out_all, void* const* d_ws_all, size_t ws_bytes) {
  (void)comm; (void)in_sizes; (void)n_in; (void)d_out; (void)out_size; (void)in_dtypes; (void)out_dtype;
  if (nranks != NR || rank < 0 || rank >= NR || ws_bytes < WS_NEED) {
    fprintf(stderr, "unsupported configuration\n");
    return;
  }
  Args a;
  a.x = (const float*)d_in[0];
  a.w = (const float*)d_in[1];
  for (int p = 0; p < NR; p++) {
    a.ws[p] = (char*)d_ws_all[p];
    a.out[p] = (unsigned char*)d_out_all[p];
  }
  a.rank = rank;
  a.pad = 0;
  fused_ag_gemm<<<NWG, 512, 0, stream>>>(a);
  hipError_t e = hipGetLastError();
  if (e != hipSuccess) fprintf(stderr, "launch error %s\n", hipGetErrorString(e));
}
```

```cpp
#include <hip/hip_runtime.h>
#include <rccl/rccl.h>
#include <cstdio>
#include <cstdint>

typedef unsigned u32x4 __attribute__((ext_vector_type(4)));
typedef unsigned u32x3 __attribute__((ext_vector_type(3)));
typedef unsigned u32x2 __attribute__((ext_vector_type(2)));
typedef float f32x4 __attribute__((ext_vector_type(4)));
typedef __bf16 bf16x8 __attribute__((ext_vector_type(8)));
typedef __bf16 bf16x2 __attribute__((ext_vector_type(2)));
typedef float f32x2 __attribute__((ext_vector_type(2)));

#define NPW 5
#define NPUSH 1
#define LOOK 1
#define STAG0 60
#define STAGGER 25

namespace {
constexpr int NR = 8;
constexpr int KDIM = 4096;
constexpr int SH = 512;
constexpr unsigned BLK = 4096;
constexpr unsigned UNIT_BYTES = 128 * BLK;
constexpr unsigned PBLK = 2944;
constexpr unsigned PUNIT_BYTES = 128 * PBLK;
constexpr unsigned NEXC = 41;
constexpr size_t STG_KIND_BYTES = (size_t)NR * 8 * UNIT_BYTES;
constexpr size_t EXP_OFF = 2 * STG_KIND_BYTES;
constexpr size_t FLAG_OFF = EXP_OFF + (size_t)16 * PUNIT_BYTES;
constexpr size_t READY_OFF = FLAG_OFF;
constexpr size_t ARR_OFF = READY_OFF + NR * 16 * 128;
constexpr size_t AMAX_OFF = ARR_OFF + NR * 16 * 256;
constexpr size_t DONE_OFF = AMAX_OFF + NR * 128 * 8;
constexpr size_t CTR_OFF = DONE_OFF + 1024;
constexpr size_t TMO_OFF = CTR_OFF + 128;
constexpr size_t QCTR_OFF = TMO_OFF + 128;
constexpr size_t RES_OFF = QCTR_OFF + 16 * 128;
constexpr size_t END_OFF = RES_OFF + 8 * 128;
constexpr size_t WS_NEED = END_OFF;
static_assert(FLAG_OFF % 128 == 0, "align");
constexpr int NGEMM = 128, NPULL = 35, NWG = 256;
constexpr unsigned long long DEADLINE = 30000000ull;
constexpr unsigned STAGE_BYTES = 8 * BLK;
#define NBUFS 4
constexpr int NBUF = NBUFS;
constexpr int PF = NBUF - 1;

__constant__ unsigned TILE_TAB[NGEMM] = {16420,2145,5154,5283,14530,10241,8928,10432,69476,67649,7906,47971,70370,15553,17664,15424,3116,74817,8450,78019,77058,16577,113792,55360,18284,75905,12130,83011,3238,113377,121888,75872,70924,82785,73762,70820,13574,1059,3115,77856,77964,5158,78690,82660,69798,18147,7435,4311,8207,55430,3939,112708,78566,76,47243,47287,121935,75622,6307,75,112902,11404,113803,55351,1081,3112,73859,11115,10991,13484,4373,69751,9433,77064,10340,75979,12335,48876,9973,12377,12153,121928,12420,78923,76015,9271,67637,47225,17561,5167,74948,55518,79599,74935,73845,75833,83033,75023,6216,70846,11477,78103,77685,2234,2238,2173,12008,78590,69845,12381,47258,55386,8478,70813,73864,81950,9501,48349,68890,69850,68766,76669,78024,112734,10493,69373,75642,79034};
__constant__ unsigned PULL_TAB[8] = {0, 0xdc980, 0xfe987, 0xfe983, 0x65421, 0xfeba0, 0xdcba7, 0xdcba3};
constexpr unsigned long long DEST_MASK_X = 0x4511111189111123ull, DEST_MASK_W = 0x2d2dc3c3e1e10f0full;

struct Args {
  const float* x;
  const float* w;
  char* ws[NR];
  unsigned char* out[NR];
  int rank;
  int pad;
};

typedef __attribute__((address_space(1))) unsigned gu32;
typedef __attribute__((address_space(1))) unsigned long long gu64;
typedef __attribute__((address_space(3))) unsigned lu32;
typedef __attribute__((address_space(3))) unsigned char lu8;
typedef __attribute__((address_space(3))) unsigned short lu16;
typedef __attribute__((address_space(3))) u32x4 lu32x4;

__device__ __forceinline__ unsigned ld_sys(const void* p) { return __hip_atomic_load((const gu32*)p, __ATOMIC_RELAXED, __HIP_MEMORY_SCOPE_SYSTEM); }
__device__ __forceinline__ void st_sys(void* p, unsigned v) { __hip_atomic_store((gu32*)p, v, __ATOMIC_RELAXED, __HIP_MEMORY_SCOPE_SYSTEM); }
__device__ __forceinline__ unsigned ld_agent(const void* p) { return __hip_atomic_load((const gu32*)p, __ATOMIC_RELAXED, __HIP_MEMORY_SCOPE_AGENT); }
__device__ __forceinline__ void st_agent(void* p, unsigned v) { __hip_atomic_store((gu32*)p, v, __ATOMIC_RELAXED, __HIP_MEMORY_SCOPE_AGENT); }

__device__ __forceinline__ __amdgpu_buffer_rsrc_t mkrsrc(const void* p, unsigned bytes) {
  unsigned long long a = (unsigned long long)p;
  unsigned lo = __builtin_amdgcn_readfirstlane((unsigned)a), hi = __builtin_amdgcn_readfirstlane((unsigned)(a >> 32));
  void* q = (void*)(((unsigned long long)hi << 32) | lo);
  return __builtin_amdgcn_make_buffer_rsrc(q, 0, bytes, 0x00020000);
}
__device__ __forceinline__ u32x4 mkrsrc_words(const void* p, unsigned bytes) {
  unsigned long long a = (unsigned long long)p;
  return u32x4{(unsigned)__builtin_amdgcn_readfirstlane((unsigned)a), (unsigned)__builtin_amdgcn_readfirstlane((unsigned)(a >> 32)) & 0xffffu,
               bytes, 0x00020000u};
}
__device__ __forceinline__ size_t stg_off(int kind, int src, int unit) {
  return (size_t)kind * STG_KIND_BYTES + (size_t)(src * 8 + unit) * UNIT_BYTES;
}
__device__ __forceinline__ unsigned cvt2(float a, float b) {
  f32x2 v = {a, b};
  bf16x2 r = __builtin_convertvector(v, bf16x2);
  return __builtin_bit_cast(unsigned, r);
}
__device__ __forceinline__ unsigned perm(unsigned hi, unsigned lo, unsigned sel) { return __builtin_amdgcn_perm(hi, lo, sel); }
__device__ __forceinline__ unsigned wave_max_u32(unsigned v) {
  v = max(v, (unsigned)__builtin_amdgcn_update_dpp(0, (int)v, 0x111, 0xf, 0xf, false));
  v = max(v, (unsigned)__builtin_amdgcn_update_dpp(0, (int)v, 0x112, 0xf, 0xf, false));
  v = max(v, (unsigned)__builtin_amdgcn_update_dpp(0, (int)v, 0x114, 0xf, 0xf, false));
  v = max(v, (unsigned)__builtin_amdgcn_update_dpp(0, (int)v, 0x118, 0xf, 0xf, false));
  v = max(v, (unsigned)__builtin_amdgcn_update_dpp(0, (int)v, 0x142, 0xa, 0xf, false));
  v = max(v, (unsigned)__builtin_amdgcn_update_dpp(0, (int)v, 0x143, 0xc, 0xf, false));
  return (unsigned)__builtin_amdgcn_readlane((int)v, 63);
}

struct Spin {
  unsigned it = 0;
  unsigned long long t0 = 0;
  __device__ __forceinline__ bool expired(char* me, unsigned call) {
    if (it == 0) t0 = __builtin_amdgcn_s_memrealtime();
    if ((++it & 15) == 0) {
      if (ld_agent(me + TMO_OFF) == call || __builtin_amdgcn_s_memrealtime() - t0 > DEADLINE) {
        st_agent(me + TMO_OFF, call);
        return true;
      }
    }
    __builtin_amdgcn_s_sleep(1);
    return false;
  }
};
constexpr int AUX_SYS = 17, AUX_AGENT = 16;

__device__ __forceinline__ unsigned piece_off(int kind, int p, int lane) {
  const int kq = lane >> 4, q = (lane >> 2) & 3, j = lane & 3;
  return kind == 0 ? p * 1024 + lane * 16 : p * 1024 + kq * 256 + q * 64 + ((j ^ p) * 16);
}

__global__ __launch_bounds__(512, 2) void fused_ag_gemm(Args a) {
  __shared__ __attribute__((aligned(16))) char smem[NBUF * STAGE_BYTES + 128];
  const int tid = threadIdx.x, lane = tid & 63;
  const int wv = __builtin_amdgcn_readfirstlane(tid >> 6);
  const int b = blockIdx.x;
  constexpr int NH0 = 16 * NPUSH;
  const int task = b < NGEMM ? b + NH0 + NPULL
                             : (b < NGEMM + NPULL ? b - NGEMM + NH0 : (b - (NGEMM + NPULL) < NH0 ? b - (NGEMM + NPULL) : b));
  const int c16 = task >> 4, u16 = task & 15, kind = u16 >> 3, unit = u16 & 7;
  const int ks = c16 * 8 + wv;
  const int kq = lane >> 4, l15 = lane & 15;
  if (c16 > NPUSH) {
    const unsigned long long tstart = __builtin_amdgcn_s_memrealtime();
    while (__builtin_amdgcn_s_memrealtime() - tstart < (unsigned long long)(STAG0 + (c16 - NPUSH - 1) * STAGGER)) __builtin_amdgcn_s_sleep(1);
  }
  f32x4 v[8];
  {
    const float* px = a.x + (size_t)(unit * 64 + l15) * KDIM + ks * 32 + 4 * kq;
    const float* pw = a.w + (size_t)(ks * 32 + 4 * kq) * SH + unit * 64 + 4 * l15;
    const float* p = kind == 0 ? px : pw;
#pragma unroll
    for (int i = 0; i < 8; i++) {
      const unsigned ox = (i >> 1) * 16 * KDIM + (i & 1) * 16, ow = ((i & 3) + (i >> 2) * 16) * SH;
      v[i] = *(const f32x4*)(p + (kind == 0 ? ox : ow));
    }
  }
  asm volatile("" ::: "memory");
  const int rank = a.rank;
  char* me = a.ws[rank];
  const unsigned ctr_v = ld_agent(me + CTR_OFF);
  const unsigned te = TILE_TAB[b & (NGEMM - 1)];
  const bool is_pull = b >= NGEMM && b < NGEMM + NPULL;
  const int plink = is_pull ? 1 + (b - NGEMM) / 5 : 0;
  const unsigned ue = PULL_TAB[plink];
  char* const wsl = a.ws[lane & 7];
  unsigned char* const outp = a.out[(rank ^ (int)(te >> 2)) & 7];
  const u32x4 rfs = mkrsrc_words(me + FLAG_OFF, (unsigned)(END_OFF - FLAG_OFF));
  const unsigned l0 = lane == 0 ? 0u : 0x40000000u;
  unsigned call, base;
  {
    const __amdgpu_buffer_rsrc_t rd = mkrsrc(me + stg_off(kind, rank, unit), UNIT_BYTES);
    call = __builtin_amdgcn_readfirstlane(ctr_v) + 1;
    base = call * 64;
    if (is_pull && wv < NPW) {
      const unsigned qoff = (unsigned)(QCTR_OFF - FLAG_OFF) + ((call & 1) * 8 + plink) * 128;
      asm volatile("v_mov_b32 v200, 2\n\ts_nop 4\n\tbuffer_atomic_add v200, %0, %1, %2 offen sc0"
                   :: "v"(l0), "s"(rfs), "s"(qoff) : "v200", "memory");
    }
    u32x4 o[4];
    if (kind == 0) {
#pragma unroll
      for (int f = 0; f < 4; f++)
        o[f] = u32x4{cvt2(v[2 * f][0], v[2 * f][1]), cvt2(v[2 * f][2], v[2 * f][3]),
                     cvt2(v[2 * f + 1][0], v[2 * f + 1][1]), cvt2(v[2 * f + 1][2], v[2 * f + 1][3])};
    } else {
#pragma unroll
      for (int r = 0; r < 4; r++)
        o[r] = u32x4{cvt2(v[0][r], v[1][r]), cvt2(v[2][r], v[3][r]), cvt2(v[4][r], v[5][r]), cvt2(v[6][r], v[7][r])};
    }
    const unsigned dmask = (unsigned)((kind ? DEST_MASK_W : DEST_MASK_X) >> (8 * unit)) & 0xff;
    if (c16 < NPUSH) {
#pragma unroll
      for (int p = 1; p < NR; p++)
        if ((dmask >> p) & 1) {
          const __amdgpu_buffer_rsrc_t rq = mkrsrc(a.ws[rank ^ p] + stg_off(kind, rank, unit), UNIT_BYTES);
#pragma unroll
          for (int i = 0; i < 4; i++)
            __builtin_amdgcn_raw_buffer_store_b128(o[i], rq, ks * BLK + piece_off(kind, i, lane), 0, AUX_SYS);
        }
    } else {
      float mx = 0.f;
#pragma unroll
      for (int i = 0; i < 8; i++) {
        mx = fmaxf(mx, fmaxf(fabsf(v[i][0]), fabsf(v[i][1])));
        mx = fmaxf(mx, fmaxf(fabsf(v[i][2]), fabsf(v[i][3])));
      }
      const unsigned mxb = wave_max_u32(__float_as_uint(mx));
      const unsigned emax = (cvt2(__uint_as_float(mxb), 0.f) >> 7) & 0xff;
      const float thr1 = __uint_as_float(emax << 23), thr2 = thr1 * 0.5f;
      unsigned n1 = 0, n2 = 0;
#pragma unroll
      for (int i = 0; i < 8; i++) {
        n1 += __builtin_popcountll(__ballot(fabsf(v[i][0]) >= thr1)) + __builtin_popcountll(__ballot(fabsf(v[i][2]) >= thr1));
        n2 += __builtin_popcountll(__ballot(fabsf(v[i][0]) >= thr2)) + __builtin_popcountll(__ballot(fabsf(v[i][2]) >= thr2));
      }
      const int top = (int)emax - (n1 <= 10 ? 1 : 0) - (n2 <= 10 ? 1 : 0);
      const unsigned delta = (top & 1) ^ 1, tp = ((top + (int)delta) >> 1) & 0x7f;
      const unsigned Md = delta ? 0x80808080u : 0u;
      const unsigned b80 = tp * 0x01010101u | 0x80808080u;
      const unsigned T = (b80 - 0x03020100u) & 0x7f7f7f7fu;
      unsigned LP[8], HH[8], S = 0, K0 = 0, K1 = 0, M = 0;
#pragma unroll
      for (int i = 0; i < 8; i++) {
        const unsigned d0 = o[i >> 1][2 * (i & 1)], d1 = o[i >> 1][2 * (i & 1) + 1];
        const unsigned L = perm(d1, d0, 0x06040200u), H = perm(d1, d0, 0x07050301u);
        const unsigned pair = ((H & 0x7f7f7f7fu) + ((L & Md) >> 7)) & 0x7f7f7f7fu;
        const unsigned code = (b80 - pair) & 0x03030303u;
        const unsigned mis = perm(T, T, code) ^ pair;
        LP[i] = L ^ Md;
        HH[i] = H;
        if (i < 4) K0 |= code << (2 * i); else K1 |= code << (2 * (i - 4));
        S |= (H & 0x80808080u) >> (7 - i);
        M |= __builtin_amdgcn_udot4(((mis + 0x7f7f7f7fu) & 0x80808080u) >> 7, 0x08040201u, 0u, false) << (4 * i);
      }
      char* hsc = smem + wv * 2304;
      lu32* hdr = (lu32*)(hsc + 2048);
      const unsigned nl = __builtin_popcount(M);
      unsigned inc = nl;
      inc += __builtin_amdgcn_update_dpp(0, (int)inc, 0x111, 0xf, 0xf, false);
      inc += __builtin_amdgcn_update_dpp(0, (int)inc, 0x112, 0xf, 0xf, false);
      inc += __builtin_amdgcn_update_dpp(0, (int)inc, 0x114, 0xf, 0xf, false);
      inc += __builtin_amdgcn_update_dpp(0, (int)inc, 0x118, 0xf, 0xf, false);
      inc += __builtin_amdgcn_update_dpp(0, (int)inc, 0x142, 0xa, 0xf, false);
      inc += __builtin_amdgcn_update_dpp(0, (int)inc, 0x143, 0xc, 0xf, false);
      const unsigned cnt = (unsigned)__builtin_amdgcn_readlane((int)inc, 63);
      if (cnt > 0 && cnt <= NEXC) {
        *(lu32x4*)(hsc + lane * 32) = u32x4{HH[0], HH[1], HH[2], HH[3]};
        *(lu32x4*)(hsc + lane * 32 + 16) = u32x4{HH[4], HH[5], HH[6], HH[7]};
        unsigned at = inc - nl;
        while (__any(M != 0)) {
          if (M != 0) {
            const unsigned jv = __builtin_ctz(M);
            M &= M - 1;
            const unsigned char hi = *(const lu8*)(hsc + lane * 32 + jv);
            const unsigned idx = (piece_off(kind, jv >> 3, lane) >> 1) + (jv & 7);
            *(lu16*)((char*)hdr + 4 + 2 * at) = (unsigned short)idx;
            *(lu8*)((char*)hdr + 4 + 2 * NEXC + at) = hi;
            at++;
          }
        }
      }
      const __amdgpu_buffer_rsrc_t rp = mkrsrc(me + EXP_OFF + (size_t)u16 * PUNIT_BYTES, PUNIT_BYTES);
      const unsigned pc = ks * PBLK;
      __builtin_amdgcn_raw_buffer_store_b128(u32x4{LP[0], LP[1], LP[2], LP[3]}, rp, pc + lane * 16, 0, AUX_SYS);
      __builtin_amdgcn_raw_buffer_store_b128(u32x4{LP[4], LP[5], LP[6], LP[7]}, rp, pc + 1024 + lane * 16, 0, AUX_SYS);
      __builtin_amdgcn_raw_buffer_store_b96(u32x3{S, K0, K1}, rp, pc + 2048 + lane * 12, 0, AUX_SYS);
      unsigned hv = hdr[lane & 31];
      if (lane == 0) hv = tp | (delta << 7) | (cnt > NEXC ? 0x10000u : cnt << 8);
      if (lane < 32) __builtin_amdgcn_raw_buffer_store_b32(hv, rp, pc + 2816 + lane * 4, 0, AUX_SYS);
    }
#pragma unroll
    for (int p = 0; p < 4; p++)
      __builtin_amdgcn_raw_buffer_store_b128(o[p], rd, ks * BLK + piece_off(kind, p, lane), 0, AUX_SYS);
    asm volatile("s_waitcnt vmcnt(0)" ::: "memory");
    const int rel = rank ^ lane;
    if (c16 < NPUSH) {
      if (lane < NR && rel != 0 && ((dmask >> rel) & 1))
        __hip_atomic_fetch_add((gu32*)(wsl + ARR_OFF + (size_t)(rank * 16 + u16) * 256 + (ks >> 1) * 4), 1u, __ATOMIC_RELAXED,
                               __HIP_MEMORY_SCOPE_SYSTEM);
    }
    __builtin_amdgcn_s_barrier();
    if (wv == 0 && lane < NR && ((dmask >> rel) & 1) && (rel == 0 || c16 >= NPUSH))
      st_sys(wsl + READY_OFF + (size_t)(rank * 16 + u16) * 128 + c16 * 4, base + 1);
  }

  if (b >= NGEMM + NPULL) {
    if (b == NWG - 1) {
      {
        lu32* sv = (lu32*)smem;
        __builtin_amdgcn_s_barrier();
        if (tid < 3) sv[tid] = 0;
        asm volatile("s_waitcnt lgkmcnt(0)" ::: "memory");
        __builtin_amdgcn_s_barrier();
        const char* gp = me + AMAX_OFF + (size_t)tid * 16;
        bool have = false, bad = false;
        unsigned gm = 0;
        for (Spin sp;;) {
          unsigned tm = 0;
          if (!have) {
            u32x4 g0;
            asm volatile("global_load_dwordx4 %0, %2, off sc0 sc1\n\tglobal_load_dword %1, %3, off sc1\n\ts_waitcnt vmcnt(0)"
                         : "=&v"(g0), "=&v"(tm) : "v"(gp), "v"(me + TMO_OFF) : "memory");
            have = g0[1] == call && g0[3] == call;
            gm = max(g0[0], g0[2]);
          }
          if (__any(tm == call)) { bad = true; break; }
          if (__all(have)) break;
          if (sp.expired(me, call)) { bad = true; break; }
        }
        const unsigned wm = wave_max_u32(gm);
        unsigned last = 0;
        if (lane == 0) {
          __hip_atomic_fetch_max(sv, wm, __ATOMIC_RELAXED, __HIP_MEMORY_SCOPE_WORKGROUP);
          if (bad) __hip_atomic_store(sv + 2, 1u, __ATOMIC_RELAXED, __HIP_MEMORY_SCOPE_WORKGROUP);
          last = __hip_atomic_fetch_add(sv + 1, 1u, __ATOMIC_ACQ_REL, __HIP_MEMORY_SCOPE_WORKGROUP);
        }
        if (__builtin_amdgcn_readfirstlane(last) == 7) {
          const unsigned am = sv[2] ? 0x7fffffffu : sv[0];
          if (lane < 8) __hip_atomic_store((gu64*)(me + RES_OFF + lane * 128), ((unsigned long long)call << 32) | am,
                                           __ATOMIC_RELAXED, __HIP_MEMORY_SCOPE_AGENT);
        }
      }
      if (wv == 0) {
        bool ok = true;
        {
          const bool a0 = ((TILE_TAB[2 * lane] >> 2) & 7) != 0, a1 = ((TILE_TAB[2 * lane + 1] >> 2) & 7) != 0;
          for (Spin sp;;) {
            u32x2 d;
            asm volatile("global_load_dwordx2 %0, %1, off sc0 sc1\n\ts_waitcnt vmcnt(0)" : "=v"(d) : "v"(me + DONE_OFF + lane * 8) : "memory");
            if (__all((!a0 || d[0] >= call) && (!a1 || d[1] >= call))) break;
            if (sp.expired(me, call)) { ok = false; break; }
          }
        }
        if (lane < 8) st_agent(me + QCTR_OFF + (size_t)(((call + 1) & 1) * 8 + lane) * 128, 0u);
        if (lane == 0) {
          if (!ok) a.out[rank][(size_t)(rank * SH + 64) * SH + 128] = 0x7f;
          st_agent(me + CTR_OFF, call);
        }
      }
    }
    return;
  }
  if (is_pull) {
    if (wv >= NPW) return;
    const unsigned src = (unsigned)(rank ^ plink);
    if (src >= NR) return;
    const __amdgpu_buffer_rsrc_t rraw = mkrsrc(a.ws[src], 2 * STG_KIND_BYTES);
    const __amdgpu_buffer_rsrc_t rd = mkrsrc(me, 2 * STG_KIND_BYTES);
    const u32x4 rss = mkrsrc_words(a.ws[src] + EXP_OFF, 16 * PUNIT_BYTES);
    const unsigned wperm = piece_off(1, 0, lane);
    char* scr = smem + 1024 + wv * BLK;
    constexpr unsigned SLOT = 3072;
    char* ring = smem + 1024 + NPW * BLK + wv * 2 * SLOT;
    unsigned long long rdy03 = 0;
    unsigned rdy4 = 0;
    bool ok = true;
    constexpr unsigned C0 = 2 * NPUSH, NITEM = (32 - C0) * 20;
    const unsigned qoff = (unsigned)(QCTR_OFF - FLAG_OFF) + ((call & 1) * 8 + plink) * 128;
    auto grab = [&]() {
      asm volatile("v_mov_b32 v200, 1\n\ts_nop 4\n\tbuffer_atomic_add v200, %0, %1, %2 offen sc0"
                   :: "v"(l0), "s"(rfs), "s"(qoff) : "v200", "memory");
    };
    auto ticket = [&]() -> unsigned {
      unsigned v;
      asm volatile("v_mov_b32 %0, v200" : "=v"(v) :: "v200", "memory");
      return __builtin_amdgcn_readfirstlane(v);
    };
    auto need = [&](unsigned n) {
      const unsigned cq = n / 20, c = C0 + cq, ui = (n - cq * 20) % 5, c2 = c >> 1;
      for (Spin sp;;) {
        const unsigned m = ui == 4 ? rdy4 : (unsigned)(rdy03 >> (16 * ui));
        if (((m >> c2) & 1) || !ok) return;
        const unsigned u16 = (ue >> (4 * ui)) & 15;
        const unsigned v = lane < 16 ? ld_sys(me + READY_OFF + (size_t)(src * 16 + u16) * 128 + lane * 4) : 0u;
        const unsigned mk = (unsigned)__ballot(v >= base + 1) & 0xffffu;
        if (ui == 4) rdy4 = mk; else rdy03 |= (unsigned long long)mk << (16 * ui);
        if ((mk >> c2) & 1) return;
        if (sp.expired(me, call)) ok = false;
      }
    };
    const unsigned vpay = lane * 16, vlast = lane < 56 ? lane * 16 : 0x40000000u;
    auto load = [&](unsigned slot, unsigned n) {
      const bool val = n < NITEM;
      const unsigned nn = val ? n : 0;
      const unsigned cq = nn / 20, c = C0 + cq, r = nn - cq * 20, w = r / 5, u16 = (ue >> (4 * (r - w * 5))) & 15;
      const unsigned oob = val ? 0u : 0x40000000u;
      const unsigned so = u16 * PUNIT_BYTES + (c * 4 + w) * PBLK;
      const unsigned lds = __builtin_amdgcn_readfirstlane((unsigned)(unsigned long long)(lu8*)(ring + slot * SLOT));
      unsigned keep;
      asm volatile("s_mov_b32 %0, m0\n\ts_mov_b32 m0, %1\n\ts_nop 0\n\t"
                   "buffer_load_dwordx4 %2, %4, %5 offen sc0 sc1 lds\n\t"
                   "buffer_load_dwordx4 %2, %4, %5 offen offset:1024 sc0 sc1 lds\n\t"
                   "buffer_load_dwordx4 %3, %4, %5 offen offset:2048 sc0 sc1 lds\n\t"
                   "s_mov_b32 m0, %0"
                   : "=&s"(keep) : "s"(lds), "v"(vpay | oob), "v"(vlast | oob), "s"(rss), "s"(so) : "memory");
    };
    auto emit = [&](unsigned slot, unsigned n) {
      const char* sl = ring + slot * SLOT;
      const unsigned h0 = __builtin_amdgcn_readfirstlane(*(const lu32*)(sl + 2816));
      const u32x4 A = *(const lu32x4*)(sl + lane * 16), B = *(const lu32x4*)(sl + 1024 + lane * 16);
      const unsigned S = *(const lu32*)(sl + 2048 + lane * 12), K0 = *(const lu32*)(sl + 2052 + lane * 12), K1 = *(const lu32*)(sl + 2056 + lane * 12);
      const unsigned cq = n / 20, c = C0 + cq, r = n - cq * 20, w = r / 5, u16 = (ue >> (4 * (r - w * 5))) & 15;
      const int kind = u16 >> 3;
      const unsigned dst = (unsigned)stg_off(kind, src, u16 & 7) + (c * 4 + w) * BLK;
      if (h0 & 0x10000u) {
        u32x4 rr[4];
#pragma unroll
        for (int p = 0; p < 4; p++) rr[p] = __builtin_amdgcn_raw_buffer_load_b128(rraw, p * 1024 + lane * 16, dst, AUX_SYS);
#pragma unroll
        for (int p = 0; p < 4; p++) __builtin_amdgcn_raw_buffer_store_b128(rr[p], rd, p * 1024 + lane * 16, dst, AUX_AGENT);
        return;
      }
      const unsigned Md = (h0 & 0x80) ? 0x80808080u : 0u;
      const unsigned TG = (((h0 & 0x7f) * 0x01010101u | 0x80808080u) - 0x03020100u) | 0x80808080u;
      u32x4 o[4];
#pragma unroll
      for (int i = 0; i < 8; i++) {
        const unsigned Lp = i < 4 ? A[i] : B[i - 4];
        const unsigned code = ((i < 4 ? K0 : K1) >> (2 * (i & 3))) & 0x03030303u;
        const unsigned x = perm(TG, TG, code) - ((~Lp & Md) >> 7);
        const unsigned hi = (x & 0x7f7f7f7fu) | ((S << (7 - i)) & 0x80808080u);
        const unsigned L = Lp ^ Md;
        o[i >> 1][2 * (i & 1)] = perm(hi, L, 0x05010400u);
        o[i >> 1][2 * (i & 1) + 1] = perm(hi, L, 0x07030602u);
      }
      const unsigned cnt = (h0 >> 8) & 0xff;
      if (cnt == 0) {
#pragma unroll
        for (int p = 0; p < 4; p++)
          __builtin_amdgcn_raw_buffer_store_b128(o[p], rd, p * 1024 + (kind ? (wperm ^ (p * 16)) : lane * 16), dst, AUX_AGENT);
      } else {
#pragma unroll
        for (int p = 0; p < 4; p++) *(lu32x4*)(scr + p * 1024 + (kind ? (wperm ^ (p * 16)) : lane * 16)) = o[p];
        if (lane < (int)cnt) {
          const unsigned idx = *(const lu16*)(sl + 2816 + 4 + 2 * lane);
          *(lu8*)(scr + 2 * (idx & 0x7ff) + 1) = *(const lu8*)(sl + 2816 + 4 + 2 * NEXC + lane);
        }
#pragma unroll
        for (int p = 0; p < 4; p++) o[p] = *(const lu32x4*)(scr + p * 1024 + lane * 16);
#pragma unroll
        for (int p = 0; p < 4; p++) __builtin_amdgcn_raw_buffer_store_b128(o[p], rd, p * 1024 + lane * 16, dst, AUX_AGENT);
      }
    };
    auto arrive = [&](unsigned n) {
      const unsigned cq = n / 20, c = C0 + cq, r = n - cq * 20, w = r / 5, u16 = (ue >> (4 * (r - w * 5))) & 15;
      const unsigned so = (unsigned)(ARR_OFF - FLAG_OFF) + (src * 16 + u16) * 256 + (c * 2 + (w >> 1)) * 4;
      asm volatile("buffer_atomic_add %0, %1, %2, %3 offen" :: "v"(1u), "v"(l0), "s"(rfs), "s"(so) : "memory");
    };
    unsigned cur = ticket();
    unsigned nxt = cur + 1, prev = NITEM, slot = 0;
    grab();
    if (cur < NITEM) need(cur);
    load(0, cur);
    while (cur < NITEM) {
      if (nxt < NITEM) need(nxt);
      load(slot ^ 1, nxt);
      asm volatile("s_waitcnt vmcnt(3)" ::: "memory");
      if (prev < NITEM) arrive(prev);
      const unsigned nn = ticket();
      grab();
      emit(slot, cur);
      prev = cur; cur = nxt; nxt = nn; slot ^= 1;
    }
    asm volatile("s_waitcnt vmcnt(0)" ::: "memory");
    if (prev < NITEM) arrive(prev);
    return;
  }

  const int t1 = (te >> 14) & 7, i1 = (te >> 11) & 7, t2 = (te >> 8) & 7, i2 = (te >> 5) & 7, tu = (te >> 2) & 7, pj = te & 3;
  const int srcW = rank ^ tu;
  char* const donep = a.ws[srcW & 7] + DONE_OFF + b * 4;
  lu32* sw = (lu32*)(smem + NBUF * STAGE_BYTES);
  if (wv >= 4) {
    const int lw = wv - 4;
    const int kind = lw >> 1;
    const int src = lw == 0 ? rank ^ t1 : (lw == 1 ? rank ^ t2 : srcW);
    const int unit = lw == 0 ? i1 : (lw == 1 ? i2 : 2 * pj + (lw & 1));
    const char* g = me + stg_off(kind, src, unit) + lane * 16;
    const bool loc = src == rank;
    const char* pollp = loc ? me + READY_OFF + (size_t)(((src & 7) * 2 + kind) * 8 + unit) * 128
                            : me + ARR_OFF + (size_t)(((src & 7) * 2 + kind) * 8 + unit) * 256;
    const unsigned pthr = loc ? base + 1 : 2 * call;
    const int nword = loc ? 16 : 64;
    auto eval = [&](int q, unsigned v) -> int {
      const int w0 = loc ? q >> 2 : q;
      const unsigned long long nr = __ballot(lane < 16 && w0 + lane < nword && v < pthr);
      const int f = __builtin_ctz((unsigned)nr | 0x10000u);
      return min(64, loc ? 4 * (w0 + f) : w0 + f);
    };
    auto wordp = [&](int q) { return pollp + min((loc ? q >> 2 : q) + (lane & 15), nword - 1) * 4; };
    int avail = 0, pend = -1;
    int nb = 0;
    for (int s = -PF; s < 64; s++) {
      __builtin_amdgcn_s_barrier();
      asm volatile("" ::: "memory");
      if (s + PF < 64) {
        const int q = s + PF;
        if (q >= avail) {
          for (Spin sp;;) {
            avail = eval(q, ld_sys(wordp(q)));
            if (avail > q) break;
            if (sp.expired(me, call)) { avail = 64; break; }
          }

        }
        const unsigned* gp = (const unsigned*)(g + (size_t)q * (2 * BLK));
        unsigned* lp = (unsigned*)(smem + nb * STAGE_BYTES + lw * (2 * BLK));
        nb = nb == NBUF - 1 ? 0 : nb + 1;
        __builtin_amdgcn_global_load_lds(gp, lp, 16, 0, AUX_SYS);
        __builtin_amdgcn_global_load_lds(gp, lp, 16, 1024, AUX_SYS);
        __builtin_amdgcn_global_load_lds(gp, lp, 16, 2048, AUX_SYS);
        __builtin_amdgcn_global_load_lds(gp, lp, 16, 3072, AUX_SYS);
        __builtin_amdgcn_global_load_lds(gp + 1024, lp + 1024, 16, 0, AUX_SYS);
        __builtin_amdgcn_global_load_lds(gp + 1024, lp + 1024, 16, 1024, AUX_SYS);
        __builtin_amdgcn_global_load_lds(gp + 1024, lp + 1024, 16, 2048, AUX_SYS);
        __builtin_amdgcn_global_load_lds(gp + 1024, lp + 1024, 16, 3072, AUX_SYS);
        if ((q & 1) == 0) {
          if (avail < 64 && avail <= q + 2 * LOOK + 1) {
            asm volatile("global_load_dword v200, %0, off sc0 sc1" :: "v"(wordp(avail)) : "v200", "memory");
            pend = avail;
          }
        } else if (pend >= 0) {
          unsigned pv;
          asm volatile("s_waitcnt vmcnt(8)\n\tv_mov_b32 %0, v200" : "=v"(pv) :: "v200", "memory");
          avail = max(avail, eval(pend, pv));
          pend = -1;
        }
        if (PF == 4) asm volatile("s_waitcnt vmcnt(24)" ::: "memory");
        else if (PF == 3) asm volatile("s_waitcnt vmcnt(16)" ::: "memory");
        else asm volatile("s_waitcnt vmcnt(8)" ::: "memory");
      } else {
        const int rem = 62 - s;
        if (rem >= 2) asm volatile("s_waitcnt vmcnt(16)" ::: "memory");
        else if (rem == 1) asm volatile("s_waitcnt vmcnt(8)" ::: "memory");
        else asm volatile("s_waitcnt vmcnt(0)" ::: "memory");
      }
    }
    if (lw == 0 && lane == 0) sw[4] = 0;
    asm volatile("s_waitcnt lgkmcnt(0)" ::: "memory");
    __builtin_amdgcn_s_barrier();
    if (lw == 0) {
      const char* rp = me + RES_OFF + (b & 7) * 128;
      for (Spin sp;;) {
        u32x2 r;
        unsigned tm;
        asm volatile("global_load_dwordx2 %0, %2, off sc1\n\tglobal_load_dword %1, %3, off sc1\n\ts_waitcnt vmcnt(0)"
                     : "=&v"(r), "=&v"(tm) : "v"(rp), "v"(me + TMO_OFF) : "memory");
        const bool fin = __builtin_amdgcn_readfirstlane(r[1]) == call;
        bool bad = __builtin_amdgcn_readfirstlane(tm) == call || (fin && __builtin_amdgcn_readfirstlane(r[0]) == 0x7fffffffu);
        if (!fin && !bad && sp.expired(me, call)) bad = true;
        if (fin || bad) {
          if (lane == 0) {
            sw[5] = r[0];
            sw[6] = bad;
            __hip_atomic_store(sw + 4, 1u, __ATOMIC_RELEASE, __HIP_MEMORY_SCOPE_WORKGROUP);
          }
          break;
        }
      }
    }
    return;
  }
  const int xu = wv >> 1, wu = wv & 1;
  f32x4 acc[4][4];
#pragma unroll
  for (int f = 0; f < 4; f++)
#pragma unroll
    for (int j = 0; j < 4; j++) acc[f][j] = f32x4{0.f, 0.f, 0.f, 0.f};
  const unsigned xoff = xu * (2 * BLK) + lane * 16;
  unsigned woff[4];
  {
    const int r = lane & 3, q = (lane >> 2) & 3, kq = lane >> 4;
#pragma unroll
    for (int j = 0; j < 4; j++) woff[j] = (2 + wu) * (2 * BLK) + r * 1024 + kq * 256 + q * 64 + ((j ^ r) * 16);
  }
  {
    bf16x8 xa[4], wa[4], xb[4], wb[4];
#pragma unroll
    for (int f = 0; f < 4; f++) { xb[f] = bf16x8{}; wb[f] = bf16x8{}; }
    for (int s = -PF; s < 0; s++) __builtin_amdgcn_s_barrier();
    int cb = 0;
#define ILV() do { _Pragma("unroll") for (int q_ = 0; q_ < 8; q_++) { __builtin_amdgcn_sched_group_barrier(0x008, 1, 0); \
    __builtin_amdgcn_sched_group_barrier(0x100, 1, 0); } __builtin_amdgcn_sched_group_barrier(0x008, 8, 0); } while (0)
    for (int s = 0; s < 64; s++) {
      asm volatile("s_waitcnt lgkmcnt(0)" ::: "memory");
      __builtin_amdgcn_s_barrier();
      asm volatile("" ::: "memory");

      const char* sb = smem + cb * STAGE_BYTES;
      cb = cb == NBUF - 1 ? 0 : cb + 1;
      __builtin_amdgcn_sched_barrier(0);
      wa[0] = *(const bf16x8*)(sb + woff[0]); xa[0] = *(const bf16x8*)(sb + xoff);
#pragma unroll
      for (int f = 1; f < 4; f++) wa[f] = *(const bf16x8*)(sb + woff[f]);
#pragma unroll
      for (int f = 1; f < 4; f++) xa[f] = *(const bf16x8*)(sb + xoff + f * 1024);
#pragma unroll
      for (int f = 0; f < 4; f++)
#pragma unroll
        for (int j = 0; j < 4; j++) acc[f][j] = __builtin_amdgcn_mfma_f32_16x16x32_bf16(wb[j], xb[f], acc[f][j], 0, 0, 0);
      ILV();
      __builtin_amdgcn_sched_barrier(0);
      wb[0] = *(const bf16x8*)(sb + woff[0] + BLK); xb[0] = *(const bf16x8*)(sb + xoff + BLK);
#pragma unroll
      for (int f = 1; f < 4; f++) wb[f] = *(const bf16x8*)(sb + woff[f] + BLK);
#pragma unroll
      for (int f = 1; f < 4; f++) xb[f] = *(const bf16x8*)(sb + xoff + BLK + f * 1024);
#pragma unroll
      for (int f = 0; f < 4; f++)
#pragma unroll
        for (int j = 0; j < 4; j++) acc[f][j] = __builtin_amdgcn_mfma_f32_16x16x32_bf16(wa[j], xa[f], acc[f][j], 0, 0, 0);
      ILV();
      __builtin_amdgcn_sched_barrier(0);
    }
#pragma unroll
    for (int f = 0; f < 4; f++)
#pragma unroll
      for (int j = 0; j < 4; j++) acc[f][j] = __builtin_amdgcn_mfma_f32_16x16x32_bf16(wb[j], xb[f], acc[f][j], 0, 0, 0);
  }
  float mf = 0.f;
#pragma unroll
  for (int f = 0; f < 4; f++)
#pragma unroll
    for (int j = 0; j < 4; j++) {
      const f32x4 t = acc[f][j];
      mf = fmaxf(fmaxf(mf, fabsf(t[0])), fabsf(t[1]));
      mf = fmaxf(fmaxf(mf, fabsf(t[2])), fabsf(t[3]));
    }
  unsigned m = wave_max_u32(__float_as_uint(mf));
  if (lane == 0) sw[wv] = m;
  asm volatile("s_waitcnt lgkmcnt(0)" ::: "memory");
  __builtin_amdgcn_s_barrier();
  asm volatile("" ::: "memory");
  m = max(max(sw[0], sw[1]), max(sw[2], sw[3]));
  if (tid < NR) {
    gu64* g = (gu64*)(wsl + AMAX_OFF + (size_t)(rank * NGEMM + b) * 8);
    __hip_atomic_store(g, ((unsigned long long)call << 32) | m, __ATOMIC_RELAXED, __HIP_MEMORY_SCOPE_SYSTEM);
  }
  while (__builtin_amdgcn_readfirstlane(__hip_atomic_load(sw + 4, __ATOMIC_ACQUIRE, __HIP_MEMORY_SCOPE_WORKGROUP)) == 0)
    __builtin_amdgcn_s_sleep(1);
  const float inv = 448.0f / __uint_as_float(__builtin_amdgcn_readfirstlane(sw[5]));
  const bool bad = __builtin_amdgcn_readfirstlane(sw[6]) != 0;
  {
    const int tx = xu == 0 ? t1 : t2, ix = xu == 0 ? i1 : i2;
    unsigned char* optr = outp + (size_t)((rank ^ tx) * SH + ix * 64 + (lane & 15)) * SH + pj * 128 + wu * 64 + (lane >> 4) * 16;
#pragma unroll
    for (int f = 0; f < 4; f++) {
      u32x4 q;
#pragma unroll
      for (int j = 0; j < 4; j++) {
        int w = __builtin_amdgcn_cvt_pk_fp8_f32(acc[f][j][0] * inv, acc[f][j][1] * inv, 0, false);
        w = __builtin_amdgcn_cvt_pk_fp8_f32(acc[f][j][2] * inv, acc[f][j][3] * inv, w, true);
        q[j] = bad ? 0x7f7f7f7fu : (unsigned)w;
      }
      if (tu != 0) asm volatile("global_store_dwordx4 %0, %1, off sc0 sc1\n\ts_nop 1" ::"v"(optr + (size_t)f * 16 * SH), "v"(q) : "memory");
      else *(u32x4*)(optr + (size_t)f * 16 * SH) = q;
    }
  }
  if (tu != 0) {
    asm volatile("s_waitcnt vmcnt(0)" ::: "memory");
    __builtin_amdgcn_s_barrier();
    if (tid == 0) st_sys(donep, call);
  }
}
}

extern "C" void kernel_launch(int rank, int nranks, ncclComm_t comm, hipStream_t stream, void* const* d_in,
                              const int* in_sizes, int n_in, void* d_out, int out_size, const int* in_dtypes,
                              int out_dtype, void* const* d_out_all, void* const* d_ws_all, size_t ws_bytes) {
  (void)comm; (void)in_sizes; (void)n_in; (void)d_out; (void)out_size; (void)in_dtypes; (void)out_dtype;
  if (nranks != NR || rank < 0 || rank >= NR || ws_bytes < WS_NEED) {
    fprintf(stderr, "unsupported configuration\n");
    return;
  }
  Args a;
  a.x = (const float*)d_in[0];
  a.w = (const float*)d_in[1];
  for (int p = 0; p < NR; p++) {
    a.ws[p] = (char*)d_ws_all[p];
    a.out[p] = (unsigned char*)d_out_all[p];
  }
  a.rank = rank;
  a.pad = 0;
  fused_ag_gemm<<<NWG, 512, 0, stream>>>(a);
  hipError_t e = hipGetLastError();
  if (e != hipSuccess) fprintf(stderr, "launch error %s\n", hipGetErrorString(e));
}
```

```cpp
#include <hip/hip_runtime.h>
#include <rccl/rccl.h>
#include <cstdio>
#include <cstdint>

typedef unsigned u32x4 __attribute__((ext_vector_type(4)));
typedef unsigned u32x3 __attribute__((ext_vector_type(3)));
typedef unsigned u32x2 __attribute__((ext_vector_type(2)));
typedef float f32x4 __attribute__((ext_vector_type(4)));
typedef __bf16 bf16x8 __attribute__((ext_vector_type(8)));
typedef __bf16 bf16x2 __attribute__((ext_vector_type(2)));
typedef float f32x2 __attribute__((ext_vector_type(2)));

#define NPW 2
#define PPL 13
#define NPWN 8
#define NPUSH 1
#define LOOK 1
#define STAG0 60
#define STAGGER 20

namespace {
constexpr int NR = 8;
constexpr int KDIM = 4096;
constexpr int SH = 512;
constexpr unsigned BLK = 4096;
constexpr unsigned UNIT_BYTES = 128 * BLK;
constexpr unsigned PBLK = 2944;
#define PPAD 0
constexpr unsigned PUNIT_BYTES = 128 * PBLK + PPAD;
constexpr unsigned NEXC = 41;
#define UPAD 4352
constexpr unsigned UNIT_STRIDE = UNIT_BYTES + UPAD;
constexpr size_t STG_KIND_BYTES = (size_t)NR * 8 * UNIT_STRIDE;
constexpr size_t EXP_OFF = 2 * STG_KIND_BYTES;
constexpr size_t FLAG_OFF = EXP_OFF + (size_t)16 * PUNIT_BYTES;
constexpr size_t READY_OFF = FLAG_OFF;
constexpr size_t ARR_OFF = READY_OFF + NR * 16 * 128;
constexpr size_t AMAX_OFF = ARR_OFF + NR * 16 * 256;
constexpr size_t DONE_OFF = AMAX_OFF + NR * 128 * 8;
constexpr size_t CTR_OFF = DONE_OFF + 1024;
constexpr size_t TMO_OFF = CTR_OFF + 128;
constexpr size_t QCTR_OFF = TMO_OFF + 128;
constexpr size_t RES_OFF = QCTR_OFF + 16 * 128;
constexpr size_t END_OFF = RES_OFF + 8 * 128;
constexpr size_t WS_NEED = END_OFF;
static_assert(FLAG_OFF % 128 == 0, "align");
constexpr int NGEMM = 128, NPULL = 7 * PPL, NWG = 256;
#define DEADLINE_TICKS 180000000ull
constexpr unsigned long long DEADLINE = DEADLINE_TICKS;
constexpr unsigned STAGE_BYTES = 8 * BLK;
#define NBUFS 4
constexpr int NBUF = NBUFS;
constexpr int PF = NBUF - 1;

__constant__ unsigned TILE_TAB[NGEMM] = {16420,2145,5154,5283,14530,10241,8928,10432,69476,67649,7906,47971,70370,15553,17664,15424,3116,74817,8450,78019,77058,16577,113792,55360,18284,75905,12130,83011,3238,113377,121888,75872,70924,82785,73762,70820,13574,1059,3115,77856,77964,5158,78690,82660,69798,18147,7435,4311,8207,55430,3939,112708,78566,76,47243,47287,121935,75622,6307,75,112902,11404,113803,55351,1081,3112,73859,11115,10991,13484,4373,69751,9433,77064,10340,75979,12335,48876,9973,12377,12153,121928,12420,78923,76015,9271,67637,47225,17561,5167,74948,55518,79599,74935,73845,75833,83033,75023,6216,70846,11477,78103,77685,2234,2238,2173,12008,78590,69845,12381,47258,55386,8478,70813,73864,81950,9501,48349,68890,69850,68766,76669,78024,112734,10493,69373,75642,79034};
__constant__ unsigned PULL_TAB[8] = {0, 0xdc980, 0xfe987, 0xfe983, 0x65421, 0xfeba0, 0xdcba7, 0xdcba3};
constexpr unsigned long long DEST_MASK_X = 0x4511111189111123ull, DEST_MASK_W = 0x2d2dc3c3e1e10f0full;

struct Args {
  const float* x;
  const float* w;
  char* ws[NR];
  unsigned char* out[NR];
  int rank;
  int pad;
};

typedef __attribute__((address_space(1))) unsigned gu32;
typedef __attribute__((address_space(1))) unsigned long long gu64;
typedef __attribute__((address_space(3))) unsigned lu32;
typedef __attribute__((address_space(3))) unsigned char lu8;
typedef __attribute__((address_space(3))) unsigned short lu16;
typedef __attribute__((address_space(3))) u32x4 lu32x4;

__device__ __forceinline__ unsigned ld_sys(const void* p) { return __hip_atomic_load((const gu32*)p, __ATOMIC_RELAXED, __HIP_MEMORY_SCOPE_SYSTEM); }
__device__ __forceinline__ void st_sys(void* p, unsigned v) { __hip_atomic_store((gu32*)p, v, __ATOMIC_RELAXED, __HIP_MEMORY_SCOPE_SYSTEM); }
__device__ __forceinline__ unsigned ld_agent(const void* p) { return __hip_atomic_load((const gu32*)p, __ATOMIC_RELAXED, __HIP_MEMORY_SCOPE_AGENT); }
__device__ __forceinline__ void st_agent(void* p, unsigned v) { __hip_atomic_store((gu32*)p, v, __ATOMIC_RELAXED, __HIP_MEMORY_SCOPE_AGENT); }

__device__ __forceinline__ __amdgpu_buffer_rsrc_t mkrsrc(const void* p, unsigned bytes) {
  unsigned long long a = (unsigned long long)p;
  unsigned lo = __builtin_amdgcn_readfirstlane((unsigned)a), hi = __builtin_amdgcn_readfirstlane((unsigned)(a >> 32));
  void* q = (void*)(((unsigned long long)hi << 32) | lo);
  return __builtin_amdgcn_make_buffer_rsrc(q, 0, bytes, 0x00020000);
}
__device__ __forceinline__ u32x4 mkrsrc_words(const void* p, unsigned bytes) {
  unsigned long long a = (unsigned long long)p;
  return u32x4{(unsigned)__builtin_amdgcn_readfirstlane((unsigned)a), (unsigned)__builtin_amdgcn_readfirstlane((unsigned)(a >> 32)) & 0xffffu,
               bytes, 0x00020000u};
}
__device__ __forceinline__ size_t stg_off(int kind, int src, int unit) {
  return (size_t)kind * STG_KIND_BYTES + (size_t)(src * 8 + unit) * UNIT_STRIDE;
}
__device__ __forceinline__ unsigned cvt2(float a, float b) {
  f32x2 v = {a, b};
  bf16x2 r = __builtin_convertvector(v, bf16x2);
  return __builtin_bit_cast(unsigned, r);
}
__device__ __forceinline__ unsigned perm(unsigned hi, unsigned lo, unsigned sel) { return __builtin_amdgcn_perm(hi, lo, sel); }
__device__ __forceinline__ unsigned wave_max_u32(unsigned v) {
  v = max(v, (unsigned)__builtin_amdgcn_update_dpp(0, (int)v, 0x111, 0xf, 0xf, false));
  v = max(v, (unsigned)__builtin_amdgcn_update_dpp(0, (int)v, 0x112, 0xf, 0xf, false));
  v = max(v, (unsigned)__builtin_amdgcn_update_dpp(0, (int)v, 0x114, 0xf, 0xf, false));
  v = max(v, (unsigned)__builtin_amdgcn_update_dpp(0, (int)v, 0x118, 0xf, 0xf, false));
  v = max(v, (unsigned)__builtin_amdgcn_update_dpp(0, (int)v, 0x142, 0xa, 0xf, false));
  v = max(v, (unsigned)__builtin_amdgcn_update_dpp(0, (int)v, 0x143, 0xc, 0xf, false));
  return (unsigned)__builtin_amdgcn_readlane((int)v, 63);
}

struct Spin {
  unsigned it = 0;
  unsigned long long t0 = 0;
  __device__ __forceinline__ bool expired(char* me, unsigned call) {
    if (it == 0) t0 = __builtin_amdgcn_s_memrealtime();
    if ((++it & 15) == 0) {
      if (ld_agent(me + TMO_OFF) == call || __builtin_amdgcn_s_memrealtime() - t0 > DEADLINE) {
        st_agent(me + TMO_OFF, call);
        return true;
      }
    }
    __builtin_amdgcn_s_sleep(1);
    return false;
  }
};
constexpr int AUX_SYS = 17, AUX_AGENT = 16;

__device__ __forceinline__ unsigned piece_off(int kind, int p, int lane) {
  const int kq = lane >> 4, q = (lane >> 2) & 3, j = lane & 3;
  return kind == 0 ? p * 1024 + lane * 16 : p * 1024 + kq * 256 + q * 64 + ((j ^ p) * 16);
}

__global__ __launch_bounds__(512, 2) void fused_ag_gemm(Args a) {
  __shared__ __attribute__((aligned(16))) char smem[NBUF * STAGE_BYTES + 128];
  const int tid = threadIdx.x, lane = tid & 63;
  const int wv = __builtin_amdgcn_readfirstlane(tid >> 6);
  const int b = blockIdx.x;
  constexpr int NH0 = 16 * NPUSH;
  const int task = b < NGEMM ? b + NH0 + NPULL
                             : (b < NGEMM + NPULL ? b - NGEMM + NH0 : (b - (NGEMM + NPULL) < NH0 ? b - (NGEMM + NPULL) : b));
  const int c16 = task >> 4, u16 = task & 15, kind = u16 >> 3, unit = u16 & 7;
  const int ks = c16 * 8 + wv;
  const int kq = lane >> 4, l15 = lane & 15;
  if (c16 > NPUSH) {
    const unsigned long long tstart = __builtin_amdgcn_s_memrealtime();
    while (__builtin_amdgcn_s_memrealtime() - tstart < (unsigned long long)(STAG0 + (c16 - NPUSH - 1) * STAGGER)) __builtin_amdgcn_s_sleep(1);
  }
  f32x4 v[8];
  {
    const float* px = a.x + (size_t)(unit * 64 + l15) * KDIM + ks * 32 + 4 * kq;
    const float* pw = a.w + (size_t)(ks * 32 + 4 * kq) * SH + unit * 64 + 4 * l15;
    const float* p = kind == 0 ? px : pw;
#pragma unroll
    for (int i = 0; i < 8; i++) {
      const unsigned ox = (i >> 1) * 16 * KDIM + (i & 1) * 16, ow = ((i & 3) + (i >> 2) * 16) * SH;
      v[i] = *(const f32x4*)(p + (kind == 0 ? ox : ow));
    }
  }
  asm volatile("" ::: "memory");
  const int rank = a.rank;
  char* me = a.ws[rank];
  const unsigned ctr_v = ld_agent(me + CTR_OFF);
  const unsigned te = TILE_TAB[b & (NGEMM - 1)];
  const bool is_pull = b >= NGEMM && b < NGEMM + NPULL;
  const int plink = is_pull ? 1 + (b - NGEMM) / PPL : 0;
  const int npw = is_pull && (b - NGEMM) % PPL >= NPWN ? NPW - 1 : NPW;
  const unsigned ue = PULL_TAB[plink];
  char* const wsl = a.ws[lane & 7];
  unsigned char* const outp = a.out[(rank ^ (int)(te >> 2)) & 7];
  const u32x4 rfs = mkrsrc_words(me + FLAG_OFF, (unsigned)(END_OFF - FLAG_OFF));
  const unsigned l0 = lane == 0 ? 0u : 0x40000000u;
  unsigned call, base;
  {
    const __amdgpu_buffer_rsrc_t rd = mkrsrc(me + stg_off(kind, rank, unit), UNIT_BYTES);
    call = __builtin_amdgcn_readfirstlane(ctr_v) + 1;
    base = call * 64;
    if (is_pull && wv < npw) {
      const unsigned qoff = (unsigned)(QCTR_OFF - FLAG_OFF) + ((call & 1) * 8 + plink) * 128;
      asm volatile("v_mov_b32 v200, 2\n\ts_nop 4\n\tbuffer_atomic_add v200, %0, %1, %2 offen sc0"
                   :: "v"(l0), "s"(rfs), "s"(qoff) : "v200", "memory");
    }
    u32x4 o[4];
    if (kind == 0) {
#pragma unroll
      for (int f = 0; f < 4; f++)
        o[f] = u32x4{cvt2(v[2 * f][0], v[2 * f][1]), cvt2(v[2 * f][2], v[2 * f][3]),
                     cvt2(v[2 * f + 1][0], v[2 * f + 1][1]), cvt2(v[2 * f + 1][2], v[2 * f + 1][3])};
    } else {
#pragma unroll
      for (int r = 0; r < 4; r++)
        o[r] = u32x4{cvt2(v[0][r], v[1][r]), cvt2(v[2][r], v[3][r]), cvt2(v[4][r], v[5][r]), cvt2(v[6][r], v[7][r])};
    }
    const unsigned dmask = (unsigned)((kind ? DEST_MASK_W : DEST_MASK_X) >> (8 * unit)) & 0xff;
    if (c16 < NPUSH) {
#pragma unroll
      for (int p = 1; p < NR; p++)
        if ((dmask >> p) & 1) {
          const __amdgpu_buffer_rsrc_t rq = mkrsrc(a.ws[rank ^ p] + stg_off(kind, rank, unit), UNIT_BYTES);
#pragma unroll
          for (int i = 0; i < 4; i++)
            __builtin_amdgcn_raw_buffer_store_b128(o[i], rq, ks * BLK + piece_off(kind, i, lane), 0, AUX_SYS);
        }
    } else {
      float mx = 0.f;
#pragma unroll
      for (int i = 0; i < 8; i++) {
        mx = fmaxf(mx, fmaxf(fabsf(v[i][0]), fabsf(v[i][1])));
        mx = fmaxf(mx, fmaxf(fabsf(v[i][2]), fabsf(v[i][3])));
      }
      const unsigned mxb = wave_max_u32(__float_as_uint(mx));
      const unsigned emax = (cvt2(__uint_as_float(mxb), 0.f) >> 7) & 0xff;
      const float thr1 = __uint_as_float(emax << 23), thr2 = thr1 * 0.5f;
      unsigned n1 = 0, n2 = 0;
#pragma unroll
      for (int i = 0; i < 8; i++) {
        n1 += __builtin_popcountll(__ballot(fabsf(v[i][0]) >= thr1)) + __builtin_popcountll(__ballot(fabsf(v[i][2]) >= thr1));
        n2 += __builtin_popcountll(__ballot(fabsf(v[i][0]) >= thr2)) + __builtin_popcountll(__ballot(fabsf(v[i][2]) >= thr2));
      }
      const int top = (int)emax - (n1 <= 10 ? 1 : 0) - (n2 <= 10 ? 1 : 0);
      const unsigned delta = (top & 1) ^ 1, tp = ((top + (int)delta) >> 1) & 0x7f;
      const unsigned Md = delta ? 0x80808080u : 0u;
      const unsigned b80 = tp * 0x01010101u | 0x80808080u;
      const unsigned T = (b80 - 0x03020100u) & 0x7f7f7f7fu;
      unsigned LP[8], HH[8], S = 0, K0 = 0, K1 = 0, M = 0;
#pragma unroll
      for (int i = 0; i < 8; i++) {
        const unsigned d0 = o[i >> 1][2 * (i & 1)], d1 = o[i >> 1][2 * (i & 1) + 1];
        const unsigned L = perm(d1, d0, 0x06040200u), H = perm(d1, d0, 0x07050301u);
        const unsigned pair = ((H & 0x7f7f7f7fu) + ((L & Md) >> 7)) & 0x7f7f7f7fu;
        const unsigned code = (b80 - pair) & 0x03030303u;
        const unsigned mis = perm(T, T, code) ^ pair;
        LP[i] = L ^ Md;
        HH[i] = H;
        if (i < 4) K0 |= code << (2 * i); else K1 |= code << (2 * (i - 4));
        S |= (H & 0x80808080u) >> (7 - i);
        M |= __builtin_amdgcn_udot4(((mis + 0x7f7f7f7fu) & 0x80808080u) >> 7, 0x08040201u, 0u, false) << (4 * i);
      }
      char* hsc = smem + wv * 2304;
      lu32* hdr = (lu32*)(hsc + 2048);
      const unsigned nl = __builtin_popcount(M);
      unsigned inc = nl;
      inc += __builtin_amdgcn_update_dpp(0, (int)inc, 0x111, 0xf, 0xf, false);
      inc += __builtin_amdgcn_update_dpp(0, (int)inc, 0x112, 0xf, 0xf, false);
      inc += __builtin_amdgcn_update_dpp(0, (int)inc, 0x114, 0xf, 0xf, false);
      inc += __builtin_amdgcn_update_dpp(0, (int)inc, 0x118, 0xf, 0xf, false);
      inc += __builtin_amdgcn_update_dpp(0, (int)inc, 0x142, 0xa, 0xf, false);
      inc += __builtin_amdgcn_update_dpp(0, (int)inc, 0x143, 0xc, 0xf, false);
      const unsigned cnt = (unsigned)__builtin_amdgcn_readlane((int)inc, 63);
      if (cnt > 0 && cnt <= NEXC) {
        *(lu32x4*)(hsc + lane * 32) = u32x4{HH[0], HH[1], HH[2], HH[3]};
        *(lu32x4*)(hsc + lane * 32 + 16) = u32x4{HH[4], HH[5], HH[6], HH[7]};
        unsigned at = inc - nl;
        while (__any(M != 0)) {
          if (M != 0) {
            const unsigned jv = __builtin_ctz(M);
            M &= M - 1;
            const unsigned char hi = *(const lu8*)(hsc + lane * 32 + jv);
            const unsigned idx = (piece_off(kind, jv >> 3, lane) >> 1) + (jv & 7);
            *(lu16*)((char*)hdr + 4 + 2 * at) = (unsigned short)idx;
            *(lu8*)((char*)hdr + 4 + 2 * NEXC + at) = hi;
            at++;
          }
        }
      }
      const __amdgpu_buffer_rsrc_t rp = mkrsrc(me + EXP_OFF + (size_t)u16 * PUNIT_BYTES, PUNIT_BYTES);
      const unsigned pc = ks * PBLK;
      __builtin_amdgcn_raw_buffer_store_b128(u32x4{LP[0], LP[1], LP[2], LP[3]}, rp, pc + lane * 16, 0, AUX_SYS);
      __builtin_amdgcn_raw_buffer_store_b128(u32x4{LP[4], LP[5], LP[6], LP[7]}, rp, pc + 1024 + lane * 16, 0, AUX_SYS);
      __builtin_amdgcn_raw_buffer_store_b96(u32x3{S, K0, K1}, rp, pc + 2048 + lane * 12, 0, AUX_SYS);
      unsigned hv = hdr[lane & 31];
      if (lane == 0) hv = tp | (delta << 7) | (cnt > NEXC ? 0x10000u : cnt << 8);
      if (lane < 32) __builtin_amdgcn_raw_buffer_store_b32(hv, rp, pc + 2816 + lane * 4, 0, AUX_SYS);
    }
#pragma unroll
    for (int p = 0; p < 4; p++)
      __builtin_amdgcn_raw_buffer_store_b128(o[p], rd, ks * BLK + piece_off(kind, p, lane), 0, AUX_SYS);
    asm volatile("s_waitcnt vmcnt(0)" ::: "memory");
    const int rel = rank ^ lane;
    if (c16 < NPUSH) {
      if (lane < NR && rel != 0 && ((dmask >> rel) & 1))
        __hip_atomic_fetch_add((gu32*)(wsl + ARR_OFF + (size_t)(rank * 16 + u16) * 256 + (ks >> 1) * 4), 1u, __ATOMIC_RELAXED,
                               __HIP_MEMORY_SCOPE_SYSTEM);
    }
    __builtin_amdgcn_s_barrier();
    if (wv == 0 && lane < NR && ((dmask >> rel) & 1) && (rel == 0 || c16 >= NPUSH))
      st_sys(wsl + READY_OFF + (size_t)(rank * 16 + u16) * 128 + c16 * 4, base + 1);
  }

  if (b >= NGEMM + NPULL) {
    if (b == NWG - 1) {
      {
        lu32* sv = (lu32*)smem;
        __builtin_amdgcn_s_barrier();
        if (tid < 3) sv[tid] = 0;
        asm volatile("s_waitcnt lgkmcnt(0)" ::: "memory");
        __builtin_amdgcn_s_barrier();
        const char* gp = me + AMAX_OFF + (size_t)tid * 16;
        bool have = false, bad = false;
        unsigned gm = 0;
        for (Spin sp;;) {
          unsigned tm = 0;
          if (!have) {
            u32x4 g0;
            asm volatile("global_load_dwordx4 %0, %2, off sc0 sc1\n\tglobal_load_dword %1, %3, off sc1\n\ts_waitcnt vmcnt(0)"
                         : "=&v"(g0), "=&v"(tm) : "v"(gp), "v"(me + TMO_OFF) : "memory");
            have = g0[1] >= call && g0[3] >= call;
            gm = max(g0[0], g0[2]);
          }
          if (__any(tm == call)) { bad = true; break; }
          if (__all(have)) break;
          if (sp.expired(me, call)) { bad = true; break; }
        }
        const unsigned wm = wave_max_u32(gm);
        unsigned last = 0;
        if (lane == 0) {
          __hip_atomic_fetch_max(sv, wm, __ATOMIC_RELAXED, __HIP_MEMORY_SCOPE_WORKGROUP);
          if (bad) __hip_atomic_store(sv + 2, 1u, __ATOMIC_RELAXED, __HIP_MEMORY_SCOPE_WORKGROUP);
          last = __hip_atomic_fetch_add(sv + 1, 1u, __ATOMIC_ACQ_REL, __HIP_MEMORY_SCOPE_WORKGROUP);
        }
        if (__builtin_amdgcn_readfirstlane(last) == 7) {
          const unsigned am = sv[2] ? 0x7fffffffu : sv[0];
          if (lane < 8) __hip_atomic_store((gu64*)(me + RES_OFF + lane * 128), ((unsigned long long)call << 32) | am,
                                           __ATOMIC_RELAXED, __HIP_MEMORY_SCOPE_AGENT);
        }
      }
      if (wv == 0) {
        bool ok = true;
        {
          const bool a0 = ((TILE_TAB[2 * lane] >> 2) & 7) != 0, a1 = ((TILE_TAB[2 * lane + 1] >> 2) & 7) != 0;
          for (Spin sp;;) {
            u32x2 d;
            asm volatile("global_load_dwordx2 %0, %1, off sc0 sc1\n\ts_waitcnt vmcnt(0)" : "=v"(d) : "v"(me + DONE_OFF + lane * 8) : "memory");
            if (__all((!a0 || d[0] >= call) && (!a1 || d[1] >= call))) break;
            if (sp.expired(me, call)) { ok = false; break; }
          }
        }
        if (lane < 8) st_agent(me + QCTR_OFF + (size_t)(((call + 1) & 1) * 8 + lane) * 128, 0u);
        if (lane == 0) {
          if (!ok) a.out[rank][(size_t)(rank * SH + 64) * SH + 128] = 0x7f;
          st_agent(me + CTR_OFF, call);
        }
      }
    }
    return;
  }
  if (is_pull) {
    if (wv >= npw) return;
    const unsigned src = (unsigned)(rank ^ plink);
    if (src >= NR) return;
    const __amdgpu_buffer_rsrc_t rraw = mkrsrc(a.ws[src], 2 * STG_KIND_BYTES);
    const __amdgpu_buffer_rsrc_t rd = mkrsrc(me, 2 * STG_KIND_BYTES);
    const u32x4 rss = mkrsrc_words(a.ws[src] + EXP_OFF, 16 * PUNIT_BYTES);
    const unsigned wperm = piece_off(1, 0, lane);
    char* scr = smem + 1024 + wv * BLK;
    constexpr unsigned SLOT = 3072;
    char* ring = smem + 1024 + NPW * BLK + wv * 2 * SLOT;
    unsigned long long rdy03 = 0;
    unsigned rdy4 = 0;
    bool ok = true;
    constexpr unsigned C0 = 2 * NPUSH, NITEM = (32 - C0) * 20;
    const unsigned qoff = (unsigned)(QCTR_OFF - FLAG_OFF) + ((call & 1) * 8 + plink) * 128;
    auto grab = [&]() {
      asm volatile("v_mov_b32 v200, 1\n\ts_nop 4\n\tbuffer_atomic_add v200, %0, %1, %2 offen sc0"
                   :: "v"(l0), "s"(rfs), "s"(qoff) : "v200", "memory");
    };
    auto ticket = [&]() -> unsigned {
      unsigned v;
      asm volatile("v_mov_b32 %0, v200" : "=v"(v) :: "v200", "memory");
      return __builtin_amdgcn_readfirstlane(v);
    };
    auto need = [&](unsigned n) {
      const unsigned cq = n / 20, c = C0 + cq, ui = (n - cq * 20) % 5, c2 = c >> 1;
      for (Spin sp;;) {
        const unsigned m = ui == 4 ? rdy4 : (unsigned)(rdy03 >> (16 * ui));
        if (((m >> c2) & 1) || !ok) return;
        const unsigned u16 = (ue >> (4 * ui)) & 15;
        const unsigned v = lane < 16 ? ld_sys(me + READY_OFF + (size_t)(src * 16 + u16) * 128 + lane * 4) : 0u;
        const unsigned mk = (unsigned)__ballot(v >= base + 1) & 0xffffu;
        if (ui == 4) rdy4 = mk; else rdy03 |= (unsigned long long)mk << (16 * ui);
        if ((mk >> c2) & 1) return;
        if (sp.expired(me, call)) ok = false;
      }
    };
    const unsigned vpay = lane * 16, vlast = lane < 56 ? lane * 16 : 0x40000000u;
    auto load = [&](unsigned slot, unsigned n) {
      const bool val = n < NITEM;
      const unsigned nn = val ? n : 0;
      const unsigned cq = nn / 20, c = C0 + cq, r = nn - cq * 20, w = r / 5, u16 = (ue >> (4 * (r - w * 5))) & 15;
      const unsigned oob = val ? 0u : 0x40000000u;
      const unsigned so = u16 * PUNIT_BYTES + (c * 4 + w) * PBLK;
      const unsigned lds = __builtin_amdgcn_readfirstlane((unsigned)(unsigned long long)(lu8*)(ring + slot * SLOT));
      unsigned keep;
      asm volatile("s_mov_b32 %0, m0\n\ts_mov_b32 m0, %1\n\ts_nop 0\n\t"
                   "buffer_load_dwordx4 %2, %4, %5 offen sc0 sc1 lds\n\t"
                   "buffer_load_dwordx4 %2, %4, %5 offen offset:1024 sc0 sc1 lds\n\t"
                   "buffer_load_dwordx4 %3, %4, %5 offen offset:2048 sc0 sc1 lds\n\t"
                   "s_mov_b32 m0, %0"
                   : "=&s"(keep) : "s"(lds), "v"(vpay | oob), "v"(vlast | oob), "s"(rss), "s"(so) : "memory");
    };
    auto emit = [&](unsigned slot, unsigned n) {
      const char* sl = ring + slot * SLOT;
      const unsigned h0 = __builtin_amdgcn_readfirstlane(*(const lu32*)(sl + 2816));
      const u32x4 A = *(const lu32x4*)(sl + lane * 16), B = *(const lu32x4*)(sl + 1024 + lane * 16);
      const unsigned S = *(const lu32*)(sl + 2048 + lane * 12), K0 = *(const lu32*)(sl + 2052 + lane * 12), K1 = *(const lu32*)(sl + 2056 + lane * 12);
      const unsigned cq = n / 20, c = C0 + cq, r = n - cq * 20, w = r / 5, u16 = (ue >> (4 * (r - w * 5))) & 15;
      const int kind = u16 >> 3;
      const unsigned dst = (unsigned)stg_off(kind, src, u16 & 7) + (c * 4 + w) * BLK;
      if (h0 & 0x10000u) {
        u32x4 rr[4];
#pragma unroll
        for (int p = 0; p < 4; p++) rr[p] = __builtin_amdgcn_raw_buffer_load_b128(rraw, p * 1024 + lane * 16, dst, AUX_SYS);
#pragma unroll
        for (int p = 0; p < 4; p++) __builtin_amdgcn_raw_buffer_store_b128(rr[p], rd, p * 1024 + lane * 16, dst, AUX_AGENT);
        return;
      }
      const unsigned Md = (h0 & 0x80) ? 0x80808080u : 0u;
      const unsigned TG = (((h0 & 0x7f) * 0x01010101u | 0x80808080u) - 0x03020100u) | 0x80808080u;
      u32x4 o[4];
#pragma unroll
      for (int i = 0; i < 8; i++) {
        const unsigned Lp = i < 4 ? A[i] : B[i - 4];
        const unsigned code = ((i < 4 ? K0 : K1) >> (2 * (i & 3))) & 0x03030303u;
        const unsigned x = perm(TG, TG, code) - ((~Lp & Md) >> 7);
        const unsigned hi = (x & 0x7f7f7f7fu) | ((S << (7 - i)) & 0x80808080u);
        const unsigned L = Lp ^ Md;
        o[i >> 1][2 * (i & 1)] = perm(hi, L, 0x05010400u);
        o[i >> 1][2 * (i & 1) + 1] = perm(hi, L, 0x07030602u);
      }
      const unsigned cnt = (h0 >> 8) & 0xff;
      if (cnt == 0) {
#pragma unroll
        for (int p = 0; p < 4; p++)
          __builtin_amdgcn_raw_buffer_store_b128(o[p], rd, p * 1024 + (kind ? (wperm ^ (p * 16)) : lane * 16), dst, AUX_AGENT);
      } else {
#pragma unroll
        for (int p = 0; p < 4; p++) *(lu32x4*)(scr + p * 1024 + (kind ? (wperm ^ (p * 16)) : lane * 16)) = o[p];
        if (lane < (int)cnt) {
          const unsigned idx = *(const lu16*)(sl + 2816 + 4 + 2 * lane);
          *(lu8*)(scr + 2 * (idx & 0x7ff) + 1) = *(const lu8*)(sl + 2816 + 4 + 2 * NEXC + lane);
        }
#pragma unroll
        for (int p = 0; p < 4; p++) o[p] = *(const lu32x4*)(scr + p * 1024 + lane * 16);
#pragma unroll
        for (int p = 0; p < 4; p++) __builtin_amdgcn_raw_buffer_store_b128(o[p], rd, p * 1024 + lane * 16, dst, AUX_AGENT);
      }
    };
    auto arrive = [&](unsigned n) {
      const unsigned cq = n / 20, c = C0 + cq, r = n - cq * 20, w = r / 5, u16 = (ue >> (4 * (r - w * 5))) & 15;
      const unsigned so = (unsigned)(ARR_OFF - FLAG_OFF) + (src * 16 + u16) * 256 + (c * 2 + (w >> 1)) * 4;
      asm volatile("buffer_atomic_add %0, %1, %2, %3 offen" :: "v"(1u), "v"(l0), "s"(rfs), "s"(so) : "memory");
    };
    unsigned cur = ticket();
    unsigned nxt = cur + 1, prev = NITEM, slot = 0;
    grab();
    if (cur < NITEM) need(cur);
    load(0, cur);
    while (cur < NITEM) {
      if (nxt < NITEM) need(nxt);
      load(slot ^ 1, nxt);
      asm volatile("s_waitcnt vmcnt(3)" ::: "memory");
      if (prev < NITEM) arrive(prev);
      const unsigned nn = ticket();
      grab();
      emit(slot, cur);
      prev = cur; cur = nxt; nxt = nn; slot ^= 1;
    }
    asm volatile("s_waitcnt vmcnt(0)" ::: "memory");
    if (prev < NITEM) arrive(prev);
    return;
  }

  const int t1 = (te >> 14) & 7, i1 = (te >> 11) & 7, t2 = (te >> 8) & 7, i2 = (te >> 5) & 7, tu = (te >> 2) & 7, pj = te & 3;
  const int srcW = rank ^ tu;
  char* const donep = a.ws[srcW & 7] + DONE_OFF + b * 4;
  lu32* sw = (lu32*)(smem + NBUF * STAGE_BYTES);
  if (wv >= 4) {
    const int lw = wv - 4;
    const int kind = lw >> 1;
    const int src = lw == 0 ? rank ^ t1 : (lw == 1 ? rank ^ t2 : srcW);
    const int unit = lw == 0 ? i1 : (lw == 1 ? i2 : 2 * pj + (lw & 1));
    const char* g = me + stg_off(kind, src, unit) + lane * 16;
    const bool loc = src == rank;
    const char* pollp = loc ? me + READY_OFF + (size_t)(((src & 7) * 2 + kind) * 8 + unit) * 128
                            : me + ARR_OFF + (size_t)(((src & 7) * 2 + kind) * 8 + unit) * 256;
    const unsigned pthr = loc ? base + 1 : 2 * call;
    const int nword = loc ? 16 : 64;
    auto eval = [&](int q, unsigned v) -> int {
      const int w0 = loc ? q >> 2 : q;
      const unsigned long long nr = __ballot(lane < 16 && w0 + lane < nword && v < pthr);
      const int f = __builtin_ctz((unsigned)nr | 0x10000u);
      return min(64, loc ? 4 * (w0 + f) : w0 + f);
    };
    auto wordp = [&](int q) { return pollp + min((loc ? q >> 2 : q) + (lane & 15), nword - 1) * 4; };
    int avail = 0, pend = -1;
    int nb = 0;
    for (int s = -PF; s < 64; s++) {
      __builtin_amdgcn_s_barrier();
      asm volatile("" ::: "memory");
      if (s + PF < 64) {
        const int q = s + PF;
        if (q >= avail) {
          for (Spin sp;;) {
            avail = eval(q, ld_sys(wordp(q)));
            if (avail > q) break;
            if (sp.expired(me, call)) { avail = 64; break; }
          }

        }
        const unsigned* gp = (const unsigned*)(g + (size_t)q * (2 * BLK));
        unsigned* lp = (unsigned*)(smem + nb * STAGE_BYTES + lw * (2 * BLK));
        nb = nb == NBUF - 1 ? 0 : nb + 1;
        __builtin_amdgcn_global_load_lds(gp, lp, 16, 0, AUX_SYS);
        __builtin_amdgcn_global_load_lds(gp, lp, 16, 1024, AUX_SYS);
        __builtin_amdgcn_global_load_lds(gp, lp, 16, 2048, AUX_SYS);
        __builtin_amdgcn_global_load_lds(gp, lp, 16, 3072, AUX_SYS);
        __builtin_amdgcn_global_load_lds(gp + 1024, lp + 1024, 16, 0, AUX_SYS);
        __builtin_amdgcn_global_load_lds(gp + 1024, lp + 1024, 16, 1024, AUX_SYS);
        __builtin_amdgcn_global_load_lds(gp + 1024, lp + 1024, 16, 2048, AUX_SYS);
        __builtin_amdgcn_global_load_lds(gp + 1024, lp + 1024, 16, 3072, AUX_SYS);
        if ((q & 1) == 0) {
          if (avail < 64 && avail <= q + 2 * LOOK + 1) {
            asm volatile("global_load_dword v200, %0, off sc0 sc1" :: "v"(wordp(avail)) : "v200", "memory");
            pend = avail;
          }
        } else if (pend >= 0) {
          unsigned pv;
          asm volatile("s_waitcnt vmcnt(8)\n\tv_mov_b32 %0, v200" : "=v"(pv) :: "v200", "memory");
          avail = max(avail, eval(pend, pv));
          pend = -1;
        }
        if (PF == 4) asm volatile("s_waitcnt vmcnt(24)" ::: "memory");
        else if (PF == 3) asm volatile("s_waitcnt vmcnt(16)" ::: "memory");
        else asm volatile("s_waitcnt vmcnt(8)" ::: "memory");
      } else {
        const int rem = 62 - s;
        if (rem >= 2) asm volatile("s_waitcnt vmcnt(16)" ::: "memory");
        else if (rem == 1) asm volatile("s_waitcnt vmcnt(8)" ::: "memory");
        else asm volatile("s_waitcnt vmcnt(0)" ::: "memory");
      }
    }
    if (lw == 0 && lane == 0) sw[4] = 0;
    asm volatile("s_waitcnt lgkmcnt(0)" ::: "memory");
    __builtin_amdgcn_s_barrier();
    if (lw == 0) {
      const char* rp = me + RES_OFF + (b & 7) * 128;
      for (Spin sp;;) {
        u32x2 r;
        unsigned tm;
        asm volatile("global_load_dwordx2 %0, %2, off sc1\n\tglobal_load_dword %1, %3, off sc1\n\ts_waitcnt vmcnt(0)"
                     : "=&v"(r), "=&v"(tm) : "v"(rp), "v"(me + TMO_OFF) : "memory");
        const bool fin = __builtin_amdgcn_readfirstlane(r[1]) >= call;
        bool bad = __builtin_amdgcn_readfirstlane(tm) == call || (fin && __builtin_amdgcn_readfirstlane(r[0]) == 0x7fffffffu);
        if (!fin && !bad && sp.expired(me, call)) bad = true;
        if (fin || bad) {
          if (lane == 0) {
            sw[5] = r[0];
            sw[6] = bad;
            __hip_atomic_store(sw + 4, 1u, __ATOMIC_RELEASE, __HIP_MEMORY_SCOPE_WORKGROUP);
          }
          break;
        }
      }
    }
    return;
  }
  const int xu = wv >> 1, wu = wv & 1;
  f32x4 acc[4][4];
#pragma unroll
  for (int f = 0; f < 4; f++)
#pragma unroll
    for (int j = 0; j < 4; j++) acc[f][j] = f32x4{0.f, 0.f, 0.f, 0.f};
  const unsigned xoff = xu * (2 * BLK) + lane * 16;
  unsigned woff[4];
  {
    const int r = lane & 3, q = (lane >> 2) & 3, kq = lane >> 4;
#pragma unroll
    for (int j = 0; j < 4; j++) woff[j] = (2 + wu) * (2 * BLK) + r * 1024 + kq * 256 + q * 64 + ((j ^ r) * 16);
  }
  {
    bf16x8 xa[4], wa[4], xb[4], wb[4];
#pragma unroll
    for (int f = 0; f < 4; f++) { xb[f] = bf16x8{}; wb[f] = bf16x8{}; }
    for (int s = -PF; s < 0; s++) __builtin_amdgcn_s_barrier();
    int cb = 0;
#define ILV() do { _Pragma("unroll") for (int q_ = 0; q_ < 8; q_++) { __builtin_amdgcn_sched_group_barrier(0x008, 1, 0); \
    __builtin_amdgcn_sched_group_barrier(0x100, 1, 0); } __builtin_amdgcn_sched_group_barrier(0x008, 8, 0); } while (0)
    for (int s = 0; s < 64; s++) {
      asm volatile("s_waitcnt lgkmcnt(0)" ::: "memory");
      __builtin_amdgcn_s_barrier();
      asm volatile("" ::: "memory");

      const char* sb = smem + cb * STAGE_BYTES;
      cb = cb == NBUF - 1 ? 0 : cb + 1;
      __builtin_amdgcn_sched_barrier(0);
      wa[0] = *(const bf16x8*)(sb + woff[0]); xa[0] = *(const bf16x8*)(sb + xoff);
#pragma unroll
      for (int f = 1; f < 4; f++) wa[f] = *(const bf16x8*)(sb + woff[f]);
#pragma unroll
      for (int f = 1; f < 4; f++) xa[f] = *(const bf16x8*)(sb + xoff + f * 1024);
#pragma unroll
      for (int f = 0; f < 4; f++)
#pragma unroll
        for (int j = 0; j < 4; j++) acc[f][j] = __builtin_amdgcn_mfma_f32_16x16x32_bf16(wb[j], xb[f], acc[f][j], 0, 0, 0);
      ILV();
      __builtin_amdgcn_sched_barrier(0);
      wb[0] = *(const bf16x8*)(sb + woff[0] + BLK); xb[0] = *(const bf16x8*)(sb + xoff + BLK);
#pragma unroll
      for (int f = 1; f < 4; f++) wb[f] = *(const bf16x8*)(sb + woff[f] + BLK);
#pragma unroll
      for (int f = 1; f < 4; f++) xb[f] = *(const bf16x8*)(sb + xoff + BLK + f * 1024);
#pragma unroll
      for (int f = 0; f < 4; f++)
#pragma unroll
        for (int j = 0; j < 4; j++) acc[f][j] = __builtin_amdgcn_mfma_f32_16x16x32_bf16(wa[j], xa[f], acc[f][j], 0, 0, 0);
      ILV();
      __builtin_amdgcn_sched_barrier(0);
    }
#pragma unroll
    for (int f = 0; f < 4; f++)
#pragma unroll
      for (int j = 0; j < 4; j++) acc[f][j] = __builtin_amdgcn_mfma_f32_16x16x32_bf16(wb[j], xb[f], acc[f][j], 0, 0, 0);
  }
  float mf = 0.f;
#pragma unroll
  for (int f = 0; f < 4; f++)
#pragma unroll
    for (int j = 0; j < 4; j++) {
      const f32x4 t = acc[f][j];
      mf = fmaxf(fmaxf(mf, fabsf(t[0])), fabsf(t[1]));
      mf = fmaxf(fmaxf(mf, fabsf(t[2])), fabsf(t[3]));
    }
  unsigned m = wave_max_u32(__float_as_uint(mf));
  if (lane == 0) sw[wv] = m;
  asm volatile("s_waitcnt lgkmcnt(0)" ::: "memory");
  __builtin_amdgcn_s_barrier();
  asm volatile("" ::: "memory");
  m = max(max(sw[0], sw[1]), max(sw[2], sw[3]));
  if (tid < NR) {
    gu64* g = (gu64*)(wsl + AMAX_OFF + (size_t)(rank * NGEMM + b) * 8);
    __hip_atomic_store(g, ((unsigned long long)call << 32) | m, __ATOMIC_RELAXED, __HIP_MEMORY_SCOPE_SYSTEM);
  }
  while (__builtin_amdgcn_readfirstlane(__hip_atomic_load(sw + 4, __ATOMIC_ACQUIRE, __HIP_MEMORY_SCOPE_WORKGROUP)) == 0)
    __builtin_amdgcn_s_sleep(1);
  const float inv = 448.0f / __uint_as_float(__builtin_amdgcn_readfirstlane(sw[5]));
  const bool bad = __builtin_amdgcn_readfirstlane(sw[6]) != 0;
  {
    const int tx = xu == 0 ? t1 : t2, ix = xu == 0 ? i1 : i2;
    unsigned char* optr = outp + (size_t)((rank ^ tx) * SH + ix * 64 + (lane & 15)) * SH + pj * 128 + wu * 64 + (lane >> 4) * 16;
#pragma unroll
    for (int f = 0; f < 4; f++) {
      u32x4 q;
#pragma unroll
      for (int j = 0; j < 4; j++) {
        int w = __builtin_amdgcn_cvt_pk_fp8_f32(acc[f][j][0] * inv, acc[f][j][1] * inv, 0, false);
        w = __builtin_amdgcn_cvt_pk_fp8_f32(acc[f][j][2] * inv, acc[f][j][3] * inv, w, true);
        q[j] = bad ? 0x7f7f7f7fu : (unsigned)w;
      }
      if (tu != 0) asm volatile("global_store_dwordx4 %0, %1, off sc0 sc1\n\ts_nop 1" ::"v"(optr + (size_t)f * 16 * SH), "v"(q) : "memory");
      else *(u32x4*)(optr + (size_t)f * 16 * SH) = q;
    }
  }
  if (tu != 0) {
    asm volatile("s_waitcnt vmcnt(0)" ::: "memory");
    __builtin_amdgcn_s_barrier();
    if (tid == 0) st_sys(donep, call);
  }
}
}

extern "C" void kernel_launch(int rank, int nranks, ncclComm_t comm, hipStream_t stream, void* const* d_in,
                              const int* in_sizes, int n_in, void* d_out, int out_size, const int* in_dtypes,
                              int out_dtype, void* const* d_out_all, void* const* d_ws_all, size_t ws_bytes) {
  (void)comm; (void)in_sizes; (void)n_in; (void)d_out; (void)out_size; (void)in_dtypes; (void)out_dtype;
  if (nranks != NR || rank < 0 || rank >= NR || ws_bytes < WS_NEED) {
    fprintf(stderr, "unsupported configuration\n");
    return;
  }
  Args a;
  a.x = (const float*)d_in[0];
  a.w = (const float*)d_in[1];
  for (int p = 0; p < NR; p++) {
    a.ws[p] = (char*)d_ws_all[p];
    a.out[p] = (unsigned char*)d_out_all[p];
  }
  a.rank = rank;
  a.pad = 0;
  fused_ag_gemm<<<NWG, 512, 0, stream>>>(a);
  hipError_t e = hipGetLastError();
  if (e != hipSuccess) fprintf(stderr, "launch error %s\n", hipGetErrorString(e));
}
```

```cpp
#include <hip/hip_runtime.h>
#include <rccl/rccl.h>
#include <cstdio>
#include <cstdint>

typedef unsigned u32x4 __attribute__((ext_vector_type(4)));
typedef unsigned u32x3 __attribute__((ext_vector_type(3)));
typedef unsigned u32x2 __attribute__((ext_vector_type(2)));
typedef float f32x4 __attribute__((ext_vector_type(4)));
typedef __bf16 bf16x8 __attribute__((ext_vector_type(8)));
typedef __bf16 bf16x2 __attribute__((ext_vector_type(2)));
typedef float f32x2 __attribute__((ext_vector_type(2)));

namespace {
constexpr int PPL = 13;
constexpr int NPW = 2;
constexpr int NPWN = 8;
constexpr int NPUSH = 1;
constexpr int LOOK = 1;
constexpr int STAG0 = 40;
constexpr int STAGGER = 10;
constexpr int NBUF = 4;
constexpr int PF = NBUF - 1;
constexpr unsigned long long DEADLINE = 180000000ull;
constexpr int NR = 8;
constexpr int KDIM = 4096;
constexpr int SH = 512;
constexpr unsigned BLK = 4096;
constexpr unsigned UNIT_BYTES = 128 * BLK;
constexpr unsigned PBLK = 2944;
constexpr unsigned PUNIT_BYTES = 128 * PBLK;
constexpr unsigned NEXC = 41;
constexpr unsigned UNIT_STRIDE = UNIT_BYTES + 4352;
constexpr size_t STG_KIND_BYTES = (size_t)NR * 8 * UNIT_STRIDE;
constexpr size_t EXP_OFF = 2 * STG_KIND_BYTES;
constexpr size_t FLAG_OFF = EXP_OFF + (size_t)16 * PUNIT_BYTES;
constexpr size_t READY_OFF = FLAG_OFF;
constexpr size_t ARR_OFF = READY_OFF + NR * 16 * 128;
constexpr size_t AMAX_OFF = ARR_OFF + NR * 16 * 256;
constexpr size_t DONE_OFF = AMAX_OFF + NR * 128 * 8;
constexpr size_t CTR_OFF = DONE_OFF + 1024;
constexpr size_t TMO_OFF = CTR_OFF + 128;
constexpr size_t QCTR_OFF = TMO_OFF + 128;
constexpr size_t RES_OFF = QCTR_OFF + 16 * 128;
constexpr size_t END_OFF = RES_OFF + 8 * 128;
constexpr size_t WS_NEED = END_OFF;
static_assert(FLAG_OFF % 128 == 0, "align");
constexpr int NGEMM = 128, NPULL = 7 * PPL, NWG = 256;
constexpr unsigned STAGE_BYTES = 8 * BLK;

__constant__ unsigned TILE_TAB[NGEMM] = {16420,2145,5154,5283,14530,10241,8928,10432,69476,67649,7906,47971,70370,15553,17664,15424,3116,74817,8450,78019,77058,16577,113792,55360,18284,75905,12130,83011,3238,113377,121888,75872,70924,82785,73762,70820,13574,1059,3115,77856,77964,5158,78690,82660,69798,18147,7435,4311,8207,55430,3939,112708,78566,76,47243,47287,121935,75622,6307,75,112902,11404,113803,55351,1081,3112,73859,11115,10991,13484,4373,69751,9433,77064,10340,75979,12335,48876,9973,12377,12153,121928,12420,78923,76015,9271,67637,47225,17561,5167,74948,55518,79599,74935,73845,75833,83033,75023,6216,70846,11477,78103,77685,2234,2238,2173,12008,78590,69845,12381,47258,55386,8478,70813,73864,81950,9501,48349,68890,69850,68766,76669,78024,112734,10493,69373,75642,79034};
__constant__ unsigned PULL_TAB[8] = {0, 0xdc980, 0xfe987, 0xfe983, 0x65421, 0xfeba0, 0xdcba7, 0xdcba3};
constexpr unsigned long long DEST_MASK_X = 0x4511111189111123ull, DEST_MASK_W = 0x2d2dc3c3e1e10f0full;

struct Args {
  const float* x;
  const float* w;
  char* ws[NR];
  unsigned char* out[NR];
  int rank;
  int pad;
};

typedef __attribute__((address_space(1))) unsigned gu32;
typedef __attribute__((address_space(1))) unsigned long long gu64;
typedef __attribute__((address_space(3))) unsigned lu32;
typedef __attribute__((address_space(3))) unsigned char lu8;
typedef __attribute__((address_space(3))) unsigned short lu16;
typedef __attribute__((address_space(3))) u32x4 lu32x4;

__device__ __forceinline__ unsigned ld_sys(const void* p) { return __hip_atomic_load((const gu32*)p, __ATOMIC_RELAXED, __HIP_MEMORY_SCOPE_SYSTEM); }
__device__ __forceinline__ void st_sys(void* p, unsigned v) { __hip_atomic_store((gu32*)p, v, __ATOMIC_RELAXED, __HIP_MEMORY_SCOPE_SYSTEM); }
__device__ __forceinline__ unsigned ld_agent(const void* p) { return __hip_atomic_load((const gu32*)p, __ATOMIC_RELAXED, __HIP_MEMORY_SCOPE_AGENT); }
__device__ __forceinline__ void st_agent(void* p, unsigned v) { __hip_atomic_store((gu32*)p, v, __ATOMIC_RELAXED, __HIP_MEMORY_SCOPE_AGENT); }

__device__ __forceinline__ __amdgpu_buffer_rsrc_t mkrsrc(const void* p, unsigned bytes) {
  unsigned long long a = (unsigned long long)p;
  unsigned lo = __builtin_amdgcn_readfirstlane((unsigned)a), hi = __builtin_amdgcn_readfirstlane((unsigned)(a >> 32));
  void* q = (void*)(((unsigned long long)hi << 32) | lo);
  return __builtin_amdgcn_make_buffer_rsrc(q, 0, bytes, 0x00020000);
}
__device__ __forceinline__ u32x4 mkrsrc_words(const void* p, unsigned bytes) {
  unsigned long long a = (unsigned long long)p;
  return u32x4{(unsigned)__builtin_amdgcn_readfirstlane((unsigned)a), (unsigned)__builtin_amdgcn_readfirstlane((unsigned)(a >> 32)) & 0xffffu,
               bytes, 0x00020000u};
}
__device__ __forceinline__ size_t stg_off(int kind, int src, int unit) {
  return (size_t)kind * STG_KIND_BYTES + (size_t)(src * 8 + unit) * UNIT_STRIDE;
}
__device__ __forceinline__ unsigned cvt2(float a, float b) {
  f32x2 v = {a, b};
  bf16x2 r = __builtin_convertvector(v, bf16x2);
  return __builtin_bit_cast(unsigned, r);
}
__device__ __forceinline__ unsigned perm(unsigned hi, unsigned lo, unsigned sel) { return __builtin_amdgcn_perm(hi, lo, sel); }
__device__ __forceinline__ unsigned wave_max_u32(unsigned v) {
  v = max(v, (unsigned)__builtin_amdgcn_update_dpp(0, (int)v, 0x111, 0xf, 0xf, false));
  v = max(v, (unsigned)__builtin_amdgcn_update_dpp(0, (int)v, 0x112, 0xf, 0xf, false));
  v = max(v, (unsigned)__builtin_amdgcn_update_dpp(0, (int)v, 0x114, 0xf, 0xf, false));
  v = max(v, (unsigned)__builtin_amdgcn_update_dpp(0, (int)v, 0x118, 0xf, 0xf, false));
  v = max(v, (unsigned)__builtin_amdgcn_update_dpp(0, (int)v, 0x142, 0xa, 0xf, false));
  v = max(v, (unsigned)__builtin_amdgcn_update_dpp(0, (int)v, 0x143, 0xc, 0xf, false));
  return (unsigned)__builtin_amdgcn_readlane((int)v, 63);
}

struct Spin {
  unsigned it = 0;
  unsigned long long t0 = 0;
  __device__ __forceinline__ bool expired(char* me, unsigned call) {
    if (it == 0) t0 = __builtin_amdgcn_s_memrealtime();
    if ((++it & 15) == 0) {
      if (ld_agent(me + TMO_OFF) == call || __builtin_amdgcn_s_memrealtime() - t0 > DEADLINE) {
        st_agent(me + TMO_OFF, call);
        return true;
      }
    }
    __builtin_amdgcn_s_sleep(1);
    return false;
  }
};
constexpr int AUX_SYS = 17, AUX_AGENT = 16;

__device__ __forceinline__ unsigned piece_off(int kind, int p, int lane) {
  const int kq = lane >> 4, q = (lane >> 2) & 3, j = lane & 3;
  return kind == 0 ? p * 1024 + lane * 16 : p * 1024 + kq * 256 + q * 64 + ((j ^ p) * 16);
}

__global__ __launch_bounds__(512, 2) void fused_ag_gemm(Args a) {
  __shared__ __attribute__((aligned(16))) char smem[NBUF * STAGE_BYTES + 128];
  const int tid = threadIdx.x, lane = tid & 63;
  const int wv = __builtin_amdgcn_readfirstlane(tid >> 6);
  const int b = blockIdx.x;
  constexpr int NH0 = 16 * NPUSH;
  const int task = b < NGEMM ? b + NH0 + NPULL
                             : (b < NGEMM + NPULL ? b - NGEMM + NH0 : (b - (NGEMM + NPULL) < NH0 ? b - (NGEMM + NPULL) : b));
  const int c16 = task >> 4, u16 = task & 15, kind = u16 >> 3, unit = u16 & 7;
  const int ks = c16 * 8 + wv;
  const int kq = lane >> 4, l15 = lane & 15;
  if (c16 > NPUSH) {
    const unsigned long long tstart = __builtin_amdgcn_s_memrealtime();
    while (__builtin_amdgcn_s_memrealtime() - tstart < (unsigned long long)(STAG0 + (c16 - NPUSH - 1) * STAGGER)) __builtin_amdgcn_s_sleep(1);
  }
  f32x4 v[8];
  {
    const float* px = a.x + (size_t)(unit * 64 + l15) * KDIM + ks * 32 + 4 * kq;
    const float* pw = a.w + (size_t)(ks * 32 + 4 * kq) * SH + unit * 64 + 4 * l15;
    const float* p = kind == 0 ? px : pw;
#pragma unroll
    for (int i = 0; i < 8; i++) {
      const unsigned ox = (i >> 1) * 16 * KDIM + (i & 1) * 16, ow = ((i & 3) + (i >> 2) * 16) * SH;
      v[i] = *(const f32x4*)(p + (kind == 0 ? ox : ow));
    }
  }
  asm volatile("" ::: "memory");
  const int rank = a.rank;
  char* me = a.ws[rank];
  const unsigned ctr_v = ld_agent(me + CTR_OFF);
  const unsigned te = TILE_TAB[b & (NGEMM - 1)];
  const bool is_pull = b >= NGEMM && b < NGEMM + NPULL;
  const int plink = is_pull ? 1 + (b - NGEMM) / PPL : 0;
  const int npw = is_pull && (b - NGEMM) % PPL >= NPWN ? NPW - 1 : NPW;
  const unsigned ue = PULL_TAB[plink];
  char* const wsl = a.ws[lane & 7];
  unsigned char* const outp = a.out[(rank ^ (int)(te >> 2)) & 7];
  const u32x4 rfs = mkrsrc_words(me + FLAG_OFF, (unsigned)(END_OFF - FLAG_OFF));
  const unsigned l0 = lane == 0 ? 0u : 0x40000000u;
  unsigned call, base;
  {
    const __amdgpu_buffer_rsrc_t rd = mkrsrc(me + stg_off(kind, rank, unit), UNIT_BYTES);
    call = __builtin_amdgcn_readfirstlane(ctr_v) + 1;
    base = call * 64;
    if (is_pull && wv < npw) {
      const unsigned qoff = (unsigned)(QCTR_OFF - FLAG_OFF) + ((call & 1) * 8 + plink) * 128;
      asm volatile("v_mov_b32 v200, 2\n\ts_nop 4\n\tbuffer_atomic_add v200, %0, %1, %2 offen sc0"
                   :: "v"(l0), "s"(rfs), "s"(qoff) : "v200", "memory");
    }
    u32x4 o[4];
    if (kind == 0) {
#pragma unroll
      for (int f = 0; f < 4; f++)
        o[f] = u32x4{cvt2(v[2 * f][0], v[2 * f][1]), cvt2(v[2 * f][2], v[2 * f][3]),
                     cvt2(v[2 * f + 1][0], v[2 * f + 1][1]), cvt2(v[2 * f + 1][2], v[2 * f + 1][3])};
    } else {
#pragma unroll
      for (int r = 0; r < 4; r++)
        o[r] = u32x4{cvt2(v[0][r], v[1][r]), cvt2(v[2][r], v[3][r]), cvt2(v[4][r], v[5][r]), cvt2(v[6][r], v[7][r])};
    }
    const unsigned dmask = (unsigned)((kind ? DEST_MASK_W : DEST_MASK_X) >> (8 * unit)) & 0xff;
    if (c16 < NPUSH) {
#pragma unroll
      for (int p = 1; p < NR; p++)
        if ((dmask >> p) & 1) {
          const __amdgpu_buffer_rsrc_t rq = mkrsrc(a.ws[rank ^ p] + stg_off(kind, rank, unit), UNIT_BYTES);
#pragma unroll
          for (int i = 0; i < 4; i++)
            __builtin_amdgcn_raw_buffer_store_b128(o[i], rq, ks * BLK + piece_off(kind, i, lane), 0, AUX_SYS);
        }
    } else {
      float mx = 0.f;
#pragma unroll
      for (int i = 0; i < 8; i++) {
        mx = fmaxf(mx, fmaxf(fabsf(v[i][0]), fabsf(v[i][1])));
        mx = fmaxf(mx, fmaxf(fabsf(v[i][2]), fabsf(v[i][3])));
      }
      const unsigned mxb = wave_max_u32(__float_as_uint(mx));
      const unsigned emax = (cvt2(__uint_as_float(mxb), 0.f) >> 7) & 0xff;
      const float thr1 = __uint_as_float(emax << 23), thr2 = thr1 * 0.5f;
      unsigned n1 = 0, n2 = 0;
#pragma unroll
      for (int i = 0; i < 8; i++) {
        n1 += __builtin_popcountll(__ballot(fabsf(v[i][0]) >= thr1)) + __builtin_popcountll(__ballot(fabsf(v[i][2]) >= thr1));
        n2 += __builtin_popcountll(__ballot(fabsf(v[i][0]) >= thr2)) + __builtin_popcountll(__ballot(fabsf(v[i][2]) >= thr2));
      }
      const int top = (int)emax - (n1 <= 10 ? 1 : 0) - (n2 <= 10 ? 1 : 0);
      const unsigned delta = (top & 1) ^ 1, tp = ((top + (int)delta) >> 1) & 0x7f;
      const unsigned Md = delta ? 0x80808080u : 0u;
      const unsigned b80 = tp * 0x01010101u | 0x80808080u;
      const unsigned T = (b80 - 0x03020100u) & 0x7f7f7f7fu;
      unsigned LP[8], HH[8], S = 0, K0 = 0, K1 = 0, M = 0;
#pragma unroll
      for (int i = 0; i < 8; i++) {
        const unsigned d0 = o[i >> 1][2 * (i & 1)], d1 = o[i >> 1][2 * (i & 1) + 1];
        const unsigned L = perm(d1, d0, 0x06040200u), H = perm(d1, d0, 0x07050301u);
        const unsigned pair = ((H & 0x7f7f7f7fu) + ((L & Md) >> 7)) & 0x7f7f7f7fu;
        const unsigned code = (b80 - pair) & 0x03030303u;
        const unsigned mis = perm(T, T, code) ^ pair;
        LP[i] = L ^ Md;
        HH[i] = H;
        if (i < 4) K0 |= code << (2 * i); else K1 |= code << (2 * (i - 4));
        S |= (H & 0x80808080u) >> (7 - i);
        M |= __builtin_amdgcn_udot4(((mis + 0x7f7f7f7fu) & 0x80808080u) >> 7, 0x08040201u, 0u, false) << (4 * i);
      }
      char* hsc = smem + wv * 2304;
      lu32* hdr = (lu32*)(hsc + 2048);
      const unsigned nl = __builtin_popcount(M);
      unsigned inc = nl;
      inc += __builtin_amdgcn_update_dpp(0, (int)inc, 0x111, 0xf, 0xf, false);
      inc += __builtin_amdgcn_update_dpp(0, (int)inc, 0x112, 0xf, 0xf, false);
      inc += __builtin_amdgcn_update_dpp(0, (int)inc, 0x114, 0xf, 0xf, false);
      inc += __builtin_amdgcn_update_dpp(0, (int)inc, 0x118, 0xf, 0xf, false);
      inc += __builtin_amdgcn_update_dpp(0, (int)inc, 0x142, 0xa, 0xf, false);
      inc += __builtin_amdgcn_update_dpp(0, (int)inc, 0x143, 0xc, 0xf, false);
      const unsigned cnt = (unsigned)__builtin_amdgcn_readlane((int)inc, 63);
      if (cnt > 0 && cnt <= NEXC) {
        *(lu32x4*)(hsc + lane * 32) = u32x4{HH[0], HH[1], HH[2], HH[3]};
        *(lu32x4*)(hsc + lane * 32 + 16) = u32x4{HH[4], HH[5], HH[6], HH[7]};
        unsigned at = inc - nl;
        while (__any(M != 0)) {
          if (M != 0) {
            const unsigned jv = __builtin_ctz(M);
            M &= M - 1;
            const unsigned char hi = *(const lu8*)(hsc + lane * 32 + jv);
            const unsigned idx = (piece_off(kind, jv >> 3, lane) >> 1) + (jv & 7);
            *(lu16*)((char*)hdr + 4 + 2 * at) = (unsigned short)idx;
            *(lu8*)((char*)hdr + 4 + 2 * NEXC + at) = hi;
            at++;
          }
        }
      }
      const __amdgpu_buffer_rsrc_t rp = mkrsrc(me + EXP_OFF + (size_t)u16 * PUNIT_BYTES, PUNIT_BYTES);
      const unsigned pc = ks * PBLK;
      __builtin_amdgcn_raw_buffer_store_b128(u32x4{LP[0], LP[1], LP[2], LP[3]}, rp, pc + lane * 16, 0, AUX_SYS);
      __builtin_amdgcn_raw_buffer_store_b128(u32x4{LP[4], LP[5], LP[6], LP[7]}, rp, pc + 1024 + lane * 16, 0, AUX_SYS);
      __builtin_amdgcn_raw_buffer_store_b96(u32x3{S, K0, K1}, rp, pc + 2048 + lane * 12, 0, AUX_SYS);
      unsigned hv = hdr[lane & 31];
      if (lane == 0) hv = tp | (delta << 7) | (cnt > NEXC ? 0x10000u : cnt << 8);
      if (lane < 32) __builtin_amdgcn_raw_buffer_store_b32(hv, rp, pc + 2816 + lane * 4, 0, AUX_SYS);
    }
#pragma unroll
    for (int p = 0; p < 4; p++)
      __builtin_amdgcn_raw_buffer_store_b128(o[p], rd, ks * BLK + piece_off(kind, p, lane), 0, AUX_SYS);
    asm volatile("s_waitcnt vmcnt(0)" ::: "memory");
    const int rel = rank ^ lane;
    if (c16 < NPUSH) {
      if (lane < NR && rel != 0 && ((dmask >> rel) & 1))
        __hip_atomic_fetch_add((gu32*)(wsl + ARR_OFF + (size_t)(rank * 16 + u16) * 256 + (ks >> 1) * 4), 1u, __ATOMIC_RELAXED,
                               __HIP_MEMORY_SCOPE_SYSTEM);
    }
    __builtin_amdgcn_s_barrier();
    if (wv == 0 && lane < NR && ((dmask >> rel) & 1) && (rel == 0 || c16 >= NPUSH))
      st_sys(wsl + READY_OFF + (size_t)(rank * 16 + u16) * 128 + c16 * 4, base + 1);
  }

  if (b >= NGEMM + NPULL) {
    if (b == NWG - 1) {
      {
        lu32* sv = (lu32*)smem;
        __builtin_amdgcn_s_barrier();
        if (tid < 3) sv[tid] = 0;
        asm volatile("s_waitcnt lgkmcnt(0)" ::: "memory");
        __builtin_amdgcn_s_barrier();
        const char* gp = me + AMAX_OFF + (size_t)tid * 16;
        bool have = false, bad = false;
        unsigned gm = 0;
        for (Spin sp;;) {
          unsigned tm = 0;
          if (!have) {
            u32x4 g0;
            asm volatile("global_load_dwordx4 %0, %2, off sc0 sc1\n\tglobal_load_dword %1, %3, off sc1\n\ts_waitcnt vmcnt(0)"
                         : "=&v"(g0), "=&v"(tm) : "v"(gp), "v"(me + TMO_OFF) : "memory");
            have = g0[1] >= call && g0[3] >= call;
            gm = max(g0[0], g0[2]);
          }
          if (__any(tm == call)) { bad = true; break; }
          if (__all(have)) break;
          if (sp.expired(me, call)) { bad = true; break; }
        }
        const unsigned wm = wave_max_u32(gm);
        unsigned last = 0;
        if (lane == 0) {
          __hip_atomic_fetch_max(sv, wm, __ATOMIC_RELAXED, __HIP_MEMORY_SCOPE_WORKGROUP);
          if (bad) __hip_atomic_store(sv + 2, 1u, __ATOMIC_RELAXED, __HIP_MEMORY_SCOPE_WORKGROUP);
          last = __hip_atomic_fetch_add(sv + 1, 1u, __ATOMIC_ACQ_REL, __HIP_MEMORY_SCOPE_WORKGROUP);
        }
        if (__builtin_amdgcn_readfirstlane(last) == 7) {
          const unsigned am = sv[2] ? 0x7fffffffu : sv[0];
          if (lane < 8) __hip_atomic_store((gu64*)(me + RES_OFF + lane * 128), ((unsigned long long)call << 32) | am,
                                           __ATOMIC_RELAXED, __HIP_MEMORY_SCOPE_AGENT);
        }
      }
      if (wv == 0) {
        bool ok = true;
        {
          const bool a0 = ((TILE_TAB[2 * lane] >> 2) & 7) != 0, a1 = ((TILE_TAB[2 * lane + 1] >> 2) & 7) != 0;
          for (Spin sp;;) {
            u32x2 d;
            asm volatile("global_load_dwordx2 %0, %1, off sc0 sc1\n\ts_waitcnt vmcnt(0)" : "=v"(d) : "v"(me + DONE_OFF + lane * 8) : "memory");
            if (__all((!a0 || d[0] >= call) && (!a1 || d[1] >= call))) break;
            if (sp.expired(me, call)) { ok = false; break; }
          }
        }
        if (lane < 8) st_agent(me + QCTR_OFF + (size_t)(((call + 1) & 1) * 8 + lane) * 128, 0u);
        if (lane == 0) {
          if (!ok) a.out[rank][(size_t)(rank * SH + 64) * SH + 128] = 0x7f;
          st_agent(me + CTR_OFF, call);
        }
      }
    }
    return;
  }
  if (is_pull) {
    if (wv >= npw) return;
    const unsigned src = (unsigned)(rank ^ plink);
    if (src >= NR) return;
    const __amdgpu_buffer_rsrc_t rraw = mkrsrc(a.ws[src], 2 * STG_KIND_BYTES);
    const __amdgpu_buffer_rsrc_t rd = mkrsrc(me, 2 * STG_KIND_BYTES);
    const u32x4 rss = mkrsrc_words(a.ws[src] + EXP_OFF, 16 * PUNIT_BYTES);
    const unsigned wperm = piece_off(1, 0, lane);
    char* scr = smem + 1024 + wv * BLK;
    constexpr unsigned SLOT = 3072;
    char* ring = smem + 1024 + NPW * BLK + wv * 2 * SLOT;
    unsigned long long rdy03 = 0;
    unsigned rdy4 = 0;
    bool ok = true;
    constexpr unsigned C0 = 2 * NPUSH, NITEM = (32 - C0) * 20;
    const unsigned qoff = (unsigned)(QCTR_OFF - FLAG_OFF) + ((call & 1) * 8 + plink) * 128;
    auto grab = [&]() {
      asm volatile("v_mov_b32 v200, 1\n\ts_nop 4\n\tbuffer_atomic_add v200, %0, %1, %2 offen sc0"
                   :: "v"(l0), "s"(rfs), "s"(qoff) : "v200", "memory");
    };
    auto ticket = [&]() -> unsigned {
      unsigned v;
      asm volatile("v_mov_b32 %0, v200" : "=v"(v) :: "v200", "memory");
      return __builtin_amdgcn_readfirstlane(v);
    };
    auto need = [&](unsigned n) {
      const unsigned cq = n / 20, c = C0 + cq, ui = (n - cq * 20) % 5, c2 = c >> 1;
      for (Spin sp;;) {
        const unsigned m = ui == 4 ? rdy4 : (unsigned)(rdy03 >> (16 * ui));
        if (((m >> c2) & 1) || !ok) return;
        const unsigned u16 = (ue >> (4 * ui)) & 15;
        const unsigned v = lane < 16 ? ld_sys(me + READY_OFF + (size_t)(src * 16 + u16) * 128 + lane * 4) : 0u;
        const unsigned mk = (unsigned)__ballot(v >= base + 1) & 0xffffu;
        if (ui == 4) rdy4 = mk; else rdy03 |= (unsigned long long)mk << (16 * ui);
        if ((mk >> c2) & 1) return;
        if (sp.expired(me, call)) ok = false;
      }
    };
    const unsigned vpay = lane * 16, vlast = lane < 56 ? lane * 16 : 0x40000000u;
    auto load = [&](unsigned slot, unsigned n) {
      const unsigned cq = n / 20, c = C0 + cq, r = n - cq * 20, w = r / 5, u16 = (ue >> (4 * (r - w * 5))) & 15;
      const unsigned so = u16 * PUNIT_BYTES + (c * 4 + w) * PBLK;
      const unsigned lds = __builtin_amdgcn_readfirstlane((unsigned)(unsigned long long)(lu8*)(ring + slot * SLOT));
      unsigned keep;
      asm volatile("s_mov_b32 %0, m0\n\ts_mov_b32 m0, %1\n\ts_nop 0\n\t"
                   "buffer_load_dwordx4 %2, %4, %5 offen sc0 sc1 lds\n\t"
                   "buffer_load_dwordx4 %2, %4, %5 offen offset:1024 sc0 sc1 lds\n\t"
                   "buffer_load_dwordx4 %3, %4, %5 offen offset:2048 sc0 sc1 lds\n\t"
                   "s_mov_b32 m0, %0"
                   : "=&s"(keep) : "s"(lds), "v"(vpay), "v"(vlast), "s"(rss), "s"(so) : "memory");
    };
    auto emit = [&](unsigned slot, unsigned n) {
      const char* sl = ring + slot * SLOT;
      const unsigned h0 = __builtin_amdgcn_readfirstlane(*(const lu32*)(sl + 2816));
      const u32x4 A = *(const lu32x4*)(sl + lane * 16), B = *(const lu32x4*)(sl + 1024 + lane * 16);
      const unsigned S = *(const lu32*)(sl + 2048 + lane * 12), K0 = *(const lu32*)(sl + 2052 + lane * 12), K1 = *(const lu32*)(sl + 2056 + lane * 12);
      const unsigned cq = n / 20, c = C0 + cq, r = n - cq * 20, w = r / 5, u16 = (ue >> (4 * (r - w * 5))) & 15;
      const int kind = u16 >> 3;
      const unsigned dst = (unsigned)stg_off(kind, src, u16 & 7) + (c * 4 + w) * BLK;
      if (h0 & 0x10000u) {
        u32x4 rr[4];
#pragma unroll
        for (int p = 0; p < 4; p++) rr[p] = __builtin_amdgcn_raw_buffer_load_b128(rraw, p * 1024 + lane * 16, dst, AUX_SYS);
#pragma unroll
        for (int p = 0; p < 4; p++) __builtin_amdgcn_raw_buffer_store_b128(rr[p], rd, p * 1024 + lane * 16, dst, AUX_AGENT);
        return;
      }
      const unsigned Md = (h0 & 0x80) ? 0x80808080u : 0u;
      const unsigned TG = (((h0 & 0x7f) * 0x01010101u | 0x80808080u) - 0x03020100u) | 0x80808080u;
      u32x4 o[4];
#pragma unroll
      for (int i = 0; i < 8; i++) {
        const unsigned Lp = i < 4 ? A[i] : B[i - 4];
        const unsigned code = ((i < 4 ? K0 : K1) >> (2 * (i & 3))) & 0x03030303u;
        const unsigned x = perm(TG, TG, code) - ((~Lp & Md) >> 7);
        const unsigned hi = (x & 0x7f7f7f7fu) | ((S << (7 - i)) & 0x80808080u);
        const unsigned L = Lp ^ Md;
        o[i >> 1][2 * (i & 1)] = perm(hi, L, 0x05010400u);
        o[i >> 1][2 * (i & 1) + 1] = perm(hi, L, 0x07030602u);
      }
      const unsigned cnt = (h0 >> 8) & 0xff;
      if (cnt == 0) {
#pragma unroll
        for (int p = 0; p < 4; p++)
          __builtin_amdgcn_raw_buffer_store_b128(o[p], rd, p * 1024 + (kind ? (wperm ^ (p * 16)) : lane * 16), dst, AUX_AGENT);
      } else {
#pragma unroll
        for (int p = 0; p < 4; p++) *(lu32x4*)(scr + p * 1024 + (kind ? (wperm ^ (p * 16)) : lane * 16)) = o[p];
        if (lane < (int)cnt) {
          const unsigned idx = *(const lu16*)(sl + 2816 + 4 + 2 * lane);
          *(lu8*)(scr + 2 * (idx & 0x7ff) + 1) = *(const lu8*)(sl + 2816 + 4 + 2 * NEXC + lane);
        }
#pragma unroll
        for (int p = 0; p < 4; p++) o[p] = *(const lu32x4*)(scr + p * 1024 + lane * 16);
#pragma unroll
        for (int p = 0; p < 4; p++) __builtin_amdgcn_raw_buffer_store_b128(o[p], rd, p * 1024 + lane * 16, dst, AUX_AGENT);
      }
    };
    auto arrive = [&](unsigned n) {
      const unsigned cq = n / 20, c = C0 + cq, r = n - cq * 20, w = r / 5, u16 = (ue >> (4 * (r - w * 5))) & 15;
      const unsigned so = (unsigned)(ARR_OFF - FLAG_OFF) + (src * 16 + u16) * 256 + (c * 2 + (w >> 1)) * 4;
      asm volatile("buffer_atomic_add %0, %1, %2, %3 offen" :: "v"(1u), "v"(l0), "s"(rfs), "s"(so) : "memory");
    };
    unsigned cur = ticket();
    unsigned nxt = cur + 1, prev = NITEM, slot = 0;
    grab();
    if (cur < NITEM) {
      need(cur);
      load(0, cur);
    }
    while (cur < NITEM) {
      if (nxt < NITEM) {
        need(nxt);
        load(slot ^ 1, nxt);
        asm volatile("s_waitcnt vmcnt(3)" ::: "memory");
      } else {
        asm volatile("s_waitcnt vmcnt(0)" ::: "memory");
      }
      if (prev < NITEM) arrive(prev);
      const unsigned nn = ticket();
      grab();
      emit(slot, cur);
      prev = cur; cur = nxt; nxt = nn; slot ^= 1;
    }
    asm volatile("s_waitcnt vmcnt(0)" ::: "memory");
    if (prev < NITEM) arrive(prev);
    return;
  }

  const int t1 = (te >> 14) & 7, i1 = (te >> 11) & 7, t2 = (te >> 8) & 7, i2 = (te >> 5) & 7, tu = (te >> 2) & 7, pj = te & 3;
  const int srcW = rank ^ tu;
  char* const donep = a.ws[srcW & 7] + DONE_OFF + b * 4;
  lu32* sw = (lu32*)(smem + NBUF * STAGE_BYTES);
  if (wv >= 4) {
    const int lw = wv - 4;
    const int kind = lw >> 1;
    const int src = lw == 0 ? rank ^ t1 : (lw == 1 ? rank ^ t2 : srcW);
    const int unit = lw == 0 ? i1 : (lw == 1 ? i2 : 2 * pj + (lw & 1));
    const char* g = me + stg_off(kind, src, unit) + lane * 16;
    const bool loc = src == rank;
    const char* pollp = loc ? me + READY_OFF + (size_t)(((src & 7) * 2 + kind) * 8 + unit) * 128
                            : me + ARR_OFF + (size_t)(((src & 7) * 2 + kind) * 8 + unit) * 256;
    const unsigned pthr = loc ? base + 1 : 2 * call;
    const int nword = loc ? 16 : 64;
    auto eval = [&](int q, unsigned v) -> int {
      const int w0 = loc ? q >> 2 : q;
      const unsigned long long nr = __ballot(lane < 16 && w0 + lane < nword && v < pthr);
      const int f = __builtin_ctz((unsigned)nr | 0x10000u);
      return min(64, loc ? 4 * (w0 + f) : w0 + f);
    };
    auto wordp = [&](int q) { return pollp + min((loc ? q >> 2 : q) + (lane & 15), nword - 1) * 4; };
    int avail = 0, pend = -1;
    int nb = 0;
    for (int s = -PF; s < 64; s++) {
      __builtin_amdgcn_s_barrier();
      asm volatile("" ::: "memory");
      if (s + PF < 64) {
        const int q = s + PF;
        if (q >= avail) {
          for (Spin sp;;) {
            avail = eval(q, ld_sys(wordp(q)));
            if (avail > q) break;
            if (sp.expired(me, call)) { avail = 64; break; }
          }

        }
        const unsigned* gp = (const unsigned*)(g + (size_t)q * (2 * BLK));
        unsigned* lp = (unsigned*)(smem + nb * STAGE_BYTES + lw * (2 * BLK));
        nb = nb == NBUF - 1 ? 0 : nb + 1;
        __builtin_amdgcn_global_load_lds(gp, lp, 16, 0, AUX_SYS);
        __builtin_amdgcn_global_load_lds(gp, lp, 16, 1024, AUX_SYS);
        __builtin_amdgcn_global_load_lds(gp, lp, 16, 2048, AUX_SYS);
        __builtin_amdgcn_global_load_lds(gp, lp, 16, 3072, AUX_SYS);
        __builtin_amdgcn_global_load_lds(gp + 1024, lp + 1024, 16, 0, AUX_SYS);
        __builtin_amdgcn_global_load_lds(gp + 1024, lp + 1024, 16, 1024, AUX_SYS);
        __builtin_amdgcn_global_load_lds(gp + 1024, lp + 1024, 16, 2048, AUX_SYS);
        __builtin_amdgcn_global_load_lds(gp + 1024, lp + 1024, 16, 3072, AUX_SYS);
        if ((q & 1) == 0) {
          if (avail < 64 && avail <= q + 2 * LOOK + 1) {
            asm volatile("global_load_dword v200, %0, off sc0 sc1" :: "v"(wordp(avail)) : "v200", "memory");
            pend = avail;
          }
        } else if (pend >= 0) {
          unsigned pv;
          asm volatile("s_waitcnt vmcnt(8)\n\tv_mov_b32 %0, v200" : "=v"(pv) :: "v200", "memory");
          avail = max(avail, eval(pend, pv));
          pend = -1;
        }
        if (PF == 4) asm volatile("s_waitcnt vmcnt(24)" ::: "memory");
        else if (PF == 3) asm volatile("s_waitcnt vmcnt(16)" ::: "memory");
        else asm volatile("s_waitcnt vmcnt(8)" ::: "memory");
      } else {
        const int rem = 62 - s;
        if (rem >= 2) asm volatile("s_waitcnt vmcnt(16)" ::: "memory");
        else if (rem == 1) asm volatile("s_waitcnt vmcnt(8)" ::: "memory");
        else asm volatile("s_waitcnt vmcnt(0)" ::: "memory");
      }
    }
    if (lw == 0 && lane == 0) sw[4] = 0;
    asm volatile("s_waitcnt lgkmcnt(0)" ::: "memory");
    __builtin_amdgcn_s_barrier();
    if (lw == 0) {
      const char* rp = me + RES_OFF + (b & 7) * 128;
      for (Spin sp;;) {
        u32x2 r;
        unsigned tm;
        asm volatile("global_load_dwordx2 %0, %2, off sc1\n\tglobal_load_dword %1, %3, off sc1\n\ts_waitcnt vmcnt(0)"
                     : "=&v"(r), "=&v"(tm) : "v"(rp), "v"(me + TMO_OFF) : "memory");
        const bool fin = __builtin_amdgcn_readfirstlane(r[1]) >= call;
        bool bad = __builtin_amdgcn_readfirstlane(tm) == call || (fin && __builtin_amdgcn_readfirstlane(r[0]) == 0x7fffffffu);
        if (!fin && !bad && sp.expired(me, call)) bad = true;
        if (fin || bad) {
          if (lane == 0) {
            sw[5] = r[0];
            sw[6] = bad;
            __hip_atomic_store(sw + 4, 1u, __ATOMIC_RELEASE, __HIP_MEMORY_SCOPE_WORKGROUP);
          }
          break;
        }
      }
    }
    return;
  }
  const int xu = wv >> 1, wu = wv & 1;
  f32x4 acc[4][4];
#pragma unroll
  for (int f = 0; f < 4; f++)
#pragma unroll
    for (int j = 0; j < 4; j++) acc[f][j] = f32x4{0.f, 0.f, 0.f, 0.f};
  const unsigned xoff = xu * (2 * BLK) + lane * 16;
  unsigned woff[4];
  {
    const int r = lane & 3, q = (lane >> 2) & 3, kq = lane >> 4;
#pragma unroll
    for (int j = 0; j < 4; j++) woff[j] = (2 + wu) * (2 * BLK) + r * 1024 + kq * 256 + q * 64 + ((j ^ r) * 16);
  }
  {
    bf16x8 xa[4], wa[4], xb[4], wb[4];
#pragma unroll
    for (int f = 0; f < 4; f++) { xb[f] = bf16x8{}; wb[f] = bf16x8{}; }
    for (int s = -PF; s < 0; s++) __builtin_amdgcn_s_barrier();
    int cb = 0;
#define ILV() do { _Pragma("unroll") for (int q_ = 0; q_ < 8; q_++) { __builtin_amdgcn_sched_group_barrier(0x008, 1, 0); \
    __builtin_amdgcn_sched_group_barrier(0x100, 1, 0); } __builtin_amdgcn_sched_group_barrier(0x008, 8, 0); } while (0)
    for (int s = 0; s < 64; s++) {
      asm volatile("s_waitcnt lgkmcnt(0)" ::: "memory");
      __builtin_amdgcn_s_barrier();
      asm volatile("" ::: "memory");

      const char* sb = smem + cb * STAGE_BYTES;
      cb = cb == NBUF - 1 ? 0 : cb + 1;
      __builtin_amdgcn_sched_barrier(0);
      wa[0] = *(const bf16x8*)(sb + woff[0]); xa[0] = *(const bf16x8*)(sb + xoff);
#pragma unroll
      for (int f = 1; f < 4; f++) wa[f] = *(const bf16x8*)(sb + woff[f]);
#pragma unroll
      for (int f = 1; f < 4; f++) xa[f] = *(const bf16x8*)(sb + xoff + f * 1024);
#pragma unroll
      for (int f = 0; f < 4; f++)
#pragma unroll
        for (int j = 0; j < 4; j++) acc[f][j] = __builtin_amdgcn_mfma_f32_16x16x32_bf16(wb[j], xb[f], acc[f][j], 0, 0, 0);
      ILV();
      __builtin_amdgcn_sched_barrier(0);
      wb[0] = *(const bf16x8*)(sb + woff[0] + BLK); xb[0] = *(const bf16x8*)(sb + xoff + BLK);
#pragma unroll
      for (int f = 1; f < 4; f++) wb[f] = *(const bf16x8*)(sb + woff[f] + BLK);
#pragma unroll
      for (int f = 1; f < 4; f++) xb[f] = *(const bf16x8*)(sb + xoff + BLK + f * 1024);
#pragma unroll
      for (int f = 0; f < 4; f++)
#pragma unroll
        for (int j = 0; j < 4; j++) acc[f][j] = __builtin_amdgcn_mfma_f32_16x16x32_bf16(wa[j], xa[f], acc[f][j], 0, 0, 0);
      ILV();
      __builtin_amdgcn_sched_barrier(0);
    }
#pragma unroll
    for (int f = 0; f < 4; f++)
#pragma unroll
      for (int j = 0; j < 4; j++) acc[f][j] = __builtin_amdgcn_mfma_f32_16x16x32_bf16(wb[j], xb[f], acc[f][j], 0, 0, 0);
  }
  float mf = 0.f;
#pragma unroll
  for (int f = 0; f < 4; f++)
#pragma unroll
    for (int j = 0; j < 4; j++) {
      const f32x4 t = acc[f][j];
      mf = fmaxf(fmaxf(mf, fabsf(t[0])), fabsf(t[1]));
      mf = fmaxf(fmaxf(mf, fabsf(t[2])), fabsf(t[3]));
    }
  unsigned m = wave_max_u32(__float_as_uint(mf));
  if (lane == 0) sw[wv] = m;
  asm volatile("s_waitcnt lgkmcnt(0)" ::: "memory");
  __builtin_amdgcn_s_barrier();
  asm volatile("" ::: "memory");
  m = max(max(sw[0], sw[1]), max(sw[2], sw[3]));
  if (tid < NR) {
    gu64* g = (gu64*)(wsl + AMAX_OFF + (size_t)(rank * NGEMM + b) * 8);
    __hip_atomic_store(g, ((unsigned long long)call << 32) | m, __ATOMIC_RELAXED, __HIP_MEMORY_SCOPE_SYSTEM);
  }
  while (__builtin_amdgcn_readfirstlane(__hip_atomic_load(sw + 4, __ATOMIC_ACQUIRE, __HIP_MEMORY_SCOPE_WORKGROUP)) == 0)
    __builtin_amdgcn_s_sleep(1);
  const float inv = 448.0f / __uint_as_float(__builtin_amdgcn_readfirstlane(sw[5]));
  const bool bad = __builtin_amdgcn_readfirstlane(sw[6]) != 0;
  {
    const int tx = xu == 0 ? t1 : t2, ix = xu == 0 ? i1 : i2;
    unsigned char* optr = outp + (size_t)((rank ^ tx) * SH + ix * 64 + (lane & 15)) * SH + pj * 128 + wu * 64 + (lane >> 4) * 16;
#pragma unroll
    for (int f = 0; f < 4; f++) {
      u32x4 q;
#pragma unroll
      for (int j = 0; j < 4; j++) {
        int w = __builtin_amdgcn_cvt_pk_fp8_f32(acc[f][j][0] * inv, acc[f][j][1] * inv, 0, false);
        w = __builtin_amdgcn_cvt_pk_fp8_f32(acc[f][j][2] * inv, acc[f][j][3] * inv, w, true);
        q[j] = bad ? 0x7f7f7f7fu : (unsigned)w;
      }
      if (tu != 0) asm volatile("global_store_dwordx4 %0, %1, off sc0 sc1\n\ts_nop 1" ::"v"(optr + (size_t)f * 16 * SH), "v"(q) : "memory");
      else *(u32x4*)(optr + (size_t)f * 16 * SH) = q;
    }
  }
  if (tu != 0) {
    asm volatile("s_waitcnt vmcnt(0)" ::: "memory");
    __builtin_amdgcn_s_barrier();
    if (tid == 0) st_sys(donep, call);
  }
}
}

extern "C" void kernel_launch(int rank, int nranks, ncclComm_t comm, hipStream_t stream, void* const* d_in,
                              const int* in_sizes, int n_in, void* d_out, int out_size, const int* in_dtypes,
                              int out_dtype, void* const* d_out_all, void* const* d_ws_all, size_t ws_bytes) {
  (void)comm; (void)in_sizes; (void)n_in; (void)d_out; (void)out_size; (void)in_dtypes; (void)out_dtype;
  if (nranks != NR || rank < 0 || rank >= NR || ws_bytes < WS_NEED) {
    fprintf(stderr, "unsupported configuration\n");
    return;
  }
  Args a;
  a.x = (const float*)d_in[0];
  a.w = (const float*)d_in[1];
  for (int p = 0; p < NR; p++) {
    a.ws[p] = (char*)d_ws_all[p];
    a.out[p] = (unsigned char*)d_out_all[p];
  }
  a.rank = rank;
  a.pad = 0;
  fused_ag_gemm<<<NWG, 512, 0, stream>>>(a);
  hipError_t e = hipGetLastError();
  if (e != hipSuccess) fprintf(stderr, "launch error %s\n", hipGetErrorString(e));
}
```

```cpp
#include <hip/hip_runtime.h>
#include <rccl/rccl.h>
#include <cstdio>
#include <cstdint>

typedef unsigned u32x4 __attribute__((ext_vector_type(4)));
typedef unsigned u32x3 __attribute__((ext_vector_type(3)));
typedef unsigned u32x2 __attribute__((ext_vector_type(2)));
typedef float f32x4 __attribute__((ext_vector_type(4)));
typedef __bf16 bf16x8 __attribute__((ext_vector_type(8)));
typedef __bf16 bf16x2 __attribute__((ext_vector_type(2)));
typedef float f32x2 __attribute__((ext_vector_type(2)));

namespace {
constexpr int PPL = 13;
constexpr int NPW = 2;
constexpr int NPWN = 8;
constexpr int NPUSH = 1;
constexpr int LOOK = 1;
constexpr int STAG0 = 40;
constexpr int STAGGER = 10;
constexpr int NBUF = 4;
constexpr int PF = NBUF - 1;
constexpr unsigned long long DEADLINE = 180000000ull;
constexpr int NR = 8;
constexpr int KDIM = 4096;
constexpr int SH = 512;
constexpr unsigned BLK = 4096;
constexpr unsigned UNIT_BYTES = 128 * BLK;
constexpr unsigned PBLK = 2944;
constexpr unsigned PUNIT_BYTES = 128 * PBLK;
constexpr unsigned NEXC = 41;
constexpr unsigned UNIT_STRIDE = UNIT_BYTES + 4352;
constexpr size_t STG_KIND_BYTES = (size_t)NR * 8 * UNIT_STRIDE;
constexpr size_t EXP_OFF = 2 * STG_KIND_BYTES;
constexpr size_t FLAG_OFF = EXP_OFF + (size_t)16 * PUNIT_BYTES;
constexpr size_t READY_OFF = FLAG_OFF;
constexpr size_t ARR_OFF = READY_OFF + NR * 16 * 128;
constexpr size_t AMAX_OFF = ARR_OFF + NR * 16 * 256;
constexpr size_t DONE_OFF = AMAX_OFF + 8 * 1024 + 1024;
constexpr size_t CTR_OFF = DONE_OFF + 1024;
constexpr size_t TMO_OFF = CTR_OFF + 128;
constexpr size_t QCTR_OFF = TMO_OFF + 128;
constexpr size_t RES_OFF = QCTR_OFF + 16 * 128;
constexpr size_t END_OFF = RES_OFF + 8 * 128;
constexpr size_t WS_NEED = END_OFF;
static_assert(FLAG_OFF % 128 == 0, "align");
constexpr int NGEMM = 132, NPULL = 7 * PPL, NWG = 256;
constexpr unsigned STAGE_BYTES = 8 * BLK;

__constant__ unsigned TILE_TAB[NGEMM] = {70845,113868,13156,68759,12015,77666,2120,76939,69473,70498,13505,55499,73914,49007,75916,75893,69764,113793,10434,13527,73775,77667,17452,4195,75873,4983,2270,70699,10240,48320,12140,121077,129,82787,81922,81990,82810,5167,10389,12451,8382,83009,68802,8384,112806,7386,18155,76985,70814,112866,77885,9286,112823,120928,113832,13461,75876,73987,10401,76928,8399,8954,121000,69849,94,4321,11460,70856,67616,4352,67609,3,73764,71395,47170,13158,69850,67674,8328,78933,5181,4300,82980,83147,73766,79023,2219,49017,68798,2284,48350,48326,122746,75834,10393,73849,57086,83084,10461,83008,114400,9251,112917,47189,122596,74882,81949,68872,83023,78563,10091,6155,9981,1100,2305,76535,8247,8290,3125,77913,74621,14371,56033,13448,55520,2306,81977,4171,4324,69911,17694,68774};
__constant__ unsigned PULL_TAB[8] = {0, 0xdc980, 0xfe987, 0xfe983, 0x65421, 0xfeba0, 0xdcba7, 0xdcba3};
constexpr unsigned long long DEST_MASK_X = 0x4511111189111123ull, DEST_MASK_W = 0x2d2dc3c3e1e10f0full;

struct Args {
  const float* x;
  const float* w;
  char* ws[NR];
  unsigned char* out[NR];
  int rank;
  int pad;
};

typedef __attribute__((address_space(1))) unsigned gu32;
typedef __attribute__((address_space(1))) unsigned long long gu64;
typedef __attribute__((address_space(3))) unsigned lu32;
typedef __attribute__((address_space(3))) unsigned char lu8;
typedef __attribute__((address_space(3))) unsigned short lu16;
typedef __attribute__((address_space(3))) u32x4 lu32x4;

__device__ __forceinline__ unsigned ld_sys(const void* p) { return __hip_atomic_load((const gu32*)p, __ATOMIC_RELAXED, __HIP_MEMORY_SCOPE_SYSTEM); }
__device__ __forceinline__ void st_sys(void* p, unsigned v) { __hip_atomic_store((gu32*)p, v, __ATOMIC_RELAXED, __HIP_MEMORY_SCOPE_SYSTEM); }
__device__ __forceinline__ unsigned ld_agent(const void* p) { return __hip_atomic_load((const gu32*)p, __ATOMIC_RELAXED, __HIP_MEMORY_SCOPE_AGENT); }
__device__ __forceinline__ void st_agent(void* p, unsigned v) { __hip_atomic_store((gu32*)p, v, __ATOMIC_RELAXED, __HIP_MEMORY_SCOPE_AGENT); }

__device__ __forceinline__ __amdgpu_buffer_rsrc_t mkrsrc(const void* p, unsigned bytes) {
  unsigned long long a = (unsigned long long)p;
  unsigned lo = __builtin_amdgcn_readfirstlane((unsigned)a), hi = __builtin_amdgcn_readfirstlane((unsigned)(a >> 32));
  void* q = (void*)(((unsigned long long)hi << 32) | lo);
  return __builtin_amdgcn_make_buffer_rsrc(q, 0, bytes, 0x00020000);
}
__device__ __forceinline__ u32x4 mkrsrc_words(const void* p, unsigned bytes) {
  unsigned long long a = (unsigned long long)p;
  return u32x4{(unsigned)__builtin_amdgcn_readfirstlane((unsigned)a), (unsigned)__builtin_amdgcn_readfirstlane((unsigned)(a >> 32)) & 0xffffu,
               bytes, 0x00020000u};
}
__device__ __forceinline__ size_t stg_off(int kind, int src, int unit) {
  return (size_t)kind * STG_KIND_BYTES + (size_t)(src * 8 + unit) * UNIT_STRIDE;
}
__device__ __forceinline__ unsigned cvt2(float a, float b) {
  f32x2 v = {a, b};
  bf16x2 r = __builtin_convertvector(v, bf16x2);
  return __builtin_bit_cast(unsigned, r);
}
__device__ __forceinline__ unsigned perm(unsigned hi, unsigned lo, unsigned sel) { return __builtin_amdgcn_perm(hi, lo, sel); }
__device__ __forceinline__ unsigned wave_max_u32(unsigned v) {
  v = max(v, (unsigned)__builtin_amdgcn_update_dpp(0, (int)v, 0x111, 0xf, 0xf, false));
  v = max(v, (unsigned)__builtin_amdgcn_update_dpp(0, (int)v, 0x112, 0xf, 0xf, false));
  v = max(v, (unsigned)__builtin_amdgcn_update_dpp(0, (int)v, 0x114, 0xf, 0xf, false));
  v = max(v, (unsigned)__builtin_amdgcn_update_dpp(0, (int)v, 0x118, 0xf, 0xf, false));
  v = max(v, (unsigned)__builtin_amdgcn_update_dpp(0, (int)v, 0x142, 0xa, 0xf, false));
  v = max(v, (unsigned)__builtin_amdgcn_update_dpp(0, (int)v, 0x143, 0xc, 0xf, false));
  return (unsigned)__builtin_amdgcn_readlane((int)v, 63);
}

struct Spin {
  unsigned it = 0;
  unsigned long long t0 = 0;
  __device__ __forceinline__ bool expired(char* me, unsigned call) {
    if (it == 0) t0 = __builtin_amdgcn_s_memrealtime();
    if ((++it & 15) == 0) {
      if (ld_agent(me + TMO_OFF) == call || __builtin_amdgcn_s_memrealtime() - t0 > DEADLINE) {
        st_agent(me + TMO_OFF, call);
        return true;
      }
    }
    __builtin_amdgcn_s_sleep(1);
    return false;
  }
};
constexpr int AUX_SYS = 17, AUX_AGENT = 16;

__device__ __forceinline__ unsigned piece_off(int kind, int p, int lane) {
  const int kq = lane >> 4, q = (lane >> 2) & 3, j = lane & 3;
  return kind == 0 ? p * 1024 + lane * 16 : p * 1024 + kq * 256 + q * 64 + ((j ^ p) * 16);
}

__global__ __launch_bounds__(512, 2) void fused_ag_gemm(Args a) {
  __shared__ __attribute__((aligned(16))) char smem[NBUF * STAGE_BYTES + 128];
  const int tid = threadIdx.x, lane = tid & 63;
  const int wv = __builtin_amdgcn_readfirstlane(tid >> 6);
  const int bx = blockIdx.x;
  const int b = bx < 16 * NPUSH ? NGEMM + NPULL + bx
                                : (bx < 16 * NPUSH + NPULL ? NGEMM + bx - 16 * NPUSH : (bx < 16 * NPUSH + NPULL + NGEMM ? bx - 16 * NPUSH - NPULL : bx));
  constexpr int NH0 = 16 * NPUSH;
  const int task = b < NGEMM ? b + NH0 + NPULL
                             : (b < NGEMM + NPULL ? b - NGEMM + NH0 : (b - (NGEMM + NPULL) < NH0 ? b - (NGEMM + NPULL) : b));
  const int c16 = task >> 4, u16 = task & 15, kind = u16 >> 3, unit = u16 & 7;
  const int ks = c16 * 8 + wv;
  const int kq = lane >> 4, l15 = lane & 15;
  if (c16 > NPUSH) {
    const unsigned long long tstart = __builtin_amdgcn_s_memrealtime();
    while (__builtin_amdgcn_s_memrealtime() - tstart < (unsigned long long)(STAG0 + (c16 - NPUSH - 1) * STAGGER)) __builtin_amdgcn_s_sleep(1);
  }
  f32x4 v[8];
  {
    const float* px = a.x + (size_t)(unit * 64 + l15) * KDIM + ks * 32 + 4 * kq;
    const float* pw = a.w + (size_t)(ks * 32 + 4 * kq) * SH + unit * 64 + 4 * l15;
    const float* p = kind == 0 ? px : pw;
#pragma unroll
    for (int i = 0; i < 8; i++) {
      const unsigned ox = (i >> 1) * 16 * KDIM + (i & 1) * 16, ow = ((i & 3) + (i >> 2) * 16) * SH;
      v[i] = *(const f32x4*)(p + (kind == 0 ? ox : ow));
    }
  }
  asm volatile("" ::: "memory");
  const int rank = a.rank;
  char* me = a.ws[rank];
  const unsigned ctr_v = ld_agent(me + CTR_OFF);
  const unsigned te = TILE_TAB[b < NGEMM ? b : 0];
  const bool is_pull = b >= NGEMM && b < NGEMM + NPULL;
  const int plink = is_pull ? 1 + (b - NGEMM) / PPL : 0;
  const int npw = is_pull && (b - NGEMM) % PPL >= NPWN ? NPW - 1 : NPW;
  const unsigned ue = PULL_TAB[plink];
  char* const wsl = a.ws[lane & 7];
  unsigned char* const outp = a.out[(rank ^ (int)(te >> 2)) & 7];
  const u32x4 rfs = mkrsrc_words(me + FLAG_OFF, (unsigned)(END_OFF - FLAG_OFF));
  const unsigned l0 = lane == 0 ? 0u : 0x40000000u;
  unsigned call, base;
  {
    const __amdgpu_buffer_rsrc_t rd = mkrsrc(me + stg_off(kind, rank, unit), UNIT_BYTES);
    call = __builtin_amdgcn_readfirstlane(ctr_v) + 1;
    base = call * 64;
    if (is_pull && wv < npw) {
      const unsigned qoff = (unsigned)(QCTR_OFF - FLAG_OFF) + ((call & 1) * 8 + plink) * 128;
      asm volatile("v_mov_b32 v200, 2\n\ts_nop 4\n\tbuffer_atomic_add v200, %0, %1, %2 offen sc0"
                   :: "v"(l0), "s"(rfs), "s"(qoff) : "v200", "memory");
    }
    u32x4 o[4];
    if (kind == 0) {
#pragma unroll
      for (int f = 0; f < 4; f++)
        o[f] = u32x4{cvt2(v[2 * f][0], v[2 * f][1]), cvt2(v[2 * f][2], v[2 * f][3]),
                     cvt2(v[2 * f + 1][0], v[2 * f + 1][1]), cvt2(v[2 * f + 1][2], v[2 * f + 1][3])};
    } else {
#pragma unroll
      for (int r = 0; r < 4; r++)
        o[r] = u32x4{cvt2(v[0][r], v[1][r]), cvt2(v[2][r], v[3][r]), cvt2(v[4][r], v[5][r]), cvt2(v[6][r], v[7][r])};
    }
    const unsigned dmask = (unsigned)((kind ? DEST_MASK_W : DEST_MASK_X) >> (8 * unit)) & 0xff;
    if (c16 < NPUSH) {
#pragma unroll
      for (int p = 1; p < NR; p++)
        if ((dmask >> p) & 1) {
          const __amdgpu_buffer_rsrc_t rq = mkrsrc(a.ws[rank ^ p] + stg_off(kind, rank, unit), UNIT_BYTES);
#pragma unroll
          for (int i = 0; i < 4; i++)
            __builtin_amdgcn_raw_buffer_store_b128(o[i], rq, ks * BLK + piece_off(kind, i, lane), 0, AUX_SYS);
        }
    } else {
      float mx = 0.f;
#pragma unroll
      for (int i = 0; i < 8; i++) {
        mx = fmaxf(mx, fmaxf(fabsf(v[i][0]), fabsf(v[i][1])));
        mx = fmaxf(mx, fmaxf(fabsf(v[i][2]), fabsf(v[i][3])));
      }
      const unsigned mxb = wave_max_u32(__float_as_uint(mx));
      const unsigned emax = (cvt2(__uint_as_float(mxb), 0.f) >> 7) & 0xff;
      const float thr1 = __uint_as_float(emax << 23), thr2 = thr1 * 0.5f;
      unsigned n1 = 0, n2 = 0;
#pragma unroll
      for (int i = 0; i < 8; i++) {
        n1 += __builtin_popcountll(__ballot(fabsf(v[i][0]) >= thr1)) + __builtin_popcountll(__ballot(fabsf(v[i][2]) >= thr1));
        n2 += __builtin_popcountll(__ballot(fabsf(v[i][0]) >= thr2)) + __builtin_popcountll(__ballot(fabsf(v[i][2]) >= thr2));
      }
      const int top = (int)emax - (n1 <= 10 ? 1 : 0) - (n2 <= 10 ? 1 : 0);
      const unsigned delta = (top & 1) ^ 1, tp = ((top + (int)delta) >> 1) & 0x7f;
      const unsigned Md = delta ? 0x80808080u : 0u;
      const unsigned b80 = tp * 0x01010101u | 0x80808080u;
      const unsigned T = (b80 - 0x03020100u) & 0x7f7f7f7fu;
      unsigned LP[8], HH[8], S = 0, K0 = 0, K1 = 0, M = 0;
#pragma unroll
      for (int i = 0; i < 8; i++) {
        const unsigned d0 = o[i >> 1][2 * (i & 1)], d1 = o[i >> 1][2 * (i & 1) + 1];
        const unsigned L = perm(d1, d0, 0x06040200u), H = perm(d1, d0, 0x07050301u);
        const unsigned pair = ((H & 0x7f7f7f7fu) + ((L & Md) >> 7)) & 0x7f7f7f7fu;
        const unsigned code = (b80 - pair) & 0x03030303u;
        const unsigned mis = perm(T, T, code) ^ pair;
        LP[i] = L ^ Md;
        HH[i] = H;
        if (i < 4) K0 |= code << (2 * i); else K1 |= code << (2 * (i - 4));
        S |= (H & 0x80808080u) >> (7 - i);
        M |= __builtin_amdgcn_udot4(((mis + 0x7f7f7f7fu) & 0x80808080u) >> 7, 0x08040201u, 0u, false) << (4 * i);
      }
      char* hsc = smem + wv * 2304;
      lu32* hdr = (lu32*)(hsc + 2048);
      const unsigned nl = __builtin_popcount(M);
      unsigned inc = nl;
      inc += __builtin_amdgcn_update_dpp(0, (int)inc, 0x111, 0xf, 0xf, false);
      inc += __builtin_amdgcn_update_dpp(0, (int)inc, 0x112, 0xf, 0xf, false);
      inc += __builtin_amdgcn_update_dpp(0, (int)inc, 0x114, 0xf, 0xf, false);
      inc += __builtin_amdgcn_update_dpp(0, (int)inc, 0x118, 0xf, 0xf, false);
      inc += __builtin_amdgcn_update_dpp(0, (int)inc, 0x142, 0xa, 0xf, false);
      inc += __builtin_amdgcn_update_dpp(0, (int)inc, 0x143, 0xc, 0xf, false);
      const unsigned cnt = (unsigned)__builtin_amdgcn_readlane((int)inc, 63);
      if (cnt > 0 && cnt <= NEXC) {
        *(lu32x4*)(hsc + lane * 32) = u32x4{HH[0], HH[1], HH[2], HH[3]};
        *(lu32x4*)(hsc + lane * 32 + 16) = u32x4{HH[4], HH[5], HH[6], HH[7]};
        unsigned at = inc - nl;
        while (__any(M != 0)) {
          if (M != 0) {
            const unsigned jv = __builtin_ctz(M);
            M &= M - 1;
            const unsigned char hi = *(const lu8*)(hsc + lane * 32 + jv);
            const unsigned idx = (piece_off(kind, jv >> 3, lane) >> 1) + (jv & 7);
            *(lu16*)((char*)hdr + 4 + 2 * at) = (unsigned short)idx;
            *(lu8*)((char*)hdr + 4 + 2 * NEXC + at) = hi;
            at++;
          }
        }
      }
      const __amdgpu_buffer_rsrc_t rp = mkrsrc(me + EXP_OFF + (size_t)u16 * PUNIT_BYTES, PUNIT_BYTES);
      const unsigned pc = ks * PBLK;
      __builtin_amdgcn_raw_buffer_store_b128(u32x4{LP[0], LP[1], LP[2], LP[3]}, rp, pc + lane * 16, 0, AUX_SYS);
      __builtin_amdgcn_raw_buffer_store_b128(u32x4{LP[4], LP[5], LP[6], LP[7]}, rp, pc + 1024 + lane * 16, 0, AUX_SYS);
      __builtin_amdgcn_raw_buffer_store_b96(u32x3{S, K0, K1}, rp, pc + 2048 + lane * 12, 0, AUX_SYS);
      unsigned hv = hdr[lane & 31];
      if (lane == 0) hv = tp | (delta << 7) | (cnt > NEXC ? 0x10000u : cnt << 8);
      if (lane < 32) __builtin_amdgcn_raw_buffer_store_b32(hv, rp, pc + 2816 + lane * 4, 0, AUX_SYS);
    }
#pragma unroll
    for (int p = 0; p < 4; p++)
      __builtin_amdgcn_raw_buffer_store_b128(o[p], rd, ks * BLK + piece_off(kind, p, lane), 0, AUX_SYS);
    asm volatile("s_waitcnt vmcnt(0)" ::: "memory");
    const int rel = rank ^ lane;
    if (c16 < NPUSH) {
      if (lane < NR && rel != 0 && ((dmask >> rel) & 1))
        __hip_atomic_fetch_add((gu32*)(wsl + ARR_OFF + (size_t)(rank * 16 + u16) * 256 + (ks >> 1) * 4), 1u, __ATOMIC_RELAXED,
                               __HIP_MEMORY_SCOPE_SYSTEM);
    }
    __builtin_amdgcn_s_barrier();
    if (wv == 0 && lane < NR && ((dmask >> rel) & 1) && (rel == 0 || c16 >= NPUSH))
      st_sys(wsl + READY_OFF + (size_t)(rank * 16 + u16) * 128 + c16 * 4, base + 1);
  }

  if (b >= NGEMM + NPULL) {
    if (b == NWG - 1) {
      {
        lu32* sv = (lu32*)smem;
        __builtin_amdgcn_s_barrier();
        if (tid < 3) sv[tid] = 0;
        asm volatile("s_waitcnt lgkmcnt(0)" ::: "memory");
        __builtin_amdgcn_s_barrier();
        static_assert(NR * NGEMM == 2 * (512 + 16), "granule pairs");
        const char* gp = me + AMAX_OFF + (size_t)tid * 16;
        const char* gp2 = me + AMAX_OFF + (size_t)(tid < 16 ? 512 + tid : tid) * 16;
        bool have = false, bad = false;
        unsigned gm = 0;
        for (Spin sp;;) {
          unsigned tm = 0;
          if (!have) {
            u32x4 g0, g1;
            asm volatile("global_load_dwordx4 %0, %3, off sc0 sc1\n\tglobal_load_dwordx4 %1, %4, off sc0 sc1\n\t"
                         "global_load_dword %2, %5, off sc1\n\ts_waitcnt vmcnt(0)"
                         : "=&v"(g0), "=&v"(g1), "=&v"(tm) : "v"(gp), "v"(gp2), "v"(me + TMO_OFF) : "memory");
            have = g0[1] >= call && g0[3] >= call && g1[1] >= call && g1[3] >= call;
            gm = max(max(g0[0], g0[2]), max(g1[0], g1[2]));
          }
          if (__any(tm == call)) { bad = true; break; }
          if (__all(have)) break;
          if (sp.expired(me, call)) { bad = true; break; }
        }
        const unsigned wm = wave_max_u32(gm);
        unsigned last = 0;
        if (lane == 0) {
          __hip_atomic_fetch_max(sv, wm, __ATOMIC_RELAXED, __HIP_MEMORY_SCOPE_WORKGROUP);
          if (bad) __hip_atomic_store(sv + 2, 1u, __ATOMIC_RELAXED, __HIP_MEMORY_SCOPE_WORKGROUP);
          last = __hip_atomic_fetch_add(sv + 1, 1u, __ATOMIC_ACQ_REL, __HIP_MEMORY_SCOPE_WORKGROUP);
        }
        if (__builtin_amdgcn_readfirstlane(last) == 7) {
          const unsigned am = sv[2] ? 0x7fffffffu : sv[0];
          if (lane < 8) __hip_atomic_store((gu64*)(me + RES_OFF + lane * 128), ((unsigned long long)call << 32) | am,
                                           __ATOMIC_RELAXED, __HIP_MEMORY_SCOPE_AGENT);
          if (lane < 8) st_agent(me + QCTR_OFF + (size_t)(((call + 1) & 1) * 8 + lane) * 128, 0u);
          if (lane == 0) st_agent(me + CTR_OFF, call);
        }
      }
      if (wv == 0) {
        bool ok = true;
        {
          const int l2 = lane < (NGEMM - 128) / 2 ? 64 + lane : lane;
          const bool a0 = ((TILE_TAB[2 * lane] >> 2) & 7) != 0, a1 = ((TILE_TAB[2 * lane + 1] >> 2) & 7) != 0;
          const bool a2 = ((TILE_TAB[2 * l2] >> 2) & 7) != 0, a3 = ((TILE_TAB[2 * l2 + 1] >> 2) & 7) != 0;
          for (Spin sp;;) {
            u32x2 d, e;
            asm volatile("global_load_dwordx2 %0, %2, off sc0 sc1\n\tglobal_load_dwordx2 %1, %3, off sc0 sc1\n\ts_waitcnt vmcnt(0)"
                         : "=&v"(d), "=&v"(e) : "v"(me + DONE_OFF + lane * 8), "v"(me + DONE_OFF + l2 * 8) : "memory");
            if (__all((!a0 || d[0] >= call) && (!a1 || d[1] >= call) && (!a2 || e[0] >= call) && (!a3 || e[1] >= call))) break;
            if (sp.expired(me, call)) { ok = false; break; }
          }
        }
        if (!ok && lane == 0) a.out[rank][(size_t)(rank * SH + 384) * SH + 128] = 0x7f;
      }
    }
    return;
  }
  if (is_pull) {
    if (wv >= npw) return;
    const unsigned src = (unsigned)(rank ^ plink);
    if (src >= NR) return;
    const __amdgpu_buffer_rsrc_t rraw = mkrsrc(a.ws[src], 2 * STG_KIND_BYTES);
    const __amdgpu_buffer_rsrc_t rd = mkrsrc(me, 2 * STG_KIND_BYTES);
    const u32x4 rss = mkrsrc_words(a.ws[src] + EXP_OFF, 16 * PUNIT_BYTES);
    const unsigned wperm = piece_off(1, 0, lane);
    char* scr = smem + 1024 + wv * BLK;
    constexpr unsigned SLOT = 3072;
    char* ring = smem + 1024 + NPW * BLK + wv * 2 * SLOT;
    unsigned long long rdy03 = 0;
    unsigned rdy4 = 0;
    bool ok = true;
    constexpr unsigned C0 = 2 * NPUSH, NITEM = (32 - C0) * 20;
    const unsigned qoff = (unsigned)(QCTR_OFF - FLAG_OFF) + ((call & 1) * 8 + plink) * 128;
    auto grab = [&]() {
      asm volatile("v_mov_b32 v200, 1\n\ts_nop 4\n\tbuffer_atomic_add v200, %0, %1, %2 offen sc0"
                   :: "v"(l0), "s"(rfs), "s"(qoff) : "v200", "memory");
    };
    auto ticket = [&]() -> unsigned {
      unsigned v;
      asm volatile("v_mov_b32 %0, v200" : "=v"(v) :: "v200", "memory");
      return __builtin_amdgcn_readfirstlane(v);
    };
    auto need = [&](unsigned n) {
      const unsigned cq = n / 20, c = C0 + cq, ui = (n - cq * 20) % 5, c2 = c >> 1;
      for (Spin sp;;) {
        const unsigned m = ui == 4 ? rdy4 : (unsigned)(rdy03 >> (16 * ui));
        if (((m >> c2) & 1) || !ok) return;
        const unsigned u16 = (ue >> (4 * ui)) & 15;
        const unsigned v = lane < 16 ? ld_sys(me + READY_OFF + (size_t)(src * 16 + u16) * 128 + lane * 4) : 0u;
        const unsigned mk = (unsigned)__ballot(v >= base + 1) & 0xffffu;
        if (ui == 4) rdy4 = mk; else rdy03 |= (unsigned long long)mk << (16 * ui);
        if ((mk >> c2) & 1) return;
        if (sp.expired(me, call)) ok = false;
      }
    };
    const unsigned vpay = lane * 16, vlast = lane < 56 ? lane * 16 : 0x40000000u;
    auto load = [&](unsigned slot, unsigned n) {
      const unsigned cq = n / 20, c = C0 + cq, r = n - cq * 20, w = r / 5, u16 = (ue >> (4 * (r - w * 5))) & 15;
      const unsigned so = u16 * PUNIT_BYTES + (c * 4 + w) * PBLK;
      const unsigned lds = __builtin_amdgcn_readfirstlane((unsigned)(unsigned long long)(lu8*)(ring + slot * SLOT));
      unsigned keep;
      asm volatile("s_mov_b32 %0, m0\n\ts_mov_b32 m0, %1\n\ts_nop 0\n\t"
                   "buffer_load_dwordx4 %2, %4, %5 offen sc0 sc1 lds\n\t"
                   "buffer_load_dwordx4 %2, %4, %5 offen offset:1024 sc0 sc1 lds\n\t"
                   "buffer_load_dwordx4 %3, %4, %5 offen offset:2048 sc0 sc1 lds\n\t"
                   "s_mov_b32 m0, %0"
                   : "=&s"(keep) : "s"(lds), "v"(vpay), "v"(vlast), "s"(rss), "s"(so) : "memory");
    };
    auto emit = [&](unsigned slot, unsigned n) {
      const char* sl = ring + slot * SLOT;
      const unsigned h0 = __builtin_amdgcn_readfirstlane(*(const lu32*)(sl + 2816));
      const u32x4 A = *(const lu32x4*)(sl + lane * 16), B = *(const lu32x4*)(sl + 1024 + lane * 16);
      const unsigned S = *(const lu32*)(sl + 2048 + lane * 12), K0 = *(const lu32*)(sl + 2052 + lane * 12), K1 = *(const lu32*)(sl + 2056 + lane * 12);
      const unsigned cq = n / 20, c = C0 + cq, r = n - cq * 20, w = r / 5, u16 = (ue >> (4 * (r - w * 5))) & 15;
      const int kind = u16 >> 3;
      const unsigned dst = (unsigned)stg_off(kind, src, u16 & 7) + (c * 4 + w) * BLK;
      if (h0 & 0x10000u) {
        u32x4 rr[4];
#pragma unroll
        for (int p = 0; p < 4; p++) rr[p] = __builtin_amdgcn_raw_buffer_load_b128(rraw, p * 1024 + lane * 16, dst, AUX_SYS);
#pragma unroll
        for (int p = 0; p < 4; p++) __builtin_amdgcn_raw_buffer_store_b128(rr[p], rd, p * 1024 + lane * 16, dst, AUX_AGENT);
        return;
      }
      const unsigned Md = (h0 & 0x80) ? 0x80808080u : 0u;
      const unsigned TG = (((h0 & 0x7f) * 0x01010101u | 0x80808080u) - 0x03020100u) | 0x80808080u;
      u32x4 o[4];
#pragma unroll
      for (int i = 0; i < 8; i++) {
        const unsigned Lp = i < 4 ? A[i] : B[i - 4];
        const unsigned code = ((i < 4 ? K0 : K1) >> (2 * (i & 3))) & 0x03030303u;
        const unsigned x = perm(TG, TG, code) - ((~Lp & Md) >> 7);
        const unsigned hi = (x & 0x7f7f7f7fu) | ((S << (7 - i)) & 0x80808080u);
        const unsigned L = Lp ^ Md;
        o[i >> 1][2 * (i & 1)] = perm(hi, L, 0x05010400u);
        o[i >> 1][2 * (i & 1) + 1] = perm(hi, L, 0x07030602u);
      }
      const unsigned cnt = (h0 >> 8) & 0xff;
      if (cnt == 0) {
#pragma unroll
        for (int p = 0; p < 4; p++)
          __builtin_amdgcn_raw_buffer_store_b128(o[p], rd, p * 1024 + (kind ? (wperm ^ (p * 16)) : lane * 16), dst, AUX_AGENT);
      } else {
#pragma unroll
        for (int p = 0; p < 4; p++) *(lu32x4*)(scr + p * 1024 + (kind ? (wperm ^ (p * 16)) : lane * 16)) = o[p];
        if (lane < (int)cnt) {
          const unsigned idx = *(const lu16*)(sl + 2816 + 4 + 2 * lane);
          *(lu8*)(scr + 2 * (idx & 0x7ff) + 1) = *(const lu8*)(sl + 2816 + 4 + 2 * NEXC + lane);
        }
#pragma unroll
        for (int p = 0; p < 4; p++) o[p] = *(const lu32x4*)(scr + p * 1024 + lane * 16);
#pragma unroll
        for (int p = 0; p < 4; p++) __builtin_amdgcn_raw_buffer_store_b128(o[p], rd, p * 1024 + lane * 16, dst, AUX_AGENT);
      }
    };
    auto arrive = [&](unsigned n) {
      const unsigned cq = n / 20, c = C0 + cq, r = n - cq * 20, w = r / 5, u16 = (ue >> (4 * (r - w * 5))) & 15;
      const unsigned so = (unsigned)(ARR_OFF - FLAG_OFF) + (src * 16 + u16) * 256 + (c * 2 + (w >> 1)) * 4;
      asm volatile("buffer_atomic_add %0, %1, %2, %3 offen" :: "v"(1u), "v"(l0), "s"(rfs), "s"(so) : "memory");
    };
    unsigned cur = ticket();
    unsigned nxt = cur + 1, prev = NITEM, slot = 0;
    grab();
    if (cur < NITEM) {
      need(cur);
      load(0, cur);
    }
    while (cur < NITEM) {
      if (nxt < NITEM) {
        need(nxt);
        load(slot ^ 1, nxt);
        asm volatile("s_waitcnt vmcnt(3)" ::: "memory");
      } else {
        asm volatile("s_waitcnt vmcnt(0)" ::: "memory");
      }
      if (prev < NITEM) arrive(prev);
      const unsigned nn = ticket();
      grab();
      emit(slot, cur);
      prev = cur; cur = nxt; nxt = nn; slot ^= 1;
    }
    asm volatile("s_waitcnt vmcnt(0)" ::: "memory");
    if (prev < NITEM) arrive(prev);
    return;
  }

  const int t1 = (te >> 14) & 7, i1 = (te >> 11) & 7, t2 = (te >> 8) & 7, i2 = (te >> 5) & 7, tu = (te >> 2) & 7, pj = te & 3;
  const int srcW = rank ^ tu;
  char* const donep = a.ws[srcW & 7] + DONE_OFF + b * 4;
  lu32* sw = (lu32*)(smem + NBUF * STAGE_BYTES);
  if (wv >= 4) {
    const int lw = wv - 4;
    const int kind = lw >> 1;
    const int src = lw == 0 ? rank ^ t1 : (lw == 1 ? rank ^ t2 : srcW);
    const int unit = lw == 0 ? i1 : (lw == 1 ? i2 : 2 * pj + (lw & 1));
    const char* g = me + stg_off(kind, src, unit) + lane * 16;
    const bool loc = src == rank;
    const char* pollp = loc ? me + READY_OFF + (size_t)(((src & 7) * 2 + kind) * 8 + unit) * 128
                            : me + ARR_OFF + (size_t)(((src & 7) * 2 + kind) * 8 + unit) * 256;
    const unsigned pthr = loc ? base + 1 : 2 * call;
    const int nword = loc ? 16 : 64;
    auto eval = [&](int q, unsigned v) -> int {
      const int w0 = loc ? q >> 2 : q;
      const unsigned long long nr = __ballot(lane < 16 && w0 + lane < nword && v < pthr);
      const int f = __builtin_ctz((unsigned)nr | 0x10000u);
      return min(64, loc ? 4 * (w0 + f) : w0 + f);
    };
    auto wordp = [&](int q) { return pollp + min((loc ? q >> 2 : q) + (lane & 15), nword - 1) * 4; };
    int avail = 0, pend = -1;
    int nb = 0;
    for (int s = -PF; s < 64; s++) {
      __builtin_amdgcn_s_barrier();
      asm volatile("" ::: "memory");
      if (s + PF < 64) {
        const int q = s + PF;
        if (q >= avail) {
          for (Spin sp;;) {
            avail = eval(q, ld_sys(wordp(q)));
            if (avail > q) break;
            if (sp.expired(me, call)) { avail = 64; break; }
          }

        }
        const unsigned* gp = (const unsigned*)(g + (size_t)q * (2 * BLK));
        unsigned* lp = (unsigned*)(smem + nb * STAGE_BYTES + lw * (2 * BLK));
        nb = nb == NBUF - 1 ? 0 : nb + 1;
        __builtin_amdgcn_global_load_lds(gp, lp, 16, 0, AUX_SYS);
        __builtin_amdgcn_global_load_lds(gp, lp, 16, 1024, AUX_SYS);
        __builtin_amdgcn_global_load_lds(gp, lp, 16, 2048, AUX_SYS);
        __builtin_amdgcn_global_load_lds(gp, lp, 16, 3072, AUX_SYS);
        __builtin_amdgcn_global_load_lds(gp + 1024, lp + 1024, 16, 0, AUX_SYS);
        __builtin_amdgcn_global_load_lds(gp + 1024, lp + 1024, 16, 1024, AUX_SYS);
        __builtin_amdgcn_global_load_lds(gp + 1024, lp + 1024, 16, 2048, AUX_SYS);
        __builtin_amdgcn_global_load_lds(gp + 1024, lp + 1024, 16, 3072, AUX_SYS);
        if ((q & 1) == 0) {
          if (avail < 64 && avail <= q + 2 * LOOK + 1) {
            asm volatile("global_load_dword v200, %0, off sc0 sc1" :: "v"(wordp(avail)) : "v200", "memory");
            pend = avail;
          }
        } else if (pend >= 0) {
          unsigned pv;
          asm volatile("s_waitcnt vmcnt(8)\n\tv_mov_b32 %0, v200" : "=v"(pv) :: "v200", "memory");
          avail = max(avail, eval(pend, pv));
          pend = -1;
        }
        if (PF == 4) asm volatile("s_waitcnt vmcnt(24)" ::: "memory");
        else if (PF == 3) asm volatile("s_waitcnt vmcnt(16)" ::: "memory");
        else asm volatile("s_waitcnt vmcnt(8)" ::: "memory");
      } else {
        const int rem = 62 - s;
        if (rem >= 2) asm volatile("s_waitcnt vmcnt(16)" ::: "memory");
        else if (rem == 1) asm volatile("s_waitcnt vmcnt(8)" ::: "memory");
        else asm volatile("s_waitcnt vmcnt(0)" ::: "memory");
      }
    }
    if (lw == 0 && lane == 0) sw[4] = 0;
    asm volatile("s_waitcnt lgkmcnt(0)" ::: "memory");
    __builtin_amdgcn_s_barrier();
    if (lw == 0) {
      const char* rp = me + RES_OFF + (b & 7) * 128;
      for (Spin sp;;) {
        u32x2 r;
        unsigned tm;
        asm volatile("global_load_dwordx2 %0, %2, off sc1\n\tglobal_load_dword %1, %3, off sc1\n\ts_waitcnt vmcnt(0)"
                     : "=&v"(r), "=&v"(tm) : "v"(rp), "v"(me + TMO_OFF) : "memory");
        const bool fin = __builtin_amdgcn_readfirstlane(r[1]) >= call;
        bool bad = __builtin_amdgcn_readfirstlane(tm) == call || (fin && __builtin_amdgcn_readfirstlane(r[0]) == 0x7fffffffu);
        if (!fin && !bad && sp.expired(me, call)) bad = true;
        if (fin || bad) {
          if (lane == 0) {
            sw[5] = r[0];
            sw[6] = bad;
            __hip_atomic_store(sw + 4, 1u, __ATOMIC_RELEASE, __HIP_MEMORY_SCOPE_WORKGROUP);
          }
          break;
        }
      }
    }
    return;
  }
  const int xu = wv >> 1, wu = wv & 1;
  f32x4 acc[4][4];
#pragma unroll
  for (int f = 0; f < 4; f++)
#pragma unroll
    for (int j = 0; j < 4; j++) acc[f][j] = f32x4{0.f, 0.f, 0.f, 0.f};
  const unsigned xoff = xu * (2 * BLK) + lane * 16;
  unsigned woff[4];
  {
    const int r = lane & 3, q = (lane >> 2) & 3, kq = lane >> 4;
#pragma unroll
    for (int j = 0; j < 4; j++) woff[j] = (2 + wu) * (2 * BLK) + r * 1024 + kq * 256 + q * 64 + ((j ^ r) * 16);
  }
  {
    bf16x8 xa[4], wa[4], xb[4], wb[4];
#pragma unroll
    for (int f = 0; f < 4; f++) { xb[f] = bf16x8{}; wb[f] = bf16x8{}; }
    for (int s = -PF; s < 0; s++) __builtin_amdgcn_s_barrier();
    int cb = 0;
#define ILV() do { _Pragma("unroll") for (int q_ = 0; q_ < 8; q_++) { __builtin_amdgcn_sched_group_barrier(0x008, 1, 0); \
    __builtin_amdgcn_sched_group_barrier(0x100, 1, 0); } __builtin_amdgcn_sched_group_barrier(0x008, 8, 0); } while (0)
    for (int s = 0; s < 64; s++) {
      asm volatile("s_waitcnt lgkmcnt(0)" ::: "memory");
      __builtin_amdgcn_s_barrier();
      asm volatile("" ::: "memory");

      const char* sb = smem + cb * STAGE_BYTES;
      cb = cb == NBUF - 1 ? 0 : cb + 1;
      __builtin_amdgcn_sched_barrier(0);
      wa[0] = *(const bf16x8*)(sb + woff[0]); xa[0] = *(const bf16x8*)(sb + xoff);
#pragma unroll
      for (int f = 1; f < 4; f++) wa[f] = *(const bf16x8*)(sb + woff[f]);
#pragma unroll
      for (int f = 1; f < 4; f++) xa[f] = *(const bf16x8*)(sb + xoff + f * 1024);
#pragma unroll
      for (int f = 0; f < 4; f++)
#pragma unroll
        for (int j = 0; j < 4; j++) acc[f][j] = __builtin_amdgcn_mfma_f32_16x16x32_bf16(wb[j], xb[f], acc[f][j], 0, 0, 0);
      ILV();
      __builtin_amdgcn_sched_barrier(0);
      wb[0] = *(const bf16x8*)(sb + woff[0] + BLK); xb[0] = *(const bf16x8*)(sb + xoff + BLK);
#pragma unroll
      for (int f = 1; f < 4; f++) wb[f] = *(const bf16x8*)(sb + woff[f] + BLK);
#pragma unroll
      for (int f = 1; f < 4; f++) xb[f] = *(const bf16x8*)(sb + xoff + BLK + f * 1024);
#pragma unroll
      for (int f = 0; f < 4; f++)
#pragma unroll
        for (int j = 0; j < 4; j++) acc[f][j] = __builtin_amdgcn_mfma_f32_16x16x32_bf16(wa[j], xa[f], acc[f][j], 0, 0, 0);
      ILV();
      __builtin_amdgcn_sched_barrier(0);
    }
#pragma unroll
    for (int f = 0; f < 4; f++)
#pragma unroll
      for (int j = 0; j < 4; j++) acc[f][j] = __builtin_amdgcn_mfma_f32_16x16x32_bf16(wb[j], xb[f], acc[f][j], 0, 0, 0);
  }
  float mf = 0.f;
#pragma unroll
  for (int f = 0; f < 4; f++)
#pragma unroll
    for (int j = 0; j < 4; j++) {
      const f32x4 t = acc[f][j];
      mf = fmaxf(fmaxf(mf, fabsf(t[0])), fabsf(t[1]));
      mf = fmaxf(fmaxf(mf, fabsf(t[2])), fabsf(t[3]));
    }
  unsigned m = wave_max_u32(__float_as_uint(mf));
  if (lane == 0) sw[wv] = m;
  asm volatile("s_waitcnt lgkmcnt(0)" ::: "memory");
  __builtin_amdgcn_s_barrier();
  asm volatile("" ::: "memory");
  m = max(max(sw[0], sw[1]), max(sw[2], sw[3]));
  if (tid < NR) {
    gu64* g = (gu64*)(wsl + AMAX_OFF + (size_t)(rank * NGEMM + b) * 8);
    __hip_atomic_store(g, ((unsigned long long)call << 32) | m, __ATOMIC_RELAXED, __HIP_MEMORY_SCOPE_SYSTEM);
  }
  while (__builtin_amdgcn_readfirstlane(__hip_atomic_load(sw + 4, __ATOMIC_ACQUIRE, __HIP_MEMORY_SCOPE_WORKGROUP)) == 0)
    __builtin_amdgcn_s_sleep(1);
  const float inv = 448.0f / __uint_as_float(__builtin_amdgcn_readfirstlane(sw[5]));
  const bool bad = __builtin_amdgcn_readfirstlane(sw[6]) != 0;
  {
    const int tx = xu == 0 ? t1 : t2, ix = xu == 0 ? i1 : i2;
    unsigned char* optr = outp + (size_t)((rank ^ tx) * SH + ix * 64 + (lane & 15)) * SH + pj * 128 + wu * 64 + (lane >> 4) * 16;
    const bool twin = xu == 1 && t1 == t2 && i1 == i2;
#pragma unroll
    for (int f = 0; f < 4 && !twin; f++) {
      u32x4 q;
#pragma unroll
      for (int j = 0; j < 4; j++) {
        int w = __builtin_amdgcn_cvt_pk_fp8_f32(acc[f][j][0] * inv, acc[f][j][1] * inv, 0, false);
        w = __builtin_amdgcn_cvt_pk_fp8_f32(acc[f][j][2] * inv, acc[f][j][3] * inv, w, true);
        q[j] = bad ? 0x7f7f7f7fu : (unsigned)w;
      }
      if (tu != 0) asm volatile("global_store_dwordx4 %0, %1, off sc0 sc1\n\ts_nop 1" ::"v"(optr + (size_t)f * 16 * SH), "v"(q) : "memory");
      else *(u32x4*)(optr + (size_t)f * 16 * SH) = q;
    }
  }
  if (tu != 0) {
    asm volatile("s_waitcnt vmcnt(0)" ::: "memory");
    __builtin_amdgcn_s_barrier();
    if (tid == 0) st_sys(donep, call);
  }
}
}

extern "C" void kernel_launch(int rank, int nranks, ncclComm_t comm, hipStream_t stream, void* const* d_in,
                              const int* in_sizes, int n_in, void* d_out, int out_size, const int* in_dtypes,
                              int out_dtype, void* const* d_out_all, void* const* d_ws_all, size_t ws_bytes) {
  (void)comm; (void)in_sizes; (void)n_in; (void)d_out; (void)out_size; (void)in_dtypes; (void)out_dtype;
  if (nranks != NR || rank < 0 || rank >= NR || ws_bytes < WS_NEED) {
    fprintf(stderr, "unsupported configuration\n");
    return;
  }
  Args a;
  a.x = (const float*)d_in[0];
  a.w = (const float*)d_in[1];
  for (int p = 0; p < NR; p++) {
    a.ws[p] = (char*)d_ws_all[p];
    a.out[p] = (unsigned char*)d_out_all[p];
  }
  a.rank = rank;
  a.pad = 0;
  fused_ag_gemm<<<NWG, 512, 0, stream>>>(a);
  hipError_t e = hipGetLastError();
  if (e != hipSuccess) fprintf(stderr, "launch error %s\n", hipGetErrorString(e));
}
```
